```python
import numpy as np

kernel_name = "gpu_gemm_rs_m4096_k4096_n16384_e5m2e5m2_dqsilu"


ABSMAX_RTOL = 0.05

COMPUTE_DTYPE = "float32"
EPILOGUE = "dequant_silu"
IN_QUANT = None
OUT_QUANT = None
IN_FP8 = "e5m2e5m2"
IN_DTYPES = {"x": "e5m2", "w_mat": "e5m2",
             "scale_x": "float32", "scale_w": "float32"}
WORLD_SIZE = 8

def _snap_fp8(x, mant_bits, max_abs, min_normal):
    a = np.abs(x.astype(np.float64))
    _m, _e = np.frexp(a)
    q = np.where(a >= min_normal, np.ldexp(1.0, _e - 1 - mant_bits),
                 min_normal * 2.0 ** -mant_bits)
    snapped = np.minimum(np.round(a / q) * q, max_abs)
    return (np.sign(x) * snapped).astype(np.float32)

def setup_inputs():

    x = _snap_fp8(np.random.randn(4096, 4096).astype(np.float32), 2, 57344.0, 2.0 ** -14)
    w_mat = _snap_fp8(np.random.randn(4096, 16384).astype(np.float32), 2, 57344.0, 2.0 ** -14)
    scale_x = (0.02 * (1.0 + np.random.rand(1))).astype(np.float32)
    scale_w = (0.02 * (1.0 + np.random.rand(1))).astype(np.float32)
    return {"x": x, "w_mat": w_mat, "scale_x": scale_x, "scale_w": scale_w}

def reference(x, w_mat, scale_x, scale_w):
    acc = (x @ w_mat).astype(np.float32)
    y = (acc * (scale_x * scale_w)).astype(np.float32)
    return (y / (1.0 + np.exp(-np.clip(y, -60.0, 60.0)))).astype(np.float32)
```

```cpp
#include <hip/hip_runtime.h>
#include <rccl/rccl.h>
#include <cstdio>
#include <cstdint>

#define HIP_CHECK(expr) do { hipError_t _e = (expr); if (_e != hipSuccess) { \
    fprintf(stderr, "HIP error %s at %s:%d\n", hipGetErrorString(_e), __FILE__, __LINE__); \
    return; } } while (0)

typedef unsigned u32;
typedef unsigned long long u64;
typedef u32 u32x4 __attribute__((ext_vector_type(4)));
typedef int i32x8 __attribute__((ext_vector_type(8)));
typedef float f32x4 __attribute__((ext_vector_type(4)));
typedef float f32x16 __attribute__((ext_vector_type(16)));
#define GAS __attribute__((address_space(1)))
#define LAS __attribute__((address_space(3)))

#define NPW_WG 64
#define NPW_WAVES 1
#define NPIECES 8
#define STAMPS 0

constexpr int NR = 8;
constexpr int GN = 16384;
constexpr int KS = 512;
constexpr int MS = 512;
constexpr int NTILE = 128;
constexpr int NWG = 256;
constexpr int NXWG = 32;
constexpr int CONS0 = 128;

constexpr u32 OFF_CNT = 0;
constexpr u32 OFF_TMO = 32u << 10;
constexpr u32 OFF_XF  = 64u << 10;
constexpr u32 OFF_WF  = 128u << 10;
constexpr u32 OFF_DBG = 256u << 10;
constexpr u32 OFF_X   = 1u << 20;
constexpr u32 X_PAR   = 2u << 20;
constexpr u32 OFF_W   = 8u << 20;
constexpr u32 W_PAR   = 64u << 20;
constexpr u32 WS_NEED = OFF_W + 2 * W_PAR;
constexpr u64 DEADLINE_TICKS = 50000000ull;

struct Args {
    const unsigned char* x;
    const unsigned char* w;
    const float* sx;
    const float* sw;
    float* out;
    unsigned char* ws[NR];
    int rank;
    int pad;
};

#define ST16(voff, data, sbase, imm) \
    asm volatile("global_store_dwordx4 %0, %1, %2 offset:" #imm " sc0 sc1\n\ts_nop 1" \
                 :: "v"(voff), "v"(data), "s"(sbase) : "memory")

__device__ __forceinline__ u64 rt() { return __builtin_amdgcn_s_memrealtime(); }

__device__ __forceinline__ void flag_store(unsigned char* base, u32 off, u32 v) {
    __hip_atomic_store((GAS u32*)(base + off), v, __ATOMIC_RELAXED, __HIP_MEMORY_SCOPE_SYSTEM);
}
__device__ __forceinline__ u32 flag_load(unsigned char* base, u32 off) {
    return __hip_atomic_load((GAS u32*)(base + off), __ATOMIC_RELAXED, __HIP_MEMORY_SCOPE_SYSTEM);
}

__global__ void __launch_bounds__(512, 2) fused_gemm_rs(Args a) {
    __shared__ __attribute__((aligned(16))) u32x4 lds[2 * 8 * 128 + 4];
    const int bid = blockIdx.x;
    const int lane = threadIdx.x & 63;
    const int wave = __builtin_amdgcn_readfirstlane(threadIdx.x >> 6);
    const int rank = a.rank;
    unsigned char* const myws = a.ws[rank];

    const u32 cnt_v = __hip_atomic_load((GAS u32*)(myws + OFF_CNT + bid * 128), __ATOMIC_RELAXED, __HIP_MEMORY_SCOPE_AGENT);
#define GET_CALL const u32 call = __builtin_amdgcn_readfirstlane(cnt_v) + 1, par = call & 1

    if (bid < CONS0) {
        if (bid < NXWG) {
            const int t = bid >> 3;
            const int q = (rank + 1 + (bid & 7)) & 7;
            const unsigned char* src = a.x + (size_t)(q * MS + wave * 64 + (lane & 31)) * KS + t * 128 + (lane >> 5) * 32;
            u32x4 v[2][2][2];
#pragma unroll
            for (int rb = 0; rb < 2; ++rb)
#pragma unroll
                for (int kb = 0; kb < 2; ++kb)
#pragma unroll
                    for (int p = 0; p < 2; ++p)
                        v[rb][kb][p] = *(const GAS u32x4*)(src + rb * 32 * KS + kb * 64 + p * 16);
            unsigned char* dst = a.ws[q];
            GET_CALL;
            asm volatile("" :: "v"(v[0][0][0]), "v"(v[0][0][1]), "v"(v[0][1][0]), "v"(v[0][1][1]),
                              "v"(v[1][0][0]), "v"(v[1][0][1]), "v"(v[1][1][0]), "v"(v[1][1][1]));
#pragma unroll
            for (int kb = 0; kb < 2; ++kb) {
                const u32 vo = OFF_X + par * X_PAR + ((rank * 8 + 2 * t + kb) * 16 + wave * 2) * 2048 + lane * 16;
                ST16(vo, v[0][kb][0], dst, 0);
                ST16(vo, v[0][kb][1], dst, 1024);
                ST16(vo, v[1][kb][0], dst, 2048);
                ST16(vo, v[1][kb][1], dst, 3072);
            }
            asm volatile("s_waitcnt vmcnt(0)" ::: "memory");
            __syncthreads();
            if (threadIdx.x == 0) flag_store(dst, OFF_XF + par * 128 + (t * 8 + rank) * 4, call);
        } else if (bid < NXWG + NPW_WG && wave < NPW_WAVES) {
            constexpr int NW = NPW_WG * NPW_WAVES;
            const int gw = (bid - NXWG) + wave * NPW_WG;
            const unsigned char* wl = a.w + (size_t)(lane >> 5) * 32 * GN + (lane & 31) * 4;
            u32 d[32];
            {
                const int tile = (((gw & 255) >> 1) + 16 * rank) & 127, kb = (gw >> 8) * 2 + (gw & 1);
                const unsigned char* s = wl + (size_t)kb * 64 * GN + tile * 128;
#pragma unroll
                for (int r = 0; r < 32; ++r) d[r] = *(const GAS u32*)(s + (size_t)r * GN);
            }
            unsigned char* const fdst = a.ws[lane & 7];
            GET_CALL;
            for (int i = gw; i < 1024; i += NW) {
                const int tile = (((i & 255) >> 1) + 16 * rank) & 127, kb = (i >> 8) * 2 + (i & 1);
                u32x4 f[4][2];
#pragma unroll
                for (int g = 0; g < 8; ++g) {
                    const u32 t0 = __builtin_amdgcn_perm(d[4 * g + 1], d[4 * g], 0x05010400u);
                    const u32 t1 = __builtin_amdgcn_perm(d[4 * g + 1], d[4 * g], 0x07030602u);
                    const u32 t2 = __builtin_amdgcn_perm(d[4 * g + 3], d[4 * g + 2], 0x05010400u);
                    const u32 t3 = __builtin_amdgcn_perm(d[4 * g + 3], d[4 * g + 2], 0x07030602u);
                    f[0][g >> 2][g & 3] = __builtin_amdgcn_perm(t2, t0, 0x05040100u);
                    f[1][g >> 2][g & 3] = __builtin_amdgcn_perm(t2, t0, 0x07060302u);
                    f[2][g >> 2][g & 3] = __builtin_amdgcn_perm(t3, t1, 0x05040100u);
                    f[3][g >> 2][g & 3] = __builtin_amdgcn_perm(t3, t1, 0x07060302u);
                }
                {
                    const int n = (i + NW < 1024) ? i + NW : i;
                    const int tile2 = (((n & 255) >> 1) + 16 * rank) & 127, kb2 = (n >> 8) * 2 + (n & 1);
                    const unsigned char* s = wl + (size_t)kb2 * 64 * GN + tile2 * 128;
#pragma unroll
                    for (int r = 0; r < 32; ++r) d[r] = *(const GAS u32*)(s + (size_t)r * GN);
                }
                const u32 vo = OFF_W + par * W_PAR + ((tile * 8 + rank) * 8 + kb) * 8192 + lane * 16 + 4096;
                int rot = gw % 7;
#pragma unroll
                for (int j = 0; j < 8; ++j) {
                    int q;
                    if (j < 7) { q = (rank + 1 + rot) & 7; rot = (rot == 6) ? 0 : rot + 1; } else q = rank;
                    unsigned char* dst = a.ws[q];
                    ST16(vo, f[0][0], dst, -4096);
                    ST16(vo, f[0][1], dst, -3072);
                    ST16(vo, f[1][0], dst, -2048);
                    ST16(vo, f[1][1], dst, -1024);
                    ST16(vo, f[2][0], dst, 0);
                    if (NPIECES > 5) ST16(vo, f[2][1], dst, 1024);
                    if (NPIECES > 6) ST16(vo, f[3][0], dst, 2048);
                    if (NPIECES > 7) ST16(vo, f[3][1], dst, 3072);
                }
                asm volatile("s_waitcnt vmcnt(0)" ::: "memory");
                if (lane < 8) {
                    const u32 fo = OFF_WF + ((par * NTILE + tile) * 64 + (kb & 1) * 32 + (kb >> 1) * 8 + rank) * 4;
                    flag_store(fdst, fo, call);
                }
            }
        }
    } else {
        const int tile = bid - CONS0;
        GET_CALL;
        LAS u32* smask = (LAS u32*)&lds[2 * 8 * 128];
        const __amdgpu_buffer_rsrc_t rs = __builtin_amdgcn_make_buffer_rsrc(myws, 0, WS_NEED, 0x00020000);
        const u32 wfo = OFF_WF + ((par * NTILE + tile) * 64 + lane) * 4;
        const u64 t0 = rt();
        bool dead = false;
        if (wave == 0) {
            const u32 xfo = OFF_XF + par * 128 + (lane & 31) * 4;
            while (!__all(flag_load(myws, xfo) >= call)) {
                if (__builtin_amdgcn_readfirstlane(__hip_atomic_load((GAS u32*)(myws + OFF_TMO), __ATOMIC_RELAXED, __HIP_MEMORY_SCOPE_AGENT)) == call
                    || rt() - t0 > DEADLINE_TICKS) { dead = true; break; }
                __builtin_amdgcn_s_sleep(2);
            }
            const u64 bal = __ballot(flag_load(myws, wfo) >= call);
            if (lane == 0) { smask[0] = (u32)bal & (u32)(bal >> 32); smask[2] = dead; }
        }
        __syncthreads();
        f32x16 acc[2][4];
#pragma unroll
        for (int i = 0; i < 2; ++i)
#pragma unroll
            for (int j = 0; j < 4; ++j)
#pragma unroll
                for (int r = 0; r < 16; ++r) acc[i][j][r] = 0.f;
        u32x4 wr[2];
        u32x4 xr[2][2][2];
        u32 taken = 0;
        int ca = -1, cb = -1, done = 0, it = 0;
        bool fail = false;
        for (;;) {
            const u32 ready = __builtin_amdgcn_readfirstlane(smask[it & 1]);
            fail = __builtin_amdgcn_readfirstlane(smask[2 + (it & 1)]) != 0;
            if (fail) break;
            const u32 avail = ready & ~taken;
            const int nxt = avail ? __builtin_ctz(avail) : -1;
            if (avail) taken |= 1u << nxt;
            u32 fl = 0;
            if (wave == 0) fl = flag_load(myws, wfo);
            const int pb = it & 1;
            if (ca >= 0) {
                lds[((pb ^ 1) * 8 + wave) * 128 + lane] = wr[0];
                lds[((pb ^ 1) * 8 + wave) * 128 + 64 + lane] = wr[1];
            }
            if (nxt >= 0) {
                const u32 so = OFF_W + par * W_PAR + ((tile * 8 + (nxt & 7)) * 8 + (nxt >> 3) * 2) * 8192 + wave * 2048;
                wr[0] = __builtin_amdgcn_raw_buffer_load_b128(rs, lane * 16, so, 17);
                wr[1] = __builtin_amdgcn_raw_buffer_load_b128(rs, lane * 16 + 1024, so, 17);
            }
            if (cb >= 0) {
#pragma unroll
                for (int kb = 0; kb < 2; ++kb) {
#pragma unroll
                    for (int c = 0; c < 4; ++c) {
                        const u32x4 b0 = lds[(pb * 8 + kb * 4 + c) * 128 + lane];
                        const u32x4 b1 = lds[(pb * 8 + kb * 4 + c) * 128 + 64 + lane];
                        const i32x8 bf = {(int)b0[0], (int)b0[1], (int)b0[2], (int)b0[3], (int)b1[0], (int)b1[1], (int)b1[2], (int)b1[3]};
#pragma unroll
                        for (int rb = 0; rb < 2; ++rb) {
                            const u32x4 a0 = xr[rb][kb][0], a1 = xr[rb][kb][1];
                            const i32x8 af = {(int)a0[0], (int)a0[1], (int)a0[2], (int)a0[3], (int)a1[0], (int)a1[1], (int)a1[2], (int)a1[3]};
                            acc[rb][c] = __builtin_amdgcn_mfma_scale_f32_32x32x64_f8f6f4(af, bf, acc[rb][c], 1, 1, 0, 0x7F7F7F7F, 0, 0x7F7F7F7F);
                        }
                    }
                }
                ++done;
            }
            if (done == 32) break;
            if (ca >= 0) {
                const u32 so = OFF_X + par * X_PAR + ((((ca & 7) * 8 + (ca >> 3) * 2) * 16) + wave * 2) * 2048;
#pragma unroll
                for (int kb = 0; kb < 2; ++kb)
#pragma unroll
                    for (int rb = 0; rb < 2; ++rb)
#pragma unroll
                        for (int p = 0; p < 2; ++p)
                            xr[rb][kb][p] = __builtin_amdgcn_raw_buffer_load_b128(rs, lane * 16 + rb * 2048 + p * 1024, so + kb * 32768, 17);
            }
            if (wave == 0) {
                u64 bal = __ballot(fl >= call);
                u32 m = (u32)bal & (u32)(bal >> 32);
                if (ca < 0 && nxt < 0) {
                    while ((m & ~taken) == 0) {
                        if (__builtin_amdgcn_readfirstlane(__hip_atomic_load((GAS u32*)(myws + OFF_TMO), __ATOMIC_RELAXED, __HIP_MEMORY_SCOPE_AGENT)) == call
                            || rt() - t0 > DEADLINE_TICKS) { dead = true; break; }
                        __builtin_amdgcn_s_sleep(1);
                        bal = __ballot(flag_load(myws, wfo) >= call);
                        m = (u32)bal & (u32)(bal >> 32);
                    }
                }
                if (lane == 0) { smask[(it + 1) & 1] = m; smask[2 + ((it + 1) & 1)] = dead; }
            }
            __syncthreads();
            cb = ca; ca = nxt; ++it;
        }
        if (fail && threadIdx.x == 0)
            __hip_atomic_store((GAS u32*)(myws + OFF_TMO), call, __ATOMIC_RELAXED, __HIP_MEMORY_SCOPE_AGENT);
        if (__builtin_amdgcn_readfirstlane(__hip_atomic_load((GAS u32*)(myws + OFF_TMO), __ATOMIC_RELAXED, __HIP_MEMORY_SCOPE_AGENT)) == call)
            fail = true;
        const float s = fail ? __builtin_nanf("") : a.sx[0] * a.sw[0];
        float* o = a.out + (size_t)(wave * 64 + (lane >> 5) * 4) * GN + tile * 128 + (lane & 31) * 4;
#pragma unroll
        for (int rb = 0; rb < 2; ++rb)
#pragma unroll
            for (int r = 0; r < 16; ++r) {
                f32x4 v;
#pragma unroll
                for (int c = 0; c < 4; ++c) {
                    const float y = acc[rb][c][r] * s;
                    const float e = __expf(-__builtin_amdgcn_fmed3f(y, -60.f, 60.f));
                    v[c] = y * __builtin_amdgcn_rcpf(1.f + e);
                }
                *(GAS f32x4*)(o + (size_t)(rb * 32 + (r & 3) + 8 * (r >> 2)) * GN) = v;
            }
    }
    __syncthreads();
    if (threadIdx.x == 0)
        __hip_atomic_store((GAS u32*)(myws + OFF_CNT + bid * 128), cnt_v + 1, __ATOMIC_RELAXED, __HIP_MEMORY_SCOPE_AGENT);
}

extern "C" void kernel_launch(int rank, int nranks, ncclComm_t comm,
                              hipStream_t stream,
                              void* const* d_in, const int* in_sizes, int n_in,
                              void* d_out, int out_size,
                              const int* in_dtypes, int out_dtype,
                              void* const* d_out_all,
                              void* const* d_ws_all, size_t ws_bytes) {
    if (nranks != NR || (unsigned)rank >= (unsigned)NR || ws_bytes < WS_NEED || n_in != 4
        || in_sizes[0] != 4096 * KS || in_sizes[1] != KS * GN || out_size != MS * GN) {
        fprintf(stderr, "unexpected problem shape\n");
        return;
    }
    Args a;
    a.x = (const unsigned char*)d_in[0];
    a.w = (const unsigned char*)d_in[1];
    a.sx = (const float*)d_in[2];
    a.sw = (const float*)d_in[3];
    a.out = (float*)d_out;
    for (int p = 0; p < NR; ++p) a.ws[p] = (unsigned char*)d_ws_all[p];
    a.rank = rank;
    a.pad = 0;
    fused_gemm_rs<<<NWG, 512, 0, stream>>>(a);
    HIP_CHECK(hipGetLastError());
}
```

```cpp
#include <hip/hip_runtime.h>
#include <rccl/rccl.h>
#include <cstdio>
#include <cstdint>
#include <type_traits>

#define HIP_CHECK(expr) do { hipError_t _e = (expr); if (_e != hipSuccess) { \
    fprintf(stderr, "HIP error %s at %s:%d\n", hipGetErrorString(_e), __FILE__, __LINE__); \
    return; } } while (0)

typedef unsigned u32;
typedef unsigned long long u64;
typedef unsigned char u8;
typedef u32 u32x2 __attribute__((ext_vector_type(2)));
typedef u32 u32x4 __attribute__((ext_vector_type(4)));
typedef int i32x8 __attribute__((ext_vector_type(8)));
typedef float f32x4 __attribute__((ext_vector_type(4)));
#define GAS __attribute__((address_space(1)))
#define LAS __attribute__((address_space(3)))

#define PIECE 640
#define WINDOW 3
#define PACE 1
#define PACEM 1
#define PACET 0x87655544u
#define XWIN 2
#define WAITALL 1
#define OUTPOL 2
#define XSPREAD 0
#define STAMPS 0

constexpr int NR = 8;
constexpr int GN = 16384;
constexpr int KS = 512;
constexpr int MS = 512;
constexpr int NWG = 256;
constexpr int NTHR = 640;

constexpr u32 OFF_CNT = 0;
constexpr u32 OFF_TMO = 32u << 10;
constexpr u32 OFF_PROG = 48u << 10;
constexpr u32 OFF_XF  = 64u << 10;
constexpr u32 OFF_WF  = 128u << 10;
constexpr u32 OFF_DBG = 256u << 10;
constexpr u32 OFF_X   = 1u << 20;
constexpr u32 X_PAR   = 2u << 20;
constexpr u32 OFF_W   = 8u << 20;
constexpr u32 VARCAP  = 5120;
constexpr u32 FIXB    = 5120;
constexpr u32 RUN     = VARCAP + 4 * FIXB;
constexpr u32 W_PAR   = 8u << 20;
constexpr u32 OFF_RAW = 24u << 20;
constexpr u32 RAW_PAR = 8u << 20;
constexpr u32 WS_NEED = OFF_RAW + 2 * RAW_PAR;
static_assert(NWG * RUN <= W_PAR, "runs fit");
constexpr u64 DEADLINE_TICKS = 50000000ull;

constexpr u32 CAP2 = 1536, CAP3 = 512, SECCAP = 624;
static_assert(32 + 8 * SECCAP <= VARCAP, "sections fit");

constexpr u32 L_LUTD = 0;
constexpr u32 L_LUTC = 1024;
constexpr u32 L_OWN  = 3072;
constexpr u32 L_DEC  = L_OWN + 32768;
constexpr u32 NSLOT  = 4;
constexpr u32 L_RING = L_DEC + 16384;
constexpr u32 L_VAR  = L_RING + NSLOT * FIXB;
constexpr u32 L_ZERO = L_VAR + 2 * VARCAP;
constexpr u32 V2S = CAP2 + 16, V3S = CAP3 + 16, EVS = 1040;
constexpr u32 L_V2   = L_ZERO;
constexpr u32 L_V3   = L_V2 + 8 * V2S;
constexpr u32 L_EV   = L_V3 + 8 * V3S;
constexpr u32 L_CTL  = L_EV + 8 * EVS;
constexpr u32 C_WB = L_CTL, C_LANDED = L_CTL + 4, C_DEAD = L_CTL + 8, C_PUB = L_CTL + 12;
constexpr u32 C_HDR = L_CTL + 16, C_T1 = L_CTL + 48;
constexpr u32 L_END  = L_CTL + 96;
static_assert(L_ZERO % 16 == 0 && L_END % 16 == 0 && L_END <= 160 * 1024, "LDS");
static_assert(CAP2 / 4 + CAP3 / 4 + CAP3 + 16 <= EVS, "section scratch");

struct Args {
    const u8* x;
    const u8* w;
    const float* sx;
    const float* sw;
    float* out;
    u8* ws[NR];
    int rank;
    int pad;
};

#define ST16(voff, data, sbase, imm) \
    asm volatile("global_store_dwordx4 %0, %1, %2 offset:" #imm " sc0 sc1\n\ts_nop 1" \
                 :: "v"(voff), "v"(data), "s"(sbase) : "memory")
#define ST4(voff, data, sbase) \
    asm volatile("global_store_dword %0, %1, %2 sc0 sc1" :: "v"(voff), "v"(data), "s"(sbase) : "memory")
#define RLX_WG __ATOMIC_RELAXED, __HIP_MEMORY_SCOPE_WORKGROUP
#define RLX_AG __ATOMIC_RELAXED, __HIP_MEMORY_SCOPE_AGENT
#define RLX_SYS __ATOMIC_RELAXED, __HIP_MEMORY_SCOPE_SYSTEM
#define CFENCE() asm volatile("" ::: "memory")

__device__ __forceinline__ u64 rt() { return __builtin_amdgcn_s_memrealtime(); }
__device__ __forceinline__ u32 uni(u32 v) { return __builtin_amdgcn_readfirstlane(v); }
__device__ __forceinline__ u32 perm(u32 hi, u32 lo, u32 sel) { return __builtin_amdgcn_perm(hi, lo, sel); }
__device__ __forceinline__ u32 scan64(u32 v) {
    v += __builtin_amdgcn_update_dpp(0u, v, 0x111, 0xf, 0xf, true);
    v += __builtin_amdgcn_update_dpp(0u, v, 0x112, 0xf, 0xf, true);
    v += __builtin_amdgcn_update_dpp(0u, v, 0x114, 0xf, 0xf, true);
    v += __builtin_amdgcn_update_dpp(0u, v, 0x118, 0xf, 0xf, true);
    v += __builtin_amdgcn_update_dpp(0u, v, 0x142, 0xa, 0xf, false);
    v += __builtin_amdgcn_update_dpp(0u, v, 0x143, 0xc, 0xf, false);
    return v;
}
__device__ __forceinline__ u32 last(u32 v) { return __builtin_amdgcn_readlane(v, 63); }
__device__ __forceinline__ u32 escbits(u32 q) { return ~(q | (q >> 1)) & 0x55555555u; }
__device__ __forceinline__ u32 classify8(u32 e4, u32 kadd, u32 thi, u32 tlo) {
    const u32 r = perm(thi, tlo, ((e4 >> 2) + kadd) & 0x1F1F1F1Fu);
    return __builtin_amdgcn_udot4((r ^ (r >> 2)) & 0x18181818u, 0x40100401u, 0u, false);
}
constexpr u32 T13 = 0x00181008u, T2L = 0x00000818u, T2H = 0x00001000u;
__device__ __forceinline__ u32 rep(u32 b) { return b * 0x01010101u; }

__global__ void __launch_bounds__(NTHR) fused_gemm_rs(Args a) {
    __shared__ __attribute__((aligned(16))) u8 lds_raw[L_END];
    LAS u8* const L = (LAS u8*)lds_raw;
#define L32(off) (*(LAS u32*)(L + (off)))
#define L16(off) (*(LAS unsigned short*)(L + (off)))
#define L64(off) (*(LAS u32x2*)(L + (off)))
#define L128(off) (*(LAS u32x4*)(L + (off)))
    const int bid = blockIdx.x;
    const int tile = (bid & ~15) | ((bid & 7) << 1) | ((bid >> 3) & 1);
    const int lane = threadIdx.x & 63;
    const int wave = uni(threadIdx.x >> 6);
    const int rank = a.rank;
    u8* const myws = a.ws[rank];
    const u64 t_begin = rt();
#define STAMP(i)

    const u32 cnt_v = __hip_atomic_load((GAS u32*)(myws + OFF_CNT + bid * 128), RLX_AG);
#define GET_CALL const u32 call = uni(cnt_v) + 1, par = call & 1
    u32 spins = 0;
    auto expired = [&](u32 call) -> bool {
        if (++spins & 15) return false;
        return uni(__hip_atomic_load((GAS u32*)(myws + OFF_TMO), RLX_AG)) == call || rt() - t_begin > DEADLINE_TICKS;
    };
    auto give_up = [&](u32 call) {
        if (lane == 0) {
            __hip_atomic_store((GAS u32*)(myws + OFF_TMO), call, RLX_AG);
            __hip_atomic_store((LAS u32*)(L + C_DEAD), 1u, RLX_WG);
        }
    };
    auto lds_init = [&]() {
        for (u32 o = L_ZERO + threadIdx.x * 16; o < L_END; o += 576 * 16) L128(o) = u32x4{0, 0, 0, 0};
        if (threadIdx.x < 256) {
            const u32 x = threadIdx.x;
            u32 d = 0, c = 0x0C0C0C0Cu, n = 0;
#pragma unroll
            for (int b = 0; b < 4; ++b) {
                const u32 code = (x >> (2 * b)) & 3;
                d |= (code ? 4 + code : n) << (8 * b);
                if (!code) c = (c & ~(0xFFu << (8 * n))) | (b << (8 * n));
                n += !code;
            }
            L32(L_LUTD + x * 4) = d;
            L64(L_LUTC + x * 8) = u32x2{c, n};
        }
    };

    if (wave == 9) {
        const int q = (rank + 1 + ((bid + XSPREAD * (bid >> 3)) & 7)) & 7, t = (bid >> 3) & 3, rg = bid >> 5;
        const u8* src = a.x + (size_t)(q * MS + rg * 64 + (lane & 15)) * KS + t * 128 + (lane >> 4) * 32;
        u32x4 v[4][2];
#pragma unroll
        for (int rb = 0; rb < 4; ++rb)
#pragma unroll
            for (int p = 0; p < 2; ++p) v[rb][p] = *(const GAS u32x4*)(src + rb * 16 * KS + p * 16);
        asm volatile("s_barrier" ::: "memory");
        GET_CALL;
        asm volatile("" :: "v"(v[0][0]), "v"(v[0][1]), "v"(v[1][0]), "v"(v[1][1]),
                          "v"(v[2][0]), "v"(v[2][1]), "v"(v[3][0]), "v"(v[3][1]));
        u8* dst = a.ws[q];
        const u32 vo = OFF_X + par * X_PAR + ((rank * 4 + t) * 8 + rg) * 8192 + lane * 16 + 4096;
#define XW() asm volatile("s_waitcnt vmcnt(%0)" :: "n"(XWIN - 1) : "memory")
        ST16(vo, v[0][0], dst, -4096); XW(); ST16(vo, v[0][1], dst, -3072); XW();
        ST16(vo, v[1][0], dst, -2048); XW(); ST16(vo, v[1][1], dst, -1024); XW();
        ST16(vo, v[2][0], dst, 0);     XW(); ST16(vo, v[2][1], dst, 1024);  XW();
        ST16(vo, v[3][0], dst, 2048);  XW(); ST16(vo, v[3][1], dst, 3072);
        asm volatile("s_waitcnt vmcnt(0)" ::: "memory");
        if (lane == 0) __hip_atomic_store((GAS u32*)(dst + OFF_XF + par * 1024 + rg * 128 + (rank * 4 + t) * 4), call, RLX_SYS);
        STAMP(1);
        return;
    }
    if (wave == 8) {
        lds_init();
        asm volatile("s_waitcnt lgkmcnt(0)\n\ts_barrier" ::: "memory");
        GET_CALL;
        {
            u32 lines_v = 0, have = 0;
            bool dead = false;
            const u32 lbase = (u32)(size_t)L;
            const u32 run = OFF_W + par * W_PAR + tile * RUN + lane * 16;
            u32 landed = 0, endbits = 0;
            auto publish = [&]() { if (lane == 0) __hip_atomic_store((LAS u32*)(L + C_LANDED), landed, RLX_WG); };
            auto drain = [&]() {
                asm volatile("s_waitcnt vmcnt(0)" ::: "memory");
                landed += __builtin_popcount(endbits);
                endbits = 0;
                publish();
            };
            auto wait_free = [&](u32 need) {
                if (uni(__hip_atomic_load((LAS u32*)(L + C_WB), RLX_WG)) < need) {
                    drain();
                    while (uni(__hip_atomic_load((LAS u32*)(L + C_WB), RLX_WG)) < need) __builtin_amdgcn_s_sleep(1);
                }
                CFENCE();
            };
            auto pull = [&](const u8* base, u32 goff, u32 loff, u32 bytes, u32 piece) {
                for (u32 o = 0; o < bytes; o += piece) {
                    if ((u32)lane * 16 < min(piece, bytes - o)) {
                        u32 keep;
                        asm volatile("s_mov_b32 %0, m0\n\ts_mov_b32 m0, %2\n\ts_nop 0\n\t"
                                     "global_load_lds_dwordx4 %1, %3 sc0 sc1\n\ts_mov_b32 m0, %0"
                                     : "=&s"(keep) : "v"(run + goff + o), "s"(lbase + loff + o), "s"(base) : "memory");
                    }
                    endbits = (endbits << 1) | (o + piece >= bytes);
                    asm volatile("s_waitcnt vmcnt(%0)" :: "n"(WINDOW - 1) : "memory");
                    if (endbits >> (WINDOW - 1)) { ++landed; publish(); endbits &= (1u << (WINDOW - 1)) - 1; }
                }
            };
            u32 piece = PIECE, order_v = 0;
            int pj = bid % 7;
            for (int j = 0; j < 7; ++j) {
                const int s = (rank + 1 + pj) & 7;
                pj = pj == 6 ? 0 : pj + 1;
                const u8* base = a.ws[s];
                if (!((have >> s) & 1)) {
                    drain();
                    for (;;) {
                        const u64 f = __hip_atomic_load((GAS u64*)(myws + OFF_WF + (par * NWG + tile) * 128 + (lane & 7) * 8), RLX_SYS);
                        lines_v = (u32)(f >> 32);
                        have = (u32)__ballot((u32)f >= call);
                        if (WAITALL ? ((have | (1u << rank)) & 0xFF) == 0xFF : (have >> s) & 1) break;
                        if (expired(call)) { dead = true; break; }
                        __builtin_amdgcn_s_sleep(2);
                    }
                    CFENCE();
                    if (dead) { give_up(call); break; }
                    if (j == 0) STAMP(2);
                }
                u32 vb = __builtin_amdgcn_readlane(lines_v, s) * 128;
                vb = vb < 128 ? 128 : (vb > VARCAP ? VARCAP : vb);
                if (j >= 2) wait_free(8 * (1 + 5 * (j - 2) + 1));
                pull(base, 0, L_VAR + (j & 1) * VARCAP, vb, piece);
                for (int t = 0; t < 4; ++t) {
                    if (j >= 1) wait_free(8 * (1 + 5 * (j - 1) + 1 + t + 1));
                    if ((t & PACEM) == PACEM && lane == 0)
                        asm volatile("global_atomic_add %0, %1, %2, %3 sc0 sc1" : "=v"(order_v) : "v"(OFF_PROG + (j * 4 + t) * 128), "v"(1u), "s"(myws) : "memory");
                    pull(base, VARCAP + t * FIXB, L_RING + t * FIXB, FIXB, piece);
                    if ((t & PACEM) == PACEM) {
                        asm volatile("" : "+v"(order_v));
                        const u32 k = (uni(order_v) & 255) >> 5;
                        piece = 128 * ((PACET >> (4 * k)) & 15);
                    }
                }
                STAMP(3 + j);
            }
            asm volatile("s_waitcnt vmcnt(0)" ::: "memory");
        }
        CFENCE();
        if (lane == 0) __hip_atomic_store((LAS u32*)(L + C_LANDED), 1000u, RLX_WG);
        STAMP(10);
        return;
    }

    u32 phase = 0;
    auto wbar = [&]() {
        phase += 8;
        asm volatile("s_waitcnt lgkmcnt(0)" ::: "memory");
        if (lane == 0) __hip_atomic_fetch_add((LAS u32*)(L + C_WB), 1u, RLX_WG);
        while (uni(__hip_atomic_load((LAS u32*)(L + C_WB), RLX_WG)) < phase) __builtin_amdgcn_s_sleep(1);
        CFENCE();
    };
    auto append = [&](auto G, u32 pos, const u32* e4, const u32x2* lut) {
        u32 carry = 0;
#pragma unroll
        for (int i = 0; i < G.value; ++i) {
            const u32 comp = perm(0u, e4[i], lut[i][0]), sh = pos << 3;
            __hip_atomic_fetch_or((LAS u32*)(L + (pos & ~3u)), (comp << (sh & 31)) | carry, RLX_WG);
            carry = __builtin_amdgcn_ubfe(comp, 0u - sh, sh);
            pos += lut[i][1];
        }
        __hip_atomic_fetch_or((LAS u32*)(L + (pos & ~3u)), carry, RLX_WG);
    };
    auto expand = [&](u32 tab, u32 pos, u32 x) -> u32 {
        const u32 sel = L32(L_LUTD + x * 4);
        const LAS u32* p = (const LAS u32*)(L + (pos & ~3u));
        return perm(tab, __builtin_amdgcn_alignbyte(p[1], p[0], pos & 3), sel);
    };

    {
        const int t = wave >> 1, h = wave & 1;
        const u8* src = a.w + (size_t)(t * 128 + (lane >> 4) * 32 + h * 16) * GN + tile * 64 + (lane & 15) * 4;
        u32 d[16];
#pragma unroll
        for (int r = 0; r < 16; ++r) d[r] = *(const GAS u32*)(src + (size_t)r * GN);
        u8* peer = a.ws[lane & 7];
        lds_init();
        asm volatile("s_waitcnt lgkmcnt(0)\n\ts_barrier" ::: "memory");
        __builtin_amdgcn_sched_barrier(0);
        u32 f[4][4];
#pragma unroll
        for (int i = 0; i < 4; ++i) {
            const u32 t0 = perm(d[4 * i + 1], d[4 * i], 0x05010400u), t1 = perm(d[4 * i + 1], d[4 * i], 0x07030602u);
            const u32 t2 = perm(d[4 * i + 3], d[4 * i + 2], 0x05010400u), t3 = perm(d[4 * i + 3], d[4 * i + 2], 0x07030602u);
            f[0][i] = perm(t2, t0, 0x05040100u); f[1][i] = perm(t2, t0, 0x07060302u);
            f[2][i] = perm(t3, t1, 0x05040100u); f[3][i] = perm(t3, t1, 0x07060302u);
        }
#define TS(i)
        TS(0);
        GET_CALL;
        asm volatile("" : "+v"(peer));
        u32 esum = 0;
#pragma unroll
        for (int c = 0; c < 4; ++c) {
            L128(L_OWN + ((t * 4 + c) * 2 + h) * 1024 + lane * 16) = u32x4{f[c][0], f[c][1], f[c][2], f[c][3]};
#pragma unroll
            for (int i = 0; i < 4; ++i) esum = __builtin_amdgcn_udot4(f[c][i] & 0x7C7C7C7Cu, 0x01010101u, esum, false);
        }
        const u32 e0 = ((last(scan64(esum)) + 14746u) >> 14) & 31;
        const u32 runo = OFF_W + par * W_PAR + tile * RUN;
        u32x4 Q, Mq;
        u32 S[2] = {0, 0};
        u32 e4[16];
        u32x2 lut[16];
        const u32 k1 = rep((33 - e0) & 31);
#pragma unroll
        for (int c = 0; c < 4; ++c) {
            u32 q = 0, m = 0;
#pragma unroll
            for (int i = 0; i < 4; ++i) {
                e4[4 * c + i] = f[c][i] & 0x7C7C7C7Cu;
                const u32 x8 = classify8(e4[4 * c + i], k1, 0u, T13);
                lut[4 * c + i] = L64(L_LUTC + x8);
                q |= i ? x8 << (8 * i - 3) : x8 >> 3;
                m |= (f[c][i] & 0x03030303u) << (2 * i);
                S[c >> 1] |= (f[c][i] & 0x80808080u) >> (4 * (c & 1) + i);
            }
            Q[c] = q; Mq[c] = m;
        }
        {
            const u32 vo = runo + VARCAP + t * FIXB + h * (FIXB / 2) + lane * 16;
            ST16(vo, Q, myws, 0);
            ST16(vo, Mq, myws, 1024);
            const u32x4 s4 = {S[0], S[1], (u32)__shfl_xor((int)S[0], 32), (u32)__shfl_xor((int)S[1], 32)};
            if (lane < 32) ST16(vo, s4, myws, 2048);
        }
        TS(1);
        u32 n = 0;
#pragma unroll
        for (int c = 0; c < 4; ++c) n += __builtin_popcount(escbits(Q[c]));
        const u32 inc = scan64(n);
        const u32 N2 = last(inc);
        const u32 u2 = L_V2 + wave * V2S, u3 = L_V3 + wave * V3S, ev = L_EV + wave * EVS;
        u32 len = 0;
        bool raw = N2 > CAP2;
        if (!raw) {
            append(std::integral_constant<int, 16>{}, u2 + inc - n, e4, lut);
            TS(2);
            const u32 k2 = rep((35 - e0) & 31);
            u32 u[6], m = 0;
            u32x2 l2[6];
#pragma unroll
            for (int i = 0; i < 3; ++i) {
                const u32x2 v = L64(u2 + lane * 24 + i * 8);
                u[2 * i] = v[0]; u[2 * i + 1] = v[1];
            }
#pragma unroll
            for (int i = 0; i < 3; ++i) {
                u32 q8 = classify8(u[2 * i], k2, T2H, T2L) | (classify8(u[2 * i + 1], k2, T2H, T2L) << 8);
                const int valid = (int)N2 - lane * 24 - i * 8;
                q8 |= (0x5555u * 8) << (2 * min(max(valid, 0), 8));
                if (valid > 0) L16(ev + lane * 6 + i * 2) = (unsigned short)(q8 >> 3);
                l2[2 * i] = L64(L_LUTC + (q8 & 0x7F8));
                l2[2 * i + 1] = L64(L_LUTC + ((q8 >> 8) & 0x7F8));
            }
#pragma unroll
            for (int i = 0; i < 6; ++i) m += l2[i][1];
            const u32 in2 = scan64(m);
            const u32 N3 = last(in2);
            raw = N3 > CAP3;
            if (!raw) {
                if (m) append(std::integral_constant<int, 6>{}, u3 + in2 - m, u, l2);
                TS(3);
                const u32 k3 = rep((38 - e0) & 31);
                const u32 off3 = ((N2 + 7) >> 3) * 2, off4 = off3 + ((N3 + 7) >> 3) * 2;
                const u32x2 v = L64(u3 + lane * 8);
                const u32 w[2] = {v[0], v[1]};
                u32 q8 = classify8(w[0], k3, 0u, T13) | (classify8(w[1], k3, 0u, T13) << 8);
                const int valid = (int)N3 - lane * 8;
                q8 |= (0x5555u * 8) << (2 * min(max(valid, 0), 8));
                if (valid > 0) L16(ev + off3 + lane * 2) = (unsigned short)(q8 >> 3);
                const u32x2 l3[2] = {L64(L_LUTC + (q8 & 0x7F8)), L64(L_LUTC + ((q8 >> 8) & 0x7F8))};
                const u32 m3 = l3[0][1] + l3[1][1];
                const u32 in3 = scan64(m3);
                if (m3) append(std::integral_constant<int, 2>{}, ev + off4 + in3 - m3, w, l3);
                len = off4 + last(in3);
                raw = len > SECCAP;
            }
        }
        if (raw) {
            len = 0;
            const u32 vo = OFF_RAW + par * RAW_PAR + ((tile * 4 + t) * 8 + h) * 1024 + lane * 16 + 4096;
            const u32x4 f0 = {f[0][0], f[0][1], f[0][2], f[0][3]}, f1 = {f[1][0], f[1][1], f[1][2], f[1][3]};
            const u32x4 f2 = {f[2][0], f[2][1], f[2][2], f[2][3]}, f3 = {f[3][0], f[3][1], f[3][2], f[3][3]};
            ST16(vo, f0, myws, -4096); ST16(vo, f1, myws, -2048); ST16(vo, f2, myws, 0); ST16(vo, f3, myws, 2048);
        }
        TS(4);
        const u32 len16 = (len + 15) >> 4;
        if (lane == 0) L32(C_HDR + wave * 4) = len16 | (raw ? 0x80000000u : (N2 << 9) | (e0 << 20));
        wbar();
        TS(5);
        const u32 hv = lane < 8 ? L32(C_HDR + lane * 4) : 0u;
        const u32 hin = scan64(hv & 0x1FF);
        const u32 hoff = 2 + hin - (hv & 0x1FF);
        if ((u32)lane < len16) {
            const u32x4 sec = L128(ev + lane * 16);
            ST16(runo + __builtin_amdgcn_readlane(hoff, wave) * 16 + lane * 16, sec, myws, 0);
        }
        if (wave == 0 && lane < 8) ST4(runo + lane * 4, (hv & ~0x1FFu) | hoff, myws);
        TS(6);
        asm volatile("s_waitcnt vmcnt(0)" ::: "memory");
        TS(7);
        u32 arrived = 0;
        if (lane == 0) arrived = __hip_atomic_fetch_add((LAS u32*)(L + C_PUB), 1u, RLX_WG);
        if (uni(arrived) == 7) {
            const u64 fl = ((u64)((2 + last(hin) + 7) >> 3) << 32) | call;
            if (lane < 8 && lane != rank)
                __hip_atomic_store((GAS u64*)(peer + OFF_WF + (par * NWG + tile) * 128 + rank * 8), fl, RLX_SYS);
            STAMP(11);
        }
    }

    GET_CALL;
    const __amdgpu_buffer_rsrc_t rs = __builtin_amdgcn_make_buffer_rsrc(myws, 0, WS_NEED, 0x00020000);
    f32x4 acc[4][4];
#pragma unroll
    for (int i = 0; i < 4; ++i)
#pragma unroll
        for (int j = 0; j < 4; ++j) acc[i][j] = f32x4{0.f, 0.f, 0.f, 0.f};
    u32 xready = 0;
    int s = rank, pj = bid % 7;
    u32 item = 0;
    const int c = wave >> 1, h = wave & 1;
    for (int n = 0; n < 32; ++n) {
        const int t = n & 3;
        if (n >= 4 && t == 0) { s = (rank + 1 + pj) & 7; pj = pj == 6 ? 0 : pj + 1; }
        while (!((xready >> (s * 4 + t)) & 1)) {
            xready = (u32)__ballot(__hip_atomic_load((GAS u32*)(myws + OFF_XF + par * 1024 + wave * 128 + (lane & 31) * 4), RLX_SYS) >= call);
            if ((xready >> (s * 4 + t)) & 1) break;
            if (expired(call)) { give_up(call); xready = ~0u; break; }
            __builtin_amdgcn_s_sleep(4);
        }
        CFENCE();
        u32x4 xr[4][2];
        {
            const u32 so = OFF_X + par * X_PAR + ((s * 4 + t) * 8 + wave) * 8192;
#pragma unroll
            for (int rb = 0; rb < 4; ++rb)
#pragma unroll
                for (int p = 0; p < 2; ++p)
                    xr[rb][p] = __builtin_amdgcn_raw_buffer_load_b128(rs, lane * 16 + (rb & 1) * 2048 + p * 1024, so + (rb >> 1) * 4096, 17);
        }
        u32 bb = L_OWN + t * 8192;
        if (n >= 4) {
            if (t == 0) {
                ++item;
                while (uni(__hip_atomic_load((LAS u32*)(L + C_LANDED), RLX_WG)) < item) __builtin_amdgcn_s_sleep(1);
                CFENCE();
                const u32 vb = L_VAR + (((n >> 2) - 1) & 1) * VARCAP;
                const u32 hdr = uni(L32(vb + wave * 4));
                const u32 e0 = (hdr >> 20) & 31;
                u32 t1 = (((e0 - 1) & 31) << 10) | (e0 << 18) | (((e0 + 1) & 31) << 26);
                if (hdr >> 31) {
                    t1 = 1;
                } else {
                    const u32 N2 = (hdr >> 9) & 0x7FF, sec = vb + (hdr & 0x1FF) * 16;
                    const u32 t2 = (((e0 - 2) & 31) << 10) | (((e0 + 2) & 31) << 18) | (((e0 - 3) & 31) << 26);
                    const u32 t3 = (((e0 - 6) & 31) << 10) | (((e0 - 5) & 31) << 18) | (((e0 - 4) & 31) << 26);
                    const u32 v2 = L_V2 + wave * V2S, v3 = L_V3 + wave * V3S;
                    u32 q[3], e[3], m = 0;
#pragma unroll
                    for (int i = 0; i < 3; ++i) {
                        q[i] = L16(sec + lane * 6 + i * 2) | 0x55550000u;
                        if ((u32)lane * 24 + i * 8 >= N2) q[i] = 0x55555555u;
                        e[i] = escbits(q[i]);
                        m += __builtin_popcount(e[i]);
                    }
                    const u32 in2 = scan64(m), N3 = last(in2);
                    const u32 off3 = sec + ((N2 + 7) >> 3) * 2, off4 = off3 + ((N3 + 7) >> 3) * 2;
                    {
                        u32 q3 = L16(off3 + lane * 2) | 0x55550000u;
                        if ((u32)lane * 8 >= N3) q3 = 0x55555555u;
                        const u32 e3 = escbits(q3), m3 = __builtin_popcount(e3);
                        const u32 p3 = off4 + scan64(m3) - m3;
                        L64(v3 + lane * 8) = u32x2{expand(t3, p3, q3 & 0xFF), expand(t3, p3 + __builtin_popcount(e3 & 0xFF), (q3 >> 8) & 0xFF)};
                    }
                    u32 pp = v3 + in2 - m;
                    u32x2 r[3];
#pragma unroll
                    for (int i = 0; i < 3; ++i) {
                        r[i] = u32x2{expand(t2, pp, q[i] & 0xFF), expand(t2, pp + __builtin_popcount(e[i] & 0xFF), (q[i] >> 8) & 0xFF)};
                        pp += __builtin_popcount(e[i]);
                    }
#pragma unroll
                    for (int i = 0; i < 3; ++i) L64(v2 + lane * 24 + i * 8) = r[i];
                }
                if (lane == 0) L32(C_T1 + wave * 4) = t1;
                wbar();
            }
            ++item;
            const u32 t1 = uni(L32(C_T1 + (t * 2 + h) * 4));
            while (uni(__hip_atomic_load((LAS u32*)(L + C_LANDED), RLX_WG)) < item) __builtin_amdgcn_s_sleep(1);
            CFENCE();
            bb = L_DEC + (n & 1) * 8192;
            u32x4 r;
            if (t1 & 1) {
                const u32 vo = OFF_RAW + par * RAW_PAR + ((tile * 4 + t) * 8 + c * 2 + h) * 1024 + lane * 16;
                asm volatile("global_load_dwordx4 %0, %1, %2 sc0 sc1\n\ts_waitcnt vmcnt(0)" : "=&v"(r) : "v"(vo), "s"(a.ws[s]) : "memory");
            } else {
                const u32 fx = L_RING + t * FIXB + h * (FIXB / 2);
                const u32x4 Q = L128(fx + lane * 16);
                const u32 m = L32(fx + 1024 + lane * 16 + c * 4);
                const u32 sg = L32(fx + 2048 + (lane & 31) * 16 + (lane >> 5) * 8 + (c >> 1) * 4) << (4 * (c & 1));
                const u32 n0 = __builtin_popcount(escbits(Q[0])), n1 = n0 + __builtin_popcount(escbits(Q[1]));
                const u32 n2 = n1 + __builtin_popcount(escbits(Q[2])), n3 = n2 + __builtin_popcount(escbits(Q[3]));
                const u32 q = c == 0 ? Q[0] : c == 1 ? Q[1] : c == 2 ? Q[2] : Q[3];
                const u32 before = c == 0 ? 0 : c == 1 ? n0 : c == 2 ? n1 : n2;
                const u32 e = escbits(q);
                u32 pp = L_V2 + (t * 2 + h) * V2S + scan64(n3) - n3 + before;
#pragma unroll
                for (int i = 0; i < 4; ++i) {
                    r[i] = expand(t1, pp, (q >> (8 * i)) & 0xFF) | ((m >> (2 * i)) & 0x03030303u) | ((sg << i) & 0x80808080u);
                    pp += __builtin_popcount(e & (0xFFu << (8 * i)));
                }
            }
            L128(bb + wave * 1024 + lane * 16) = r;
            wbar();
        }
#pragma unroll
        for (int cc = 0; cc < 4; ++cc) {
            const u32x4 b0 = L128(bb + cc * 2048 + lane * 16), b1 = L128(bb + cc * 2048 + 1024 + lane * 16);
            const i32x8 bf = {(int)b0[0], (int)b0[1], (int)b0[2], (int)b0[3], (int)b1[0], (int)b1[1], (int)b1[2], (int)b1[3]};
#pragma unroll
            for (int rb = 0; rb < 4; ++rb) {
                const u32x4 a0 = xr[rb][0], a1 = xr[rb][1];
                const i32x8 af = {(int)a0[0], (int)a0[1], (int)a0[2], (int)a0[3], (int)a1[0], (int)a1[1], (int)a1[2], (int)a1[3]};
                acc[rb][cc] = __builtin_amdgcn_mfma_scale_f32_16x16x128_f8f6f4(af, bf, acc[rb][cc], 1, 1, 0, 0x7F7F7F7F, 0, 0x7F7F7F7F);
            }
        }
    }
    const bool fail = uni(__hip_atomic_load((LAS u32*)(L + C_DEAD), RLX_WG)) != 0
                      || uni(__hip_atomic_load((GAS u32*)(myws + OFF_TMO), RLX_AG)) == call;
    const float sc = fail ? __builtin_nanf("") : a.sx[0] * a.sw[0];
    float* o = a.out + (size_t)(wave * 64 + (lane >> 4) * 4) * GN + tile * 64 + (lane & 15) * 4;
#pragma unroll
    for (int rb = 0; rb < 4; ++rb)
#pragma unroll
        for (int r = 0; r < 4; ++r) {
            f32x4 v;
#pragma unroll
            for (int cc = 0; cc < 4; ++cc) {
                const float y = acc[rb][cc][r] * sc;
                const float e = __expf(-__builtin_amdgcn_fmed3f(y, -60.f, 60.f));
                v[cc] = y * __builtin_amdgcn_rcpf(1.f + e);
            }
            asm volatile("global_store_dwordx4 %0, %1, off sc1\n\ts_nop 1" :: "v"(o + (size_t)(rb * 16 + r) * GN), "v"(v) : "memory");
        }
    if (threadIdx.x == 0) __hip_atomic_store((GAS u32*)(myws + OFF_CNT + bid * 128), call, RLX_AG);
}

extern "C" void kernel_launch(int rank, int nranks, ncclComm_t comm,
                              hipStream_t stream,
                              void* const* d_in, const int* in_sizes, int n_in,
                              void* d_out, int out_size,
                              const int* in_dtypes, int out_dtype,
                              void* const* d_out_all,
                              void* const* d_ws_all, size_t ws_bytes) {
    if (nranks != NR || (unsigned)rank >= (unsigned)NR || ws_bytes < WS_NEED || n_in != 4
        || in_sizes[0] != 4096 * KS || in_sizes[1] != KS * GN || out_size != MS * GN) {
        fprintf(stderr, "unexpected problem shape\n");
        return;
    }
    Args a;
    a.x = (const u8*)d_in[0];
    a.w = (const u8*)d_in[1];
    a.sx = (const float*)d_in[2];
    a.sw = (const float*)d_in[3];
    a.out = (float*)d_out;
    for (int p = 0; p < NR; ++p) a.ws[p] = (u8*)d_ws_all[p];
    a.rank = rank;
    a.pad = 0;
    fused_gemm_rs<<<NWG, NTHR, 0, stream>>>(a);
    HIP_CHECK(hipGetLastError());
}
```

```cpp
#include <hip/hip_runtime.h>
#include <rccl/rccl.h>
#include <cstdio>
#include <cstdint>
#include <type_traits>

#define HIP_CHECK(expr) do { hipError_t _e = (expr); if (_e != hipSuccess) { \
    fprintf(stderr, "HIP error %s at %s:%d\n", hipGetErrorString(_e), __FILE__, __LINE__); \
    return; } } while (0)

typedef unsigned u32;
typedef unsigned long long u64;
typedef unsigned char u8;
typedef u32 u32x2 __attribute__((ext_vector_type(2)));
typedef u32 u32x4 __attribute__((ext_vector_type(4)));
typedef int i32x8 __attribute__((ext_vector_type(8)));
typedef float f32x4 __attribute__((ext_vector_type(4)));
#define GAS __attribute__((address_space(1)))
#define LAS __attribute__((address_space(3)))

#define WINDOW 7
#define GSLEEP 16
#define XWIN 3
#define STAMPS 0

constexpr int NR = 8;
constexpr int GN = 16384;
constexpr int KS = 512;
constexpr int MS = 512;
constexpr int NWG = 256;
constexpr int NTHR = 704;
constexpr u32 NITEM = 216;

constexpr u32 VARCAP = 5120;
constexpr u32 HALFB  = 2560;
constexpr u32 RUN    = VARCAP + 8 * HALFB;
constexpr u32 CAP2 = 1536, CAP3 = 512, SECCAP = 624;
static_assert(32 + 8 * SECCAP <= VARCAP, "sections fit");
constexpr u32 EXCCAP = 1024;
constexpr u32 UNITB  = 6144;
constexpr u32 OHALF  = EXCCAP + 4 * UNITB;
constexpr u32 OSECCAP = 240;
static_assert(16 + 4 * OSECCAP <= EXCCAP && OHALF == RUN, "sections fit");

constexpr u32 OFF_CNT = 0;
constexpr u32 OFF_TMO = 32u << 10;
constexpr u32 OFF_CT  = 40u << 10;
constexpr u32 OFF_PF  = 48u << 10;
constexpr u32 OFF_Q   = 56u << 10;
constexpr u32 OFF_XF  = 64u << 10;
constexpr u32 OFF_F   = 128u << 10;
constexpr u32 OFF_DBG = 256u << 10;
constexpr u32 PSTR    = 32 * 16384 + 4352;
constexpr u32 OFF_XO  = 1u << 20;
constexpr u32 OFF_SP  = 6u << 20;
constexpr u32 OFF_P   = 32u << 20;
constexpr u32 P_PAR   = 8u << 20;
constexpr u32 OFF_RAW = 48u << 20;
constexpr u32 RAW_PAR = 10u << 20;
constexpr u32 OFF_OP  = 68u << 20;
constexpr u32 OP_PAR  = 25u << 20;
constexpr u32 OFF_OR  = 118u << 20;
constexpr u32 OR_PAR  = 32u << 20;
constexpr u32 WS_NEED = OFF_OR + 2 * OR_PAR;
static_assert(OFF_XO + 8 * PSTR <= OFF_SP && OFF_SP + 48 * PSTR <= OFF_P && 320 * RUN <= P_PAR
              && 320 * 32768 <= RAW_PAR && 1024 * OHALF <= OP_PAR, "workspace");
constexpr u64 DEADLINE_TICKS = 50000000ull;

constexpr u32 L_LUTD = 0;
constexpr u32 L_LUTC = 1024;
constexpr u32 L_BIG  = 3072;
constexpr u32 L_FINE = L_BIG + 4 * RUN;
constexpr u32 L_ZERO = L_BIG + 131072;
constexpr u32 V2S = CAP2 + 16, V3S = CAP3 + 16, EVS = 1040;
constexpr u32 WSCR   = V2S + V3S + EVS;
constexpr u32 L_SCR  = L_ZERO;
constexpr u32 L_CTL  = L_SCR + 8 * WSCR;
constexpr u32 C_WB = L_CTL, C_LANDED = L_CTL + 4, C_DEAD = L_CTL + 8, C_PUB = L_CTL + 12, C_DONE = L_CTL + 16;
constexpr u32 C_HDR = L_CTL + 32, C_RDY = L_CTL + 64, C_START = L_CTL + 96, C_NEXT = L_CTL + 100;
constexpr u32 L_END  = L_CTL + 192;
static_assert(L_FINE + 8192 <= L_ZERO && WSCR % 16 == 0 && L_END % 16 == 0 && L_END <= 160 * 1024, "LDS");
static_assert(CAP2 / 4 + CAP3 / 4 + CAP3 + 16 <= EVS, "section scratch");

struct Args {
    const u8* x;
    const u8* w;
    const float* sx;
    const float* sw;
    float* out;
    u8* ws[NR];
    int rank;
    int pad;
};

#define ST16(voff, data, sbase, imm) \
    asm volatile("global_store_dwordx4 %0, %1, %2 offset:" #imm " sc0 sc1\n\ts_nop 1" \
                 :: "v"(voff), "v"(data), "s"(sbase) : "memory")
#define ST4(voff, data, sbase) \
    asm volatile("global_store_dword %0, %1, %2 sc0 sc1" :: "v"(voff), "v"(data), "s"(sbase) : "memory")
#define DMA16(voff, ldsaddr, sbase) do { u32 keep_; \
    asm volatile("s_mov_b32 %0, m0\n\ts_mov_b32 m0, %2\n\ts_nop 0\n\t" \
                 "global_load_lds_dwordx4 %1, %3 sc0 sc1\n\ts_mov_b32 m0, %0" \
                 : "=&s"(keep_) : "v"(voff), "s"(ldsaddr), "s"(sbase) : "memory"); } while (0)
#define WAITV(n) asm volatile("s_waitcnt vmcnt(%0)" :: "n"(n) : "memory")
#define WAITL() asm volatile("s_waitcnt lgkmcnt(0)" ::: "memory")
#define BAR() asm volatile("s_waitcnt lgkmcnt(0)\n\ts_barrier" ::: "memory")
#define RLX_WG __ATOMIC_RELAXED, __HIP_MEMORY_SCOPE_WORKGROUP
#define RLX_AG __ATOMIC_RELAXED, __HIP_MEMORY_SCOPE_AGENT
#define RLX_SYS __ATOMIC_RELAXED, __HIP_MEMORY_SCOPE_SYSTEM
#define CFENCE() asm volatile("" ::: "memory")

__device__ __forceinline__ u64 rt() { return __builtin_amdgcn_s_memrealtime(); }
__device__ __forceinline__ u32 uni(u32 v) { return __builtin_amdgcn_readfirstlane(v); }
__device__ __forceinline__ u32 perm(u32 hi, u32 lo, u32 sel) { return __builtin_amdgcn_perm(hi, lo, sel); }
__device__ __forceinline__ u32 scan64(u32 v) {
    v += __builtin_amdgcn_update_dpp(0u, v, 0x111, 0xf, 0xf, true);
    v += __builtin_amdgcn_update_dpp(0u, v, 0x112, 0xf, 0xf, true);
    v += __builtin_amdgcn_update_dpp(0u, v, 0x114, 0xf, 0xf, true);
    v += __builtin_amdgcn_update_dpp(0u, v, 0x118, 0xf, 0xf, true);
    v += __builtin_amdgcn_update_dpp(0u, v, 0x142, 0xa, 0xf, false);
    v += __builtin_amdgcn_update_dpp(0u, v, 0x143, 0xc, 0xf, false);
    return v;
}
__device__ __forceinline__ u32 last(u32 v) { return __builtin_amdgcn_readlane(v, 63); }
__device__ __forceinline__ u32 escbits(u32 q) { return ~(q | (q >> 1)) & 0x55555555u; }
__device__ __forceinline__ u32 classify8(u32 e4, u32 kadd, u32 thi, u32 tlo) {
    const u32 r = perm(thi, tlo, ((e4 >> 2) + kadd) & 0x1F1F1F1Fu);
    return __builtin_amdgcn_udot4((r ^ (r >> 2)) & 0x18181818u, 0x40100401u, 0u, false);
}
constexpr u32 T13 = 0x00181008u, T2L = 0x00000818u, T2H = 0x00001000u;
__device__ __forceinline__ u32 rep(u32 b) { return b * 0x01010101u; }
__device__ __forceinline__ float silu(float acc, float sc) {
    const float y = acc * sc;
    const float e = __expf(-__builtin_amdgcn_fmed3f(y, -60.f, 60.f));
    return y * __builtin_amdgcn_rcpf(1.f + e);
}
__device__ __forceinline__ int cvt_i32(float f) { int r; asm("v_cvt_i32_f32 %0, %1" : "=v"(r) : "v"(f)); return r; }

__global__ void __launch_bounds__(NTHR) fused_gemm_rs(Args a) {
    __shared__ __attribute__((aligned(16))) u8 lds_raw[L_END];
    LAS u8* const L = (LAS u8*)lds_raw;
#define L8(off) (*(LAS u8*)(L + (off)))
#define L16(off) (*(LAS unsigned short*)(L + (off)))
#define L32(off) (*(LAS u32*)(L + (off)))
#define L64(off) (*(LAS u32x2*)(L + (off)))
#define L128(off) (*(LAS u32x4*)(L + (off)))
#define LPOLL(off) uni(__hip_atomic_load((LAS u32*)(L + (off)), RLX_WG))
    const u32 bid = blockIdx.x;
    const u32 lane = threadIdx.x & 63;
    const u32 wave = uni(threadIdx.x >> 6);
    const u32 rank = a.rank;
    const bool is_link = (bid >> 3) < 7;
    u8* const myws = a.ws[rank];
    const u32 lbase = (u32)(size_t)L;
    const u64 t_begin = rt();
#define STAMP(i)

    const u32 cnt_v = __hip_atomic_load((GAS u32*)(myws + OFF_CNT + bid * 128), RLX_AG);
#define GET_CALL const u32 call = uni(cnt_v) + 1, par = call & 1
    u32 spins = 0;
    auto expired = [&](u32 call) -> bool {
        if (++spins & 15) return false;
        return uni(__hip_atomic_load((GAS u32*)(myws + OFF_TMO), RLX_AG)) == call || rt() - t_begin > DEADLINE_TICKS;
    };
    auto give_up = [&](u32 call) {
        if (lane == 0) {
            __hip_atomic_store((GAS u32*)(myws + OFF_TMO), call, RLX_AG);
            __hip_atomic_store((LAS u32*)(L + C_DEAD), 1u, RLX_WG);
        }
    };
    auto lds_init = [&]() {
        for (u32 o = L_ZERO + threadIdx.x * 16; o < L_END; o += 576 * 16) L128(o) = u32x4{0, 0, 0, 0};
        if (threadIdx.x < 256) {
            const u32 x = threadIdx.x;
            u32 d = 0, c = 0x0C0C0C0Cu, n = 0;
#pragma unroll
            for (int b = 0; b < 4; ++b) {
                const u32 code = (x >> (2 * b)) & 3;
                d |= (code ? 4 + code : n) << (8 * b);
                if (!code) c = (c & ~(0xFFu << (8 * n))) | (b << (8 * n));
                n += !code;
            }
            L32(L_LUTD + x * 4) = d;
            L64(L_LUTC + x * 8) = u32x2{c, n};
        }
    };
    const u32 li = bid & 7;
    auto run_id = [&](u32 n) -> u32 {
        if (n >= 14 && n < 46) return rank * 32 + n - 14;
        const u32 j = n < 14 ? n : n - 32, l = j / 14, r = j - l * 14;
        return 256 + ((rank + 1 + (r >> 1)) & 7) * 8 + 2 * l + (r & 1);
    };

    if (wave == 9) {
        const u32 q = (rank + 1 + bid) & 7, t = (bid >> 3) & 3, rg = bid >> 5;
        const u8* src = a.x + (size_t)(q * MS + rg * 64 + (lane & 15)) * KS + t * 128 + (lane >> 4) * 32;
        u32x4 v[4][2];
#pragma unroll
        for (int rb = 0; rb < 4; ++rb)
#pragma unroll
            for (int p = 0; p < 2; ++p) v[rb][p] = *(const GAS u32x4*)(src + rb * 16 * KS + p * 16);
        GET_CALL;
        asm volatile("s_barrier" ::: "memory");
        asm volatile("" :: "v"(v[0][0]), "v"(v[0][1]), "v"(v[1][0]), "v"(v[1][1]),
                          "v"(v[2][0]), "v"(v[2][1]), "v"(v[3][0]), "v"(v[3][1]));
        u8* dst = a.ws[q];
        const u32 vo = OFF_XO + (par * 4 + (rg >> 1)) * PSTR + (rank * 4 + t) * 16384 + (rg & 1) * 8192 + lane * 16 + 4096;
#define XW() WAITV(XWIN - 1)
        ST16(vo, v[0][0], dst, -4096); XW(); ST16(vo, v[0][1], dst, -3072); XW();
        ST16(vo, v[1][0], dst, -2048); XW(); ST16(vo, v[1][1], dst, -1024); XW();
        ST16(vo, v[2][0], dst, 0);     XW(); ST16(vo, v[2][1], dst, 1024);  XW();
        ST16(vo, v[3][0], dst, 2048);  XW(); ST16(vo, v[3][1], dst, 3072);
        WAITV(0);
        if (lane == 0) __hip_atomic_store((GAS u32*)(dst + OFF_XF + par * 1024 + rg * 128 + (rank * 4 + t) * 4), call, RLX_SYS);
        STAMP(1);
        return;
    }
    if (wave == 10) {
        GET_CALL;
        asm volatile("s_barrier" ::: "memory");
        if (!is_link) return;
        const u32 s = (rank + 1 + (bid >> 3)) & 7;
        for (u32 k = 0;; ++k) {
            u32 n = li;
            if (k) {
                while (LPOLL(C_START) < k) __builtin_amdgcn_s_sleep(2);
                u32 tk = 0;
                if (lane == 0) tk = __hip_atomic_fetch_add((GAS u32*)(myws + OFF_Q + (8 + (bid >> 3)) * 128), 1u, RLX_AG);
                n = 8 + uni(tk) - NITEM * (call - 1);
            }
            u32 lines = 0;
            if (n < NITEM) {
                for (;;) {
                    const u64 f = __hip_atomic_load((GAS u64*)(myws + OFF_F + (par * 8 + s) * 2048 + n * 8), RLX_SYS);
                    lines = uni((u32)(f >> 32)) & 0xFF;
                    if (uni((u32)f) >= call) break;
                    if (expired(call)) { give_up(call); n = 255; break; }
                    __builtin_amdgcn_s_sleep(2);
                }
            } else {
                n = 255;
            }
            CFENCE();
            if (lane == 0) __hip_atomic_store((LAS u32*)(L + C_RDY + (k & 7) * 4), ((k + 1) << 16) | (n << 8) | lines, RLX_WG);
            if (n == 255) return;
        }
    }
    if (wave == 8) {
        lds_init();
        GET_CALL;
        BAR();
        if (!is_link) return;
        const u8* base = a.ws[(rank + 1 + (bid >> 3)) & 7];
        u32 hist[WINDOW];
#pragma unroll
        for (int k = 0; k < WINDOW; ++k) hist[k] = 0;
        auto publish = [&](u32 v) { if (lane == 0) __hip_atomic_store((LAS u32*)(L + C_LANDED), v, RLX_WG); };
        auto drain = [&]() {
            WAITV(0);
#pragma unroll
            for (int k = 1; k < WINDOW; ++k) hist[k] = hist[0];
            publish(hist[0]);
        };
        auto pull = [&](u32 goff, u32 loff, u32 bytes, u32 mark) {
            if (lane * 16 < bytes) DMA16(goff + lane * 16, lbase + loff, base);
#pragma unroll
            for (int k = WINDOW - 1; k > 0; --k) hist[k] = hist[k - 1];
            hist[0] = mark;
            WAITV(WINDOW - 1);
            publish(hist[WINDOW - 1]);
        };
        for (u32 m = 0;; ++m) {
            u32 rdy = LPOLL(C_RDY + (m & 7) * 4);
            const u32 dn = C_DONE + (m & 3) * 4, need = 8 * (m >> 2);
            if ((rdy >> 16) != m + 1 || LPOLL(dn) < need) {
                drain();
                while (((rdy = LPOLL(C_RDY + (m & 7) * 4)) >> 16) != m + 1 || LPOLL(dn) < need) __builtin_amdgcn_s_sleep(1);
            }
            CFENCE();
            const u32 n = (rdy >> 8) & 0xFF;
            if (n == 255) break;
            if (lane == 0) __hip_atomic_store((LAS u32*)(L + C_START), m + 1, RLX_WG);
            const bool is_out = n >= 88;
            const u32 g0 = is_out ? OFF_OP + par * OP_PAR + (rank * 128 + n - 88) * OHALF : OFF_P + par * P_PAR + run_id(n) * RUN;
            const u32 cap = is_out ? EXCCAP : VARCAP;
            const u32 l0 = L_BIG + (m & 3) * RUN;
            u32 vb = (rdy & 0xFF) * 128;
            vb = vb < 128 ? 128 : (vb > cap ? cap : vb);
            for (u32 o = 0; o < vb; o += 1024) pull(g0 + o, l0 + o, vb - o, m * 64 + (o + 1024 >= vb ? 8 : 0));
            for (u32 o = cap, p = m * 64 + 9; o < RUN; o += 1024, ++p) pull(g0 + o, l0 + o, 1024, p);
        }
#pragma unroll
        for (int k = WINDOW - 2; k >= 0; --k) {
            switch (k) { case 0: WAITV(0); break; case 1: WAITV(1); break; case 2: WAITV(2); break; case 3: WAITV(3); break;
                         case 4: WAITV(4); break; case 5: WAITV(5); break; case 6: WAITV(6); break; default: WAITV(7); break; }
            if (k <= 7) publish(hist[k]);
        }
        WAITV(0);
        publish(0x7FFFFFFFu);
        STAMP(21);
        return;
    }

    u32 phase = 0;
    auto wbar = [&]() {
        phase += 8;
        WAITL();
        if (lane == 0) __hip_atomic_fetch_add((LAS u32*)(L + C_WB), 1u, RLX_WG);
        while (LPOLL(C_WB) < phase) __builtin_amdgcn_s_sleep(1);
        CFENCE();
    };
    auto append = [&](auto G, u32 pos, const u32* e4, const u32x2* lut) {
        u32 carry = 0;
#pragma unroll
        for (int i = 0; i < G.value; ++i) {
            const u32 comp = perm(0u, e4[i], lut[i][0]), sh = pos << 3;
            __hip_atomic_fetch_or((LAS u32*)(L + (pos & ~3u)), (comp << (sh & 31)) | carry, RLX_WG);
            carry = __builtin_amdgcn_ubfe(comp, 0u - sh, sh);
            pos += lut[i][1];
        }
        __hip_atomic_fetch_or((LAS u32*)(L + (pos & ~3u)), carry, RLX_WG);
    };
    auto expand = [&](u32 tab, u32 pos, u32 x) -> u32 {
        const u32 sel = L32(L_LUTD + x * 4);
        const LAS u32* p = (const LAS u32*)(L + (pos & ~3u));
        return perm(tab, __builtin_amdgcn_alignbyte(p[1], p[0], pos & 3), sel);
    };
    auto panel_of = [&](u32 id) -> u32 { return id >= 256 ? (id - 256) >> 1 : 32 + ((id & 31) >> 1); };
    auto stage_off = [&](u32 id, u32 s) -> u32 {
        return OFF_SP + panel_of(id) * PSTR + (s * 4 + (wave >> 1)) * 16384 + (id & 1) * 8192 + (wave & 1) * 1024 + lane * 16 + 4096;
    };
    auto announce = [&](u32 id, u32 call) {
        if (lane == 0 && __hip_atomic_fetch_add((GAS u32*)(myws + OFF_CT + panel_of(id) * 128), 1u, RLX_AG) == 128 * call - 1)
            __hip_atomic_store((GAS u32*)(myws + OFF_PF + panel_of(id) * 128), call, RLX_AG);
    };
    const u32 v2 = L_SCR + wave * WSCR, v3 = v2 + V2S, ev = v3 + V3S;
    u8* peer = a.ws[lane & 7];
    u32 npub = 0;

    const u32 tile = (((bid & ~15u) | ((bid & 7) << 1) | ((bid >> 3) & 1)) + 32 * (rank + 1)) & 255;
    const u32 xrun = 256 + ((bid >> 1) & 7) * 8 + (7 - (bid >> 5)) * 4 + ((bid >> 4) & 1) * 2 + (bid & 1);
    for (u32 job = 0; job < (bid >= 192 ? 2u : 1u); ++job) {
        const u32 id = (bid >= 192 && job == (bid >= 224 ? 0u : 1u)) ? xrun : tile;
        const u32 t = wave >> 1, h = wave & 1;
        u32 f[4][4];
        if (id >= 256) {
            const u8* src = a.x + (size_t)((id - 256) * 64 + (lane & 15)) * KS + t * 128 + (lane >> 4) * 32 + h * 16;
#pragma unroll
            for (int c = 0; c < 4; ++c) {
                const u32x4 v = *(const GAS u32x4*)(src + c * 16 * KS);
                f[c][0] = v[0]; f[c][1] = v[1]; f[c][2] = v[2]; f[c][3] = v[3];
            }
        } else {
            const u8* src = a.w + (size_t)(t * 128 + (lane >> 4) * 32 + h * 16) * GN + id * 64 + (lane & 15) * 4;
#pragma unroll
            for (int r = 0; r < 16; ++r) f[r >> 2][r & 3] = *(const GAS u32*)(src + (size_t)r * GN);
        }
        if (job == 0) {
            lds_init();
            BAR();
        } else {
            for (u32 o = lane * 16; o < WSCR; o += 1024) L128(v2 + o) = u32x4{0, 0, 0, 0};
        }
        __builtin_amdgcn_sched_barrier(0);
        if (id < 256) {
            u32 g[4][4];
#pragma unroll
            for (int i = 0; i < 4; ++i) {
                const u32 t0 = perm(f[i][1], f[i][0], 0x05010400u), t1 = perm(f[i][1], f[i][0], 0x07030602u);
                const u32 t2 = perm(f[i][3], f[i][2], 0x05010400u), t3 = perm(f[i][3], f[i][2], 0x07030602u);
                g[0][i] = perm(t2, t0, 0x05040100u); g[1][i] = perm(t2, t0, 0x07060302u);
                g[2][i] = perm(t3, t1, 0x05040100u); g[3][i] = perm(t3, t1, 0x07060302u);
            }
#pragma unroll
            for (int i = 0; i < 16; ++i) f[i >> 2][i & 3] = g[i >> 2][i & 3];
        }
        GET_CALL;
        asm volatile("" : "+v"(peer));
        const u32x4 f0 = {f[0][0], f[0][1], f[0][2], f[0][3]}, f1 = {f[1][0], f[1][1], f[1][2], f[1][3]};
        const u32x4 f2 = {f[2][0], f[2][1], f[2][2], f[2][3]}, f3 = {f[3][0], f[3][1], f[3][2], f[3][3]};
        const bool mine = id >= 256 ? ((id - 256) >> 3) != rank : (id >> 5) == rank;
        auto keep = [&]() {
            const u32 vo = stage_off(id, rank);
            ST16(vo, f0, myws, -4096); ST16(vo, f1, myws, -2048); ST16(vo, f2, myws, 0); ST16(vo, f3, myws, 2048);
            WAITV(0);
            announce(id, call);
        };
        if (id < 256 && mine) {
            keep();
            continue;
        }
        u32 esum = 0;
#pragma unroll
        for (int i = 0; i < 16; ++i) esum = __builtin_amdgcn_udot4(f[i >> 2][i & 3] & 0x7C7C7C7Cu, 0x01010101u, esum, false);
        const u32 e0 = ((last(scan64(esum)) + 14746u) >> 14) & 31;
        const u32 runo = OFF_P + par * P_PAR + id * RUN;
        u32x4 Q, Mq;
        u32 S[2] = {0, 0};
        u32 e4[16];
        u32x2 lut[16];
        const u32 k1 = rep((33 - e0) & 31);
#pragma unroll
        for (int c = 0; c < 4; ++c) {
            u32 q = 0, m = 0;
#pragma unroll
            for (int i = 0; i < 4; ++i) {
                e4[4 * c + i] = f[c][i] & 0x7C7C7C7Cu;
                const u32 x8 = classify8(e4[4 * c + i], k1, 0u, T13);
                lut[4 * c + i] = L64(L_LUTC + x8);
                q |= i ? x8 << (8 * i - 3) : x8 >> 3;
                m |= (f[c][i] & 0x03030303u) << (2 * i);
                S[c >> 1] |= (f[c][i] & 0x80808080u) >> (4 * (c & 1) + i);
            }
            Q[c] = q; Mq[c] = m;
        }
        {
            const u32 vo = runo + VARCAP + wave * HALFB + lane * 16;
            ST16(vo, Q, myws, 0);
            ST16(vo, Mq, myws, 1024);
            const u32x4 s4 = {S[0], S[1], (u32)__shfl_xor((int)S[0], 32), (u32)__shfl_xor((int)S[1], 32)};
            if (lane < 32) ST16(vo, s4, myws, 2048);
        }
        u32 n = 0;
#pragma unroll
        for (int c = 0; c < 4; ++c) n += __builtin_popcount(escbits(Q[c]));
        const u32 inc = scan64(n);
        const u32 N2 = last(inc);
        u32 len = 0;
        bool raw = N2 > CAP2;
        if (!raw) {
            append(std::integral_constant<int, 16>{}, v2 + inc - n, e4, lut);
            const u32 k2 = rep((35 - e0) & 31);
            u32 u[6], m = 0;
            u32x2 l2[6];
#pragma unroll
            for (int i = 0; i < 3; ++i) {
                const u32x2 v = L64(v2 + lane * 24 + i * 8);
                u[2 * i] = v[0]; u[2 * i + 1] = v[1];
            }
#pragma unroll
            for (int i = 0; i < 3; ++i) {
                u32 q8 = classify8(u[2 * i], k2, T2H, T2L) | (classify8(u[2 * i + 1], k2, T2H, T2L) << 8);
                const int valid = (int)N2 - (int)lane * 24 - i * 8;
                q8 |= (0x5555u * 8) << (2 * min(max(valid, 0), 8));
                if (valid > 0) L16(ev + lane * 6 + i * 2) = (unsigned short)(q8 >> 3);
                l2[2 * i] = L64(L_LUTC + (q8 & 0x7F8));
                l2[2 * i + 1] = L64(L_LUTC + ((q8 >> 8) & 0x7F8));
            }
#pragma unroll
            for (int i = 0; i < 6; ++i) m += l2[i][1];
            const u32 in2 = scan64(m);
            const u32 N3 = last(in2);
            raw = N3 > CAP3;
            if (!raw) {
                if (m) append(std::integral_constant<int, 6>{}, v3 + in2 - m, u, l2);
                const u32 k3 = rep((38 - e0) & 31);
                const u32 off3 = ((N2 + 7) >> 3) * 2, off4 = off3 + ((N3 + 7) >> 3) * 2;
                const u32x2 v = L64(v3 + lane * 8);
                const u32 w[2] = {v[0], v[1]};
                u32 q8 = classify8(w[0], k3, 0u, T13) | (classify8(w[1], k3, 0u, T13) << 8);
                const int valid = (int)N3 - (int)lane * 8;
                q8 |= (0x5555u * 8) << (2 * min(max(valid, 0), 8));
                if (valid > 0) L16(ev + off3 + lane * 2) = (unsigned short)(q8 >> 3);
                const u32x2 l3[2] = {L64(L_LUTC + (q8 & 0x7F8)), L64(L_LUTC + ((q8 >> 8) & 0x7F8))};
                const u32 m3 = l3[0][1] + l3[1][1];
                const u32 in3 = scan64(m3);
                if (m3) append(std::integral_constant<int, 2>{}, ev + off4 + in3 - m3, w, l3);
                len = off4 + last(in3);
                raw = len > SECCAP;
            }
        }
        if (raw) {
            len = 0;
            const u32 vo = OFF_RAW + par * RAW_PAR + (id * 8 + wave) * 4096 + lane * 16;
            ST16(vo, f0, myws, 0); ST16(vo, f1, myws, 1024); ST16(vo, f2, myws, 2048); ST16(vo, f3, myws, 3072);
        }
        const u32 len16 = (len + 15) >> 4;
        if (lane == 0) L32(C_HDR + wave * 4) = len16 | (raw ? 0x80000000u : (N2 << 9) | (e0 << 20));
        wbar();
        const u32 hv = lane < 8 ? L32(C_HDR + lane * 4) : 0u;
        const u32 hin = scan64(hv & 0x1FF);
        const u32 hoff = 2 + hin - (hv & 0x1FF);
        if (lane < len16) {
            const u32x4 sec = L128(ev + lane * 16);
            ST16(runo + __builtin_amdgcn_readlane(hoff, wave) * 16 + lane * 16, sec, myws, 0);
        }
        if (wave == 0 && lane < 8) ST4(runo + lane * 4, (hv & ~0x1FFu) | hoff, myws);
        WAITV(0);
        u32 arrived = 0;
        npub += 8;
        if (lane == 0) arrived = __hip_atomic_fetch_add((LAS u32*)(L + C_PUB), 1u, RLX_WG);
        if (uni(arrived) == npub - 1) {
            const u64 fl = ((u64)((2 + last(hin) + 7) >> 3) << 32) | call;
            u32 slot;
            bool to;
            if (id >= 256) {
                const u32 q = (id - 256) >> 3, j = ((id >> 1) & 3) * 14 + ((q - lane - 1) & 7) * 2 + (id & 1);
                slot = j < 14 ? j : j + 32;
                to = lane < 8 && lane != rank && lane != q;
            } else {
                slot = 14 + (id & 31);
                to = lane == (id >> 5);
            }
            if (to) __hip_atomic_store((GAS u64*)(peer + OFF_F + (par * 8 + rank) * 2048 + slot * 8), fl, RLX_SYS);
            STAMP(22 + job);
        }
        if (mine) keep();
        if (bid >= 192 && job == 0) wbar();
    }

    GET_CALL;
    const float sc = a.sx[0] * a.sw[0];
    const u32 rg = wave >> 1, ch = wave & 1;
    auto out_ptr = [&](u32 row0, u32 col0) -> float* {
        return a.out + (size_t)(row0 + rg * 32 + (lane >> 4) * 4) * GN + col0 + ch * 64 + (lane & 15) * 4;
    };
    auto out_store = [&](float* o, int g, f32x4 v) {
        asm volatile("global_store_dwordx4 %0, %1, off sc1\n\ts_nop 1" :: "v"(o + (size_t)((g >> 2) * 16 + (g & 3)) * GN), "v"(v) : "memory");
    };

    if (is_link) {
        const u32 s = (rank + 1 + (bid >> 3)) & 7;
        const u8* base = a.ws[s];
        auto landed = [&](u32 mark) {
            while (LPOLL(C_LANDED) < mark) __builtin_amdgcn_s_sleep(1);
            CFENCE();
        };
        auto done = [&](u32 m) {
            WAITL();
            if (lane == 0) __hip_atomic_fetch_add((LAS u32*)(L + C_DONE + (m & 3) * 4), 1u, RLX_WG);
        };
        u32 m = 0, n;
        for (;; ++m) {
            landed(m * 64 + 8);
            n = (LPOLL(C_RDY + (m & 7) * 4) >> 8) & 0xFF;
            if (n >= 88) break;
            const u32 id = run_id(n), vb = L_BIG + (m & 3) * RUN;
            const u32 hdr = uni(L32(vb + wave * 4));
            u32x4 r[4];
            if (hdr >> 31) {
                const u32 vo = OFF_RAW + par * RAW_PAR + (id * 8 + wave) * 4096 + lane * 16;
                asm volatile("global_load_dwordx4 %0, %4, %5 sc0 sc1\n\tglobal_load_dwordx4 %1, %4, %5 offset:1024 sc0 sc1\n\t"
                             "global_load_dwordx4 %2, %4, %5 offset:2048 sc0 sc1\n\tglobal_load_dwordx4 %3, %4, %5 offset:3072 sc0 sc1\n\t"
                             "s_waitcnt vmcnt(0)"
                             : "=&v"(r[0]), "=&v"(r[1]), "=&v"(r[2]), "=&v"(r[3]) : "v"(vo), "s"(base) : "memory");
            } else {
                const u32 e0 = (hdr >> 20) & 31, N2 = (hdr >> 9) & 0x7FF, sec = vb + (hdr & 0x1FF) * 16;
                const u32 t1 = (((e0 - 1) & 31) << 10) | (e0 << 18) | (((e0 + 1) & 31) << 26);
                const u32 t2 = (((e0 - 2) & 31) << 10) | (((e0 + 2) & 31) << 18) | (((e0 - 3) & 31) << 26);
                const u32 t3 = (((e0 - 6) & 31) << 10) | (((e0 - 5) & 31) << 18) | (((e0 - 4) & 31) << 26);
                {
                    u32 q[3], e[3], n = 0;
#pragma unroll
                    for (int i = 0; i < 3; ++i) {
                        q[i] = L16(sec + lane * 6 + i * 2) | 0x55550000u;
                        if (lane * 24 + i * 8 >= N2) q[i] = 0x55555555u;
                        e[i] = escbits(q[i]);
                        n += __builtin_popcount(e[i]);
                    }
                    const u32 in2 = scan64(n), N3 = last(in2);
                    const u32 off3 = sec + ((N2 + 7) >> 3) * 2, off4 = off3 + ((N3 + 7) >> 3) * 2;
                    u32 q3 = L16(off3 + lane * 2) | 0x55550000u;
                    if (lane * 8 >= N3) q3 = 0x55555555u;
                    const u32 e3 = escbits(q3), m3 = __builtin_popcount(e3);
                    const u32 p3 = off4 + scan64(m3) - m3;
                    L64(v3 + lane * 8) = u32x2{expand(t3, p3, q3 & 0xFF), expand(t3, p3 + __builtin_popcount(e3 & 0xFF), (q3 >> 8) & 0xFF)};
                    u32 pp = v3 + in2 - n;
                    u32x2 z[3];
#pragma unroll
                    for (int i = 0; i < 3; ++i) {
                        z[i] = u32x2{expand(t2, pp, q[i] & 0xFF), expand(t2, pp + __builtin_popcount(e[i] & 0xFF), (q[i] >> 8) & 0xFF)};
                        pp += __builtin_popcount(e[i]);
                    }
#pragma unroll
                    for (int i = 0; i < 3; ++i) L64(v2 + lane * 24 + i * 8) = z[i];
                }
                landed(m * 64 + 8 + (((wave + 1) * HALFB + 1023) >> 10));
                const u32 fx = vb + VARCAP + wave * HALFB;
                const u32x4 Q = L128(fx + lane * 16), Mq = L128(fx + 1024 + lane * 16);
                const u32x2 S = L64(fx + 2048 + (lane & 31) * 16 + (lane >> 5) * 8);
                u32 E[4], n = 0;
#pragma unroll
                for (int c = 0; c < 4; ++c) { E[c] = escbits(Q[c]); n += __builtin_popcount(E[c]); }
                u32 pp = v2 + scan64(n) - n;
#pragma unroll
                for (int c = 0; c < 4; ++c)
#pragma unroll
                    for (int i = 0; i < 4; ++i) {
                        r[c][i] = expand(t1, pp, (Q[c] >> (8 * i)) & 0xFF) | ((Mq[c] >> (2 * i)) & 0x03030303u)
                                  | ((S[c >> 1] << (4 * (c & 1) + i)) & 0x80808080u);
                        pp += __builtin_popcount(E[c] & (0xFFu << (8 * i)));
                    }
            }
            const u32 vo = stage_off(id, s);
            ST16(vo, r[0], myws, -4096); ST16(vo, r[1], myws, -2048); ST16(vo, r[2], myws, 0); ST16(vo, r[3], myws, 2048);
            done(m);
            WAITV(0);
            announce(id, call);
        }
        const u32 u = wave >> 1, rb = wave & 1, fine = L_FINE + wave * 1024;
        while (n != 255) {
            const u32 id = n - 88, vb = L_BIG + (m & 3) * RUN;
            float* const o0 = a.out + (size_t)((id >> 5) * 128 + (id & 1) * 64 + (u >> 1) * 32 + rb * 16) * GN
                              + s * 2048 + ((id >> 1) & 15) * 128 + (u & 1) * 64;
            float* const o = o0 + (size_t)(lane >> 4) * 4 * GN + (lane & 15) * 4;
            const u32 hdr = uni(L32(vb + u * 4));
            if (hdr >> 31) {
                done(m);
                const u32 vo = OFF_OR + par * OR_PAR + ((rank * 128 + id) * 8 + wave) * 4096 + lane * 16;
                f32x4 r[4];
                asm volatile("global_load_dwordx4 %0, %4, %5 sc0 sc1\n\tglobal_load_dwordx4 %1, %4, %5 offset:1024 sc0 sc1\n\t"
                             "global_load_dwordx4 %2, %4, %5 offset:2048 sc0 sc1\n\tglobal_load_dwordx4 %3, %4, %5 offset:3072 sc0 sc1\n\t"
                             "s_waitcnt vmcnt(0)"
                             : "=&v"(r[0]), "=&v"(r[1]), "=&v"(r[2]), "=&v"(r[3]) : "v"(vo), "s"(base) : "memory");
#pragma unroll
                for (int k = 0; k < 4; ++k)
                    out_store(o, k, f32x4{silu(r[k][0], sc), silu(r[k][1], sc), silu(r[k][2], sc), silu(r[k][3], sc)});
            } else {
                const u32 sec = vb + (hdr & 0x3F) * 16, n1 = (hdr >> 9) & 0x7F, n2 = (hdr >> 17) & 31;
                L128(fine + lane * 16) = u32x4{0, 0, 0, 0};
#pragma unroll
                for (u32 b = 0; b < 128; b += 64)
                    if (b + lane < n1) {
                        const u32 e = L16(sec + (b + lane) * 2);
                        if (((e >> 9) & 1) == rb) L8(fine + (e >> 10) * 16 + ((e >> 5) & 15)) = (u8)(e << 3);
                    }
                u32x2 hard = {(rb ^ 1) << 4, 0};
                if (lane < n2) { const LAS u32* p = (const LAS u32*)(L + sec + ((n1 + 1) >> 1) * 4 + lane * 8); hard = u32x2{p[0], p[1]}; }
                const u32x4 F = L128(fine + lane * 16);
                landed(m * 64 + 8 + (wave + 1) * 3);
                u32 d[12];
#pragma unroll
                for (int k = 0; k < 3; ++k) {
                    const u32x4 v = L128(vb + EXCCAP + wave * 3072 + k * 1024 + lane * 16);
                    d[4 * k] = v[0]; d[4 * k + 1] = v[1]; d[4 * k + 2] = v[2]; d[4 * k + 3] = v[3];
                }
                done(m);
                const float dn = __uint_as_float(((hdr >> 22) & 0xFF) << 23);
#pragma unroll
                for (int g = 0; g < 4; ++g) {
                    const u32 d0 = d[3 * g], d1 = d[3 * g + 1], d2 = d[3 * g + 2];
                    const int k0 = perm(d0, F[g], 0x07060500u), k1 = perm(d1, F[g], 0x07060501u), k2 = perm(d2, F[g], 0x07060502u);
                    const int k3 = perm(perm(d2, d1, 0x04000000u), perm(d0, F[g], 0x00000403u), 0x07060100u);
                    out_store(o, g, f32x4{silu((float)k0 * dn, sc), silu((float)k1 * dn, sc), silu((float)k2 * dn, sc), silu((float)k3 * dn, sc)});
                }
                if (n2) {
                    WAITV(0);
                    if (((hard[0] >> 4) & 1) == rb) {
                        const u32 l = (hard[0] >> 5) & 63;
                        float* e = o0 + (size_t)((l >> 4) * 4 + ((hard[0] >> 2) & 3)) * GN + (l & 15) * 4 + (hard[0] & 3);
                        asm volatile("global_store_dword %0, %1, off sc1" :: "v"(e), "v"(silu(__uint_as_float(hard[1]), sc)) : "memory");
                    }
                }
            }
            ++m;
            landed(m * 64 + 8);
            n = (LPOLL(C_RDY + (m & 7) * 4) >> 8) & 0xFF;
        }
        if (LPOLL(C_DEAD) || uni(__hip_atomic_load((GAS u32*)(myws + OFF_TMO), RLX_AG)) == call) {
            WAITV(0);
            for (u32 r = wave * 64 + lane; r < MS; r += 512)
                for (u32 c = 0; c < 2048; c += 64) a.out[(size_t)r * GN + s * 2048 + c] = __builtin_nanf("");
        }
    } else {
        for (u32 n = (bid >> 3) - 7, round = 0; n < 64; ++round) {
            const u32 layer = n < 56 ? n / 14 : (n - 56) >> 1, qi = n < 56 ? (n % 14) >> 1 : 7, ct = 2 * li + (n & 1);
            const u32 q = (rank + 1 + qi) & 7;
            if (wave == 0) {
                const u32 fo = qi == 7 ? OFF_XF + par * 1024 + layer * 256 + lane * 4 : OFF_PF + (q * 4 + layer) * 128;
                bool ok = !LPOLL(C_DEAD);
                while (ok) {
                    const u32 fa = __hip_atomic_load((GAS u32*)(myws + fo), RLX_SYS);
                    const u32 fb = __hip_atomic_load((GAS u32*)(myws + OFF_PF + (32 + ct) * 128), RLX_SYS);
                    if (__all(fa >= call && fb >= call)) break;
                    if (expired(call)) { give_up(call); ok = false; }
                    __builtin_amdgcn_s_sleep(GSLEEP);
                }
            }
            BAR();
            const bool dead = LPOLL(C_DEAD);
            if (wave == 0) STAMP(44 + min(round, 3u) * 3);
            f32x4 acc[2][4];
#pragma unroll
            for (int i = 0; i < 8; ++i) acc[i >> 2][i & 3] = f32x4{0.f, 0.f, 0.f, 0.f};
            if (!dead) {
                const u32 ga = (qi == 7 ? OFF_XO + (par * 4 + layer) * PSTR : OFF_SP + (q * 4 + layer) * PSTR) + wave * 2048 + lane * 16;
                const u32 gb = OFF_SP + (32 + ct) * PSTR + wave * 2048 + lane * 16;
                auto dma = [&](u32 k, int j) {
                    const u32 l = lbase + L_BIG + (k & 3) * 32768 + wave * 2048 + (j >> 1) * 16384 + (j & 1) * 1024;
                    DMA16((j >> 1 ? gb : ga) + k * 16384 + (j & 1) * 1024, l, myws);
                };
                auto stage = [&](u32 k) { dma(k, 0); dma(k, 1); dma(k, 2); dma(k, 3); };
                u32x4 fr[2][12];
                auto step = [&](auto B, auto MM, auto RD, auto ST, u32 k) {
                    const u32 sa = L_BIG + (k & 3) * 32768 + rg * 4096 + lane * 16, sb = L_BIG + (k & 3) * 32768 + 16384 + ch * 8192 + lane * 16;
#pragma unroll
                    for (int i = 0; i < 8; ++i) {
                        if (MM.value) {
                            const u32x4 *pa = &fr[B.value ^ 1][(i & 1) * 2], *pb = &fr[B.value ^ 1][4 + (i >> 1) * 2];
                            const i32x8 af = {(int)pa[0][0], (int)pa[0][1], (int)pa[0][2], (int)pa[0][3], (int)pa[1][0], (int)pa[1][1], (int)pa[1][2], (int)pa[1][3]};
                            const i32x8 bf = {(int)pb[0][0], (int)pb[0][1], (int)pb[0][2], (int)pb[0][3], (int)pb[1][0], (int)pb[1][1], (int)pb[1][2], (int)pb[1][3]};
                            acc[i & 1][i >> 1] = __builtin_amdgcn_mfma_scale_f32_16x16x128_f8f6f4(af, bf, acc[i & 1][i >> 1], 1, 1, 0, 0x7F7F7F7F, 0, 0x7F7F7F7F);
                        }
                        __builtin_amdgcn_sched_barrier(0);
                        if (RD.value && i < 6) {
                            const u32 o = i < 2 ? sa + i * 2048 : sb + (i - 2) * 2048;
                            fr[B.value][2 * i] = L128(o);
                            fr[B.value][2 * i + 1] = L128(o + 1024);
                        }
                        if (ST.value && (i & 1)) dma(k + 3, i >> 1);
                        __builtin_amdgcn_sched_barrier(0);
                    }
                };
                constexpr std::integral_constant<int, 0> N{};
                constexpr std::integral_constant<int, 1> Y{};
                stage(0); stage(1); stage(2);
                WAITV(8); BAR();
                step(N, N, Y, Y, 0);
                for (u32 k = 1; k < 29; k += 2) {
                    WAITV(8); BAR();
                    step(Y, Y, Y, Y, k);
                    WAITV(8); BAR();
                    step(N, Y, Y, Y, k + 1);
                }
                WAITV(8); BAR();
                step(Y, Y, Y, N, 29);
                WAITV(4); BAR();
                step(N, Y, Y, N, 30);
                WAITV(0); BAR();
                step(Y, Y, Y, N, 31);
                step(N, Y, N, N, 32);
            }
            u32 ticket = 0;
            if (threadIdx.x == 0) ticket = __hip_atomic_fetch_add((GAS u32*)(myws + OFF_Q + li * 128), 1u, RLX_AG);
            if (wave == 0) STAMP(45 + min(round, 3u) * 3);
            if (qi == 7) {
                const bool fail = dead || uni(__hip_atomic_load((GAS u32*)(myws + OFF_TMO), RLX_AG)) == call;
                const float s2 = fail ? __builtin_nanf("") : sc;
                float* o = out_ptr(layer * 128, rank * 2048 + ct * 128);
#pragma unroll
                for (int g = 0; g < 8; ++g)
                    out_store(o, g, f32x4{silu(acc[g >> 2][0][g & 3], s2), silu(acc[g >> 2][1][g & 3], s2),
                                          silu(acc[g >> 2][2][g & 3], s2), silu(acc[g >> 2][3][g & 3], s2)});
                WAITV(0);
            } else if (!dead) {
                const u32 id = (layer * 16 + ct) * 2 + (wave >> 2), to = OFF_OP + par * OP_PAR + (q * 128 + id) * OHALF;
                const u32 fine = L_BIG + wave * 10240, dump = fine + 2048;
                u32 esum = 0;
#pragma unroll
                for (int i = 0; i < 32; ++i) esum += (__float_as_uint(acc[i >> 4][(i >> 2) & 3][i & 3]) >> 19) & 0xFFF;
                const u32 etop = min(max((last(scan64(esum)) + 114688u) >> 15, 40u), 250u);
                const float up = __uint_as_float((285 - etop) << 23), dn = __uint_as_float((etop - 31) << 23);
                u32 d[24], F[8], hardm = 0, softm = 0;
#pragma unroll
                for (int g = 0; g < 8; ++g) {
                    u32 k[4];
#pragma unroll
                    for (int cc = 0; cc < 4; ++cc) {
                        const float v = acc[g >> 2][cc][g & 3];
                        k[cc] = (u32)cvt_i32(v * up) & ~7u;
                        hardm |= (u32)(__float_as_uint((float)(int)k[cc] * dn) != __float_as_uint(v)) << (4 * g + cc);
                    }
                    d[3 * g] = perm(k[3], k[0], 0x03020105u); d[3 * g + 1] = perm(k[3], k[1], 0x03020106u); d[3 * g + 2] = perm(k[3], k[2], 0x03020107u);
                    F[g] = perm(perm(k[3], k[2], 0x04000000u), perm(k[1], k[0], 0x00000400u), 0x07060100u);
                    softm |= ((((F[g] >> 1) + 0x7C7C7C7Cu) & 0x80808080u)) >> g;
                }
                {
                    const u32 vo = to + EXCCAP + (wave & 3) * UNITB + lane * 16 + 2048;
                    ST16(vo, (u32x4{d[0], d[1], d[2], d[3]}), myws, -2048);     ST16(vo, (u32x4{d[4], d[5], d[6], d[7]}), myws, -1024);
                    ST16(vo, (u32x4{d[8], d[9], d[10], d[11]}), myws, 0);       ST16(vo, (u32x4{d[12], d[13], d[14], d[15]}), myws, 1024);
                    ST16(vo, (u32x4{d[16], d[17], d[18], d[19]}), myws, 2048);  ST16(vo, (u32x4{d[20], d[21], d[22], d[23]}), myws, 3072);
                }
                L128(fine + lane * 32) = u32x4{F[0], F[1], F[2], F[3]};
                L128(fine + lane * 32 + 16) = u32x4{F[4], F[5], F[6], F[7]};
                const u32 cnt = __builtin_popcount(softm) | (__builtin_popcount(hardm) << 16);
                const u32 inc = scan64(cnt), tot = last(inc);
                const u32 n1 = tot & 0xFFFF, n2 = tot >> 16, hoff0 = ((n1 + 1) >> 1) * 4;
                u32 len = hoff0 + n2 * 8;
                const bool raw = len > OSECCAP;
                if (raw) {
                    len = 0;
                    const u32 vo = OFF_OR + par * OR_PAR + ((q * 128 + id) * 4 + (wave & 3)) * 8192 + lane * 16 + 4096;
#define ROW(g) (f32x4{acc[g >> 2][0][g & 3], acc[g >> 2][1][g & 3], acc[g >> 2][2][g & 3], acc[g >> 2][3][g & 3]})
                    ST16(vo, ROW(0), myws, -4096); ST16(vo, ROW(1), myws, -3072); ST16(vo, ROW(2), myws, -2048); ST16(vo, ROW(3), myws, -1024);
                    ST16(vo, ROW(4), myws, 0);     ST16(vo, ROW(5), myws, 1024);  ST16(vo, ROW(6), myws, 2048);  ST16(vo, ROW(7), myws, 3072);
                } else {
                    u32 o = ev + ((inc - cnt) & 0xFFFF) * 2;
                    while (softm) {
                        const u32 b = __builtin_ctz(softm), p = lane * 32 + (7 - (b & 7)) * 4 + (b >> 3);
                        softm &= softm - 1;
                        L16(o) = (unsigned short)((p << 5) | (L8(fine + p) >> 3));
                        o += 2;
                    }
                    if (n2) {
#pragma unroll
                        for (int g = 0; g < 8; ++g) *(LAS f32x4*)(L + dump + g * 1024 + lane * 16) = ROW(g);
                        o = ev + hoff0 + ((inc - cnt) >> 16) * 8;
                        while (hardm) {
                            const u32 b = __builtin_ctz(hardm);
                            hardm &= hardm - 1;
                            L32(o) = lane * 32 + b;
                            L32(o + 4) = L32(dump + (b >> 2) * 1024 + lane * 16 + (b & 3) * 4);
                            o += 8;
                        }
                    }
                }
                const u32 len16 = (len + 15) >> 4;
                if (lane == 0) L32(C_HDR + wave * 4) = len16 | (raw ? 0x80000000u : (n1 << 9) | (n2 << 17) | ((etop - 31) << 22));
                BAR();
                const u32 hv = lane < 4 ? L32(C_HDR + (wave & 4) * 4 + lane * 4) : 0u;
                const u32 hin = scan64(hv & 0x1FF);
                const u32 hoff = 1 + hin - (hv & 0x1FF);
                if (lane < len16) {
                    const u32x4 sec = L128(ev + lane * 16);
                    ST16(to + __builtin_amdgcn_readlane(hoff, wave & 3) * 16 + lane * 16, sec, myws, 0);
                }
                if ((wave & 3) == 0 && lane < 4) ST4(to + lane * 4, (hv & ~0x1FFu) | hoff, myws);
                WAITV(0);
                BAR();
                if ((wave & 3) == 1 && lane == 0) {
                    const u64 fl = ((u64)((1 + last(hin) + 7) >> 3) << 32) | call;
                    __hip_atomic_store((GAS u64*)(a.ws[q] + OFF_F + (par * 8 + rank) * 2048 + (88 + id) * 8), fl, RLX_SYS);
                }
            }
            if (threadIdx.x == 0) L32(C_NEXT) = 25 + (ticket & 63);
            BAR();
            n = LPOLL(C_NEXT);
            if (wave == 0) STAMP(46 + min(round, 3u) * 3);
        }
    }
    if (threadIdx.x == 0) __hip_atomic_store((GAS u32*)(myws + OFF_CNT + bid * 128), call, RLX_AG);
}

extern "C" void kernel_launch(int rank, int nranks, ncclComm_t comm,
                              hipStream_t stream,
                              void* const* d_in, const int* in_sizes, int n_in,
                              void* d_out, int out_size,
                              const int* in_dtypes, int out_dtype,
                              void* const* d_out_all,
                              void* const* d_ws_all, size_t ws_bytes) {
    if (nranks != NR || (unsigned)rank >= (unsigned)NR || ws_bytes < WS_NEED || n_in != 4
        || in_sizes[0] != 4096 * KS || in_sizes[1] != KS * GN || out_size != MS * GN) {
        fprintf(stderr, "unexpected problem shape\n");
        return;
    }
    Args a;
    a.x = (const u8*)d_in[0];
    a.w = (const u8*)d_in[1];
    a.sx = (const float*)d_in[2];
    a.sw = (const float*)d_in[3];
    a.out = (float*)d_out;
    for (int p = 0; p < NR; ++p) a.ws[p] = (u8*)d_ws_all[p];
    a.rank = rank;
    a.pad = 0;
    fused_gemm_rs<<<NWG, NTHR, 0, stream>>>(a);
    HIP_CHECK(hipGetLastError());
}
```

```cpp
#include <hip/hip_runtime.h>
#include <rccl/rccl.h>
#include <cstdio>
#include <cstdint>
#include <type_traits>

#define HIP_CHECK(expr) do { hipError_t _e = (expr); if (_e != hipSuccess) { \
    fprintf(stderr, "HIP error %s at %s:%d\n", hipGetErrorString(_e), __FILE__, __LINE__); \
    return; } } while (0)

typedef unsigned u32;
typedef unsigned long long u64;
typedef unsigned char u8;
typedef u32 u32x2 __attribute__((ext_vector_type(2)));
typedef u32 u32x4 __attribute__((ext_vector_type(4)));
typedef int i32x8 __attribute__((ext_vector_type(8)));
typedef float f32x4 __attribute__((ext_vector_type(4)));
#define GAS __attribute__((address_space(1)))
#define LAS __attribute__((address_space(3)))

#define WINDOW 8
#define WINDOW2 14
#define BOOSTN 0
#define NPL 2
#define EARLY 1
#define WARM 0
#define XMAP 0
#define FORCE_RAW 0
#define GSLEEP 8
#define XWIN 4
#define STAMPS 0

constexpr int NR = 8;
constexpr int GN = 16384;
constexpr int KS = 512;
constexpr int MS = 512;
constexpr int NWG = 256;
constexpr int NTHR = 704;
constexpr u32 NRUN = 88 - 2 * NPL, NITEM = NRUN + 128;
constexpr u32 NJOB = 120;

constexpr u32 VARCAP = 5120;
constexpr u32 HALFB  = 2560;
constexpr u32 RUN    = VARCAP + 8 * HALFB;
constexpr u32 CAP2 = 1536, CAP3 = 512, SECCAP = 608;
static_assert(32 + 8 * SECCAP <= VARCAP - 128, "sections fit; the last line of the variable part is never used");
constexpr u32 EXCCAP = 3072;
constexpr u32 HEADB  = 1536;
constexpr u32 OHALF  = EXCCAP + 4 * HEADB + 8 * 2048;
constexpr u32 OCAP2 = 1024, OSECCAP = 752, HARDCAP = 15;
static_assert(16 + 4 * OSECCAP <= EXCCAP && OHALF == RUN, "sections fit");

constexpr u32 OFF_CNT = 0;
constexpr u32 OFF_TMO = 32u << 10;
constexpr u32 OFF_CT  = 68u << 10;
constexpr u32 OFF_PF  = 96u << 10;
constexpr u32 OFF_Q   = 56u << 10;
constexpr u32 OFF_XF  = 64u << 10;
constexpr u32 OFF_F   = 128u << 10;
constexpr u32 OFF_DBG = 256u << 10;
constexpr u32 PSTR    = 32 * 16384 + 4352;
constexpr u32 OFF_XO  = 1u << 20;
constexpr u32 OFF_SP  = 6u << 20;
constexpr u32 OFF_P   = 32u << 20;
constexpr u32 P_PAR   = 8u << 20;
constexpr u32 OFF_RAW = 48u << 20;
constexpr u32 RAW_PAR = 10u << 20;
constexpr u32 OFF_OP  = 68u << 20;
constexpr u32 OP_PAR  = 25u << 20;
constexpr u32 OFF_OR  = 118u << 20;
constexpr u32 OR_PAR  = 32u << 20;
constexpr u32 OFF_LW  = 182u << 20;
constexpr u32 WS_NEED = OFF_LW + (8u << 20);
static_assert(OFF_XO + 8 * PSTR <= OFF_SP && OFF_SP + 48 * PSTR <= OFF_P && 320 * RUN <= P_PAR
              && 320 * 32768 <= RAW_PAR && 1024 * OHALF <= OP_PAR, "workspace");
constexpr u64 DEADLINE_TICKS = 50000000ull;

constexpr u32 L_LUTD = 0;
constexpr u32 L_LUTC = 1024;
constexpr u32 L_BIG  = 3072;
constexpr u32 L_ZERO = L_BIG + 131072;
constexpr u32 V2S = CAP2 + 16, V3S = CAP3 + 16, EVS = 1040;
constexpr u32 WSCR   = V2S + V3S + EVS;
constexpr u32 L_SCR  = L_ZERO;
constexpr u32 L_CTL  = L_SCR + 8 * WSCR;
constexpr u32 C_WB = L_CTL, C_LANDED = L_CTL + 4, C_DEAD = L_CTL + 8, C_PUB = L_CTL + 12, C_DONE = L_CTL + 16;
constexpr u32 C_HDR = L_CTL + 32, C_RDY = L_CTL + 64, C_START = L_CTL + 96, C_NEXT = L_CTL + 100, C_PUB1 = L_CTL + 108;
constexpr u32 L_END  = L_CTL + 192;
static_assert(L_BIG + 4 * RUN <= L_ZERO && WSCR % 16 == 0 && L_END % 16 == 0 && L_END <= 160 * 1024, "LDS");
static_assert(CAP2 / 4 + CAP3 / 4 + CAP3 + 16 <= EVS && 8 * HARDCAP + OCAP2 / 4 + CAP3 / 4 + CAP3 + 16 <= EVS, "section scratch");

struct Args {
    const u8* x;
    const u8* w;
    const float* sx;
    const float* sw;
    float* out;
    u8* self;
    u8* ws[NR];
    int rank;
    int pad;
};

#define ST16(voff, data, sbase, imm) \
    asm volatile("global_store_dwordx4 %0, %1, %2 offset:" #imm " sc0 sc1\n\ts_nop 1" \
                 :: "v"(voff), "v"(data), "s"(sbase) : "memory")
#define ST4(voff, data, sbase) \
    asm volatile("global_store_dword %0, %1, %2 sc0 sc1" :: "v"(voff), "v"(data), "s"(sbase) : "memory")
#define DMA16(voff, ldsaddr, sbase) do { u32 keep_; \
    asm volatile("s_mov_b32 %0, m0\n\ts_mov_b32 m0, %2\n\ts_nop 0\n\t" \
                 "global_load_lds_dwordx4 %1, %3 sc0 sc1\n\ts_mov_b32 m0, %0" \
                 : "=&s"(keep_) : "v"(voff), "s"(ldsaddr), "s"(sbase) : "memory"); } while (0)
#define WAITV(n) asm volatile("s_waitcnt vmcnt(%0)" :: "n"(n) : "memory")
#define WAITL() asm volatile("s_waitcnt lgkmcnt(0)" ::: "memory")
#define BAR() asm volatile("s_waitcnt lgkmcnt(0)\n\ts_barrier" ::: "memory")
#define RLX_WG __ATOMIC_RELAXED, __HIP_MEMORY_SCOPE_WORKGROUP
#define RLX_AG __ATOMIC_RELAXED, __HIP_MEMORY_SCOPE_AGENT
#define RLX_SYS __ATOMIC_RELAXED, __HIP_MEMORY_SCOPE_SYSTEM
#define CFENCE() asm volatile("" ::: "memory")

__device__ __forceinline__ u64 rt() { return __builtin_amdgcn_s_memrealtime(); }
__device__ __forceinline__ u32 uni(u32 v) { return __builtin_amdgcn_readfirstlane(v); }
__device__ __forceinline__ u32 perm(u32 hi, u32 lo, u32 sel) { return __builtin_amdgcn_perm(hi, lo, sel); }
__device__ __forceinline__ u32 scan64(u32 v) {
    v += __builtin_amdgcn_update_dpp(0u, v, 0x111, 0xf, 0xf, true);
    v += __builtin_amdgcn_update_dpp(0u, v, 0x112, 0xf, 0xf, true);
    v += __builtin_amdgcn_update_dpp(0u, v, 0x114, 0xf, 0xf, true);
    v += __builtin_amdgcn_update_dpp(0u, v, 0x118, 0xf, 0xf, true);
    v += __builtin_amdgcn_update_dpp(0u, v, 0x142, 0xa, 0xf, false);
    v += __builtin_amdgcn_update_dpp(0u, v, 0x143, 0xc, 0xf, false);
    return v;
}
__device__ __forceinline__ u32 last(u32 v) { return __builtin_amdgcn_readlane(v, 63); }
__device__ __forceinline__ u32 escbits(u32 q) { return ~(q | (q >> 1)) & 0x55555555u; }
__device__ __forceinline__ u32 lookup8(u32 sel, u32 thi, u32 tlo) {
    const u32 r = perm(thi, tlo, sel);
    return __builtin_amdgcn_udot4((r ^ (r >> 2)) & 0x18181818u, 0x40100401u, 0u, false);
}
__device__ __forceinline__ u32 classify8(u32 e4, u32 kadd, u32 thi, u32 tlo) {
    return lookup8(((e4 >> 2) + kadd) & 0x1F1F1F1Fu, thi, tlo);
}
constexpr u32 T13 = 0x00181008u, T2L = 0x00000818u, T2H = 0x00001000u;
__device__ __forceinline__ u32 rep(u32 b) { return b * 0x01010101u; }
__device__ __forceinline__ float silu(float acc, float sc) {
    const float y = acc * sc;
    const float e = __expf(-__builtin_amdgcn_fmed3f(y, -60.f, 60.f));
    return y * __builtin_amdgcn_rcpf(1.f + e);
}
__device__ __forceinline__ int cvt_i32(float f) { int r; asm("v_cvt_i32_f32 %0, %1" : "=v"(r) : "v"(f)); return r; }

__global__ void __launch_bounds__(NTHR) fused_gemm_rs(Args a) {
    __shared__ __attribute__((aligned(16))) u8 lds_raw[L_END];
    LAS u8* const L = (LAS u8*)lds_raw;
#define L8(off) (*(LAS u8*)(L + (off)))
#define L16(off) (*(LAS unsigned short*)(L + (off)))
#define L32(off) (*(LAS u32*)(L + (off)))
#define L64(off) (*(LAS u32x2*)(L + (off)))
#define L128(off) (*(LAS u32x4*)(L + (off)))
#define LPOLL(off) uni(__hip_atomic_load((LAS u32*)(L + (off)), RLX_WG))
    const u32 bid = blockIdx.x;
    u32 lane = threadIdx.x & 63;
#define FRESH_LANE() do { asm volatile("" : "+v"(lane)); __builtin_assume(lane < 64); } while (0)
    const u32 wave = uni(threadIdx.x >> 6);
    const u32 rank = a.rank;
    const bool is_link = (bid >> 3) < 7;
    u8* const myws = a.self;
    const u32 lbase = (u32)(size_t)L;
    const u64 t_begin = rt();
#define STAMP(i)

    const u32 tile = (((bid & ~15u) | ((bid & 7) << 1) | ((bid >> 3) & 1)) + 32 * (rank + 1)) & 255;
    const u32 xrun = 256 + ((bid >> 1) & 7) * 8 + (7 - (bid >> 5)) * 4 + ((bid >> 4) & 1) * 2 + (bid & 1);
    u32 f[4][4];
    auto fetch_run = [&](u32 id) {
        if (id >= 256) {
            const __amdgpu_buffer_rsrc_t rx = __builtin_amdgcn_make_buffer_rsrc((void*)a.x, 0, 4096 * KS, 0x00020000);
            const u32 vo = (lane & 15) * KS + (lane >> 4) * 32, so = (id - 256) * 64 * KS + (wave >> 1) * 128 + (wave & 1) * 16;
#pragma unroll
            for (int c = 0; c < 4; ++c) {
                const u32x4 v = __builtin_amdgcn_raw_buffer_load_b128(rx, vo, so + c * 16 * KS, 0);
                f[c][0] = v[0]; f[c][1] = v[1]; f[c][2] = v[2]; f[c][3] = v[3];
            }
        } else {
            const __amdgpu_buffer_rsrc_t rw = __builtin_amdgcn_make_buffer_rsrc((void*)a.w, 0, KS * GN, 0x00020000);
            const u32 vo = (lane >> 4) * 32 * GN + (lane & 15) * 4, so = ((wave >> 1) * 128 + (wave & 1) * 16) * GN + id * 64;
#pragma unroll
            for (int r = 0; r < 16; ++r) f[r >> 2][r & 3] = __builtin_amdgcn_raw_buffer_load_b32(rw, vo, so + r * GN, 0);
        }
        __builtin_amdgcn_sched_barrier(0);
    };
    if (wave < 8) fetch_run(bid >= 224 ? xrun : tile);

    const u32 cnt_v = __hip_atomic_load((GAS u32*)(myws + OFF_CNT + bid * 128), RLX_AG);
#define GET_CALL const u32 call = uni(cnt_v) + 1, par = call & 1
    u32 spins = 0;
    auto expired = [&](u32 call) -> bool {
        if (++spins & 15) return false;
        return uni(__hip_atomic_load((GAS u32*)(myws + OFF_TMO), RLX_AG)) == call || rt() - t_begin > DEADLINE_TICKS;
    };
    auto give_up = [&](u32 call) {
        if (lane == 0) {
            __hip_atomic_store((GAS u32*)(myws + OFF_TMO), call, RLX_AG);
            __hip_atomic_store((LAS u32*)(L + C_DEAD), 1u, RLX_WG);
        }
    };
    auto lds_init = [&]() {
        for (u32 o = L_ZERO + threadIdx.x * 16; o < L_END; o += 576 * 16) L128(o) = u32x4{0, 0, 0, 0};
        if (threadIdx.x < 256) {
            const u32 x = threadIdx.x;
            u32 d = 0, c = 0x0C0C0C0Cu, n = 0;
#pragma unroll
            for (int b = 0; b < 4; ++b) {
                const u32 code = (x >> (2 * b)) & 3;
                d |= (code ? 4 + code : n) << (8 * b);
                if (!code) c = (c & ~(0xFFu << (8 * n))) | (b << (8 * n));
                n += !code;
            }
            L32(L_LUTD + x * 4) = d;
            L64(L_LUTC + x * 8) = u32x2{c, n};
        }
    };
    const u32 li = bid & 7;
    auto run_id = [&](u32 n) -> u32 {
        if (n >= 80) return 256 + rank * 8 + 2 * NPL + n - 80;
        if (n >= 12 && n < 44) return rank * 32 + n - 12;
        const u32 j = n < 12 ? n : n - 32, l = j / 12, r = j - l * 12, qi = (r >> 1) + ((r >> 1) >= (bid >> 3));
        return 256 + ((rank + 1 + qi) & 7) * 8 + 2 * l + (r & 1);
    };

    if (wave == 9) {
        const u32 q = (rank + 1 + (XMAP ? bid >> 3 : bid)) & 7, t = (XMAP ? bid : bid >> 3) & 3, rg = XMAP ? (bid >> 6) * 2 + ((bid >> 2) & 1) : bid >> 5;
        if (q != rank && rg >= 2 * NPL) {
            asm volatile("s_barrier" ::: "memory");
            return;
        }
        const __amdgpu_buffer_rsrc_t rx = __builtin_amdgcn_make_buffer_rsrc((void*)a.x, 0, 4096 * KS, 0x00020000);
        u32x4 v[4][2];
#pragma unroll
        for (int rb = 0; rb < 4; ++rb)
#pragma unroll
            for (int p = 0; p < 2; ++p)
                v[rb][p] = __builtin_amdgcn_raw_buffer_load_b128(rx, (lane & 15) * KS + (lane >> 4) * 32 + p * 16, (q * MS + rg * 64 + rb * 16) * KS + t * 128, 0);
        u8* dst = a.ws[q];
        asm volatile("s_barrier" ::: "memory");
        GET_CALL;
        asm volatile("" :: "v"(v[0][0]), "v"(v[0][1]), "v"(v[1][0]), "v"(v[1][1]),
                          "v"(v[2][0]), "v"(v[2][1]), "v"(v[3][0]), "v"(v[3][1]));
        const u32 vo = OFF_XO + (par * 4 + (rg >> 1)) * PSTR + (rank * 4 + t) * 16384 + (rg & 1) * 8192 + lane * 16 + 4096;
        STAMP(4);
#define XW() WAITV(XWIN - 1)
        ST16(vo, v[0][0], dst, -4096); XW(); ST16(vo, v[0][1], dst, -3072); XW();
        ST16(vo, v[1][0], dst, -2048); XW(); ST16(vo, v[1][1], dst, -1024); XW();
        ST16(vo, v[2][0], dst, 0);     XW(); ST16(vo, v[2][1], dst, 1024);  XW();
        ST16(vo, v[3][0], dst, 2048);  XW(); ST16(vo, v[3][1], dst, 3072);
        WAITV(0);
        if (lane == 0) __hip_atomic_store((GAS u32*)(dst + OFF_XF + par * 1024 + rg * 128 + (rank * 4 + t) * 4), call, RLX_SYS);
        STAMP(1);
        return;
    }
    if (wave == 10) {
        asm volatile("s_barrier" ::: "memory");
        GET_CALL;
        if (WARM && lane == 0) {
            u32 sink = 0;
            __builtin_amdgcn_s_sleep(WARM);
#define TOUCH(off, base) asm volatile("global_load_dword %0, %1, %2 sc0 sc1" : "+v"(sink) : "v"(off), "s"(base) : "memory")
            if (is_link) TOUCH(OFF_P + par * P_PAR + run_id(li) * RUN + VARCAP - 64, a.ws[(rank + 1 + (bid >> 3)) & 7]);
            asm volatile("s_waitcnt vmcnt(0)" : "+v"(sink) :: "memory");
        }
        if (!is_link) return;
        const u32 s = (rank + 1 + (bid >> 3)) & 7;
        bool part_ok = false;
        for (u32 k = 0;; ++k) {
            u32 n = li + 8 * k;
            if (k > 1) {
                while (LPOLL(C_START) < k) __builtin_amdgcn_s_sleep(2);
                u32 tk = 0;
                if (lane == 0) tk = __hip_atomic_fetch_add((GAS u32*)(myws + OFF_Q + (8 + (bid >> 3)) * 128), 1u, RLX_AG);
                n = 16 + uni(tk) - (NITEM - 8) * (call - 1);
            }
            u32 lines = 0;
            if (n < NITEM) {
                while (n >= NRUN && !part_ok) {
                    part_ok = uni(__hip_atomic_load((GAS u32*)(myws + OFF_PF + (48 + s) * 128), RLX_SYS)) >= call;
                    if (!part_ok && expired(call)) break;
                }
                for (;;) {
                    const u64 f = __hip_atomic_load((GAS u64*)(myws + OFF_F + (par * 8 + s) * 2048 + n * 8), RLX_SYS);
                    lines = uni((u32)(f >> 32)) & 0xFF;
                    if (uni((u32)f) >= call && (n < NRUN || part_ok)) break;
                    if (expired(call)) { give_up(call); n = 255; break; }
                    __builtin_amdgcn_s_sleep(2);
                }
            } else {
                n = 255;
                if (uni(__hip_atomic_load((GAS u32*)(myws + OFF_TMO), RLX_AG)) == call) give_up(call);
            }
            CFENCE();
            lines = min(max(lines, 1u), (n < NRUN ? VARCAP : EXCCAP) / 128);
            if (lane == 0) __hip_atomic_store((LAS u32*)(L + C_RDY + (k & 7) * 4), ((k + 1) << 16) | (n << 8) | lines, RLX_WG);
            if (n == 255) return;
        }
    }
    if (wave == 8) {
        lds_init();
        GET_CALL;
        BAR();
        if (!is_link) return;
        const u8* base = a.ws[(rank + 1 + (bid >> 3)) & 7];
        int issued = 0, told = 0;
        auto publish = [&](int v) {
            if (v > told) { told = v; if (lane == 0) __hip_atomic_store((LAS u32*)(L + C_LANDED), (u32)v, RLX_WG); }
        };
        auto drain = [&]() { WAITV(0); publish(issued); };
        auto pull = [&](u32 goff, u32 loff, u32 bytes, bool boost) {
            if (lane * 16 < bytes) DMA16(goff + lane * 16, lbase + loff, base);
            ++issued;
            if (boost) { WAITV(WINDOW2 - 1); publish(issued - (WINDOW2 - 1)); }
            else { WAITV(WINDOW - 1); publish(issued - (WINDOW - 1)); }
        };
        bool early = false;
        for (;;) {
            early = uni(__hip_atomic_load((GAS u32*)(myws + OFF_F + (par * 8 + ((rank + 1 + (bid >> 3)) & 7)) * 2048 + (EARLY ? 240 + li : li) * 8), RLX_SYS)) >= call;
            if (early) break;
            if (expired(call)) { give_up(call); break; }
        }
        CFENCE();
        STAMP(2);
        if (!early) issued = 8 * HALFB / 1024;
        else
            for (u32 o = VARCAP; o < RUN; o += 1024) pull(OFF_P + par * P_PAR + run_id(li) * RUN + o, L_BIG + o, 1024, false);
        for (u32 m = 0;; ++m) {
            u32 rdy = LPOLL(C_RDY + (m & 7) * 4);
            const u32 dn = C_DONE + (m & 3) * 4, need = 8 * (m >> 2);
            if ((rdy >> 16) != m + 1 || LPOLL(dn) < need) {
                drain();
                while (((rdy = LPOLL(C_RDY + (m & 7) * 4)) >> 16) != m + 1 || LPOLL(dn) < need) __builtin_amdgcn_s_sleep(1);
            }
            CFENCE();
            const u32 n = (rdy >> 8) & 0xFF;
            if (n == 255) break;
            if (lane == 0) __hip_atomic_store((LAS u32*)(L + C_START), m + 1, RLX_WG);
            const bool is_out = n >= NRUN, boost = n >= NITEM - BOOSTN;
            const u32 g0 = is_out ? OFF_OP + par * OP_PAR + (rank * 128 + (n - NRUN)) * OHALF : OFF_P + par * P_PAR + run_id(n) * RUN;
            const u32 cap = is_out ? EXCCAP : VARCAP;
            const u32 l0 = L_BIG + (m & 3) * RUN, vb = (rdy & 0xFF) * 128;
            for (u32 o = 0; o < vb; o += 1024) pull(g0 + o, l0 + o, vb - o, boost);
            if (m) for (u32 o = cap; o < RUN; o += 1024) pull(g0 + o, l0 + o, 1024, boost);
        }
        WAITV(12); publish(issued - 12); WAITV(9); publish(issued - 9); WAITV(6); publish(issued - 6);
        WAITV(3); publish(issued - 3); WAITV(0); publish(issued);
        STAMP(21);
        return;
    }

    u32 phase = 0;
    auto wbar = [&]() {
        phase += 8;
        WAITL();
        if (lane == 0) __hip_atomic_fetch_add((LAS u32*)(L + C_WB), 1u, RLX_WG);
        while (LPOLL(C_WB) < phase) __builtin_amdgcn_s_sleep(1);
        CFENCE();
    };
    auto append = [&](auto G, u32 pos, const u32* e4, const u32x2* lut) {
        u32 carry = 0;
#pragma unroll
        for (int i = 0; i < G.value; ++i) {
            const u32 comp = perm(0u, e4[i], lut[i][0]), sh = pos << 3;
            __hip_atomic_fetch_or((LAS u32*)(L + (pos & ~3u)), (comp << (sh & 31)) | carry, RLX_WG);
            carry = __builtin_amdgcn_ubfe(comp, 0u - sh, sh);
            pos += lut[i][1];
        }
        __hip_atomic_fetch_or((LAS u32*)(L + (pos & ~3u)), carry, RLX_WG);
    };
    auto expand = [&](u32 tab, u32 pos, u32 x) -> u32 {
        const u32 sel = L32(L_LUTD + x * 4);
        const LAS u32* p = (const LAS u32*)(L + (pos & ~3u));
        return perm(tab, __builtin_amdgcn_alignbyte(p[1], p[0], pos & 3), sel);
    };
    auto panel_of = [&](u32 id) -> u32 { return id >= 256 ? (id - 256) >> 1 : 32 + ((id & 31) >> 1); };
    auto stage_off = [&](u32 id, u32 s, u32 par) -> u32 {
        return ((id - 256) >> 3 == rank ? OFF_XO + (par * 4 + ((id >> 1) & 3)) * PSTR : OFF_SP + panel_of(id) * PSTR) + (s * 4 + (wave >> 1)) * 16384 + (id & 1) * 8192 + (wave & 1) * 1024 + lane * 16 + 4096;
    };
    auto arrive = [&](u32 c) -> u32 {
        u32 t = 0;
        if (lane == 0) t = __hip_atomic_fetch_add((GAS u32*)(myws + OFF_CT + c * 128), 1u, RLX_AG);
        return t;
    };
    auto raise = [&](u32 c, u32 t, u32 call) {
        const u32 per_call = c < 32 ? 112 : c < 48 ? 128 : c < 64 ? 64 : 16;
        if (lane == 0 && t == per_call * call - 1) __hip_atomic_store((GAS u32*)(myws + OFF_PF + c * 128), call, RLX_AG);
    };
    const u32 v2 = L_SCR + wave * WSCR, v3 = v2 + V2S, ev = v3 + V3S;
    auto pack_levels = [&](auto NP, u32 N2, u32 e, auto cls2, auto cls3) -> u32 {
        u32 u[2 * NP.value], m = 0;
        u32x2 l2[2 * NP.value];
#pragma unroll
        for (int i = 0; i < NP.value; ++i) {
            const u32x2 v = L64(v2 + lane * (8 * NP.value) + i * 8);
            u[2 * i] = v[0]; u[2 * i + 1] = v[1];
        }
#pragma unroll
        for (int i = 0; i < NP.value; ++i) {
            u32 q8 = cls2(u[2 * i]) | (cls2(u[2 * i + 1]) << 8);
            const int valid = (int)N2 - (int)lane * (8 * NP.value) - i * 8;
            q8 |= (0x5555u * 8) << (2 * min(max(valid, 0), 8));
            if (valid > 0) L16(e + lane * (2 * NP.value) + i * 2) = (unsigned short)(q8 >> 3);
            l2[2 * i] = L64(L_LUTC + (q8 & 0x7F8));
            l2[2 * i + 1] = L64(L_LUTC + ((q8 >> 8) & 0x7F8));
        }
#pragma unroll
        for (int i = 0; i < 2 * NP.value; ++i) m += l2[i][1];
        const u32 in2 = scan64(m);
        const u32 N3 = last(in2);
        if (N3 > CAP3) return ~0u;
        if (m) append(std::integral_constant<int, 2 * NP.value>{}, v3 + in2 - m, u, l2);
        const u32 off3 = ((N2 + 7) >> 3) * 2, off4 = off3 + ((N3 + 7) >> 3) * 2;
        const u32x2 v = L64(v3 + lane * 8);
        const u32 w[2] = {v[0], v[1]};
        u32 q8 = cls3(w[0]) | (cls3(w[1]) << 8);
        const int valid = (int)N3 - (int)lane * 8;
        q8 |= (0x5555u * 8) << (2 * min(max(valid, 0), 8));
        if (valid > 0) L16(e + off3 + lane * 2) = (unsigned short)(q8 >> 3);
        const u32x2 l3[2] = {L64(L_LUTC + (q8 & 0x7F8)), L64(L_LUTC + ((q8 >> 8) & 0x7F8))};
        const u32 m3 = l3[0][1] + l3[1][1];
        const u32 in3 = scan64(m3);
        if (m3) append(std::integral_constant<int, 2>{}, e + off4 + in3 - m3, w, l3);
        return off4 + last(in3);
    };
    auto unpack_levels = [&](auto NP, u32 N2, u32 sec, u32 t2, u32 t3) {
        u32 q[NP.value], e[NP.value], n = 0;
#pragma unroll
        for (int i = 0; i < NP.value; ++i) {
            q[i] = L16(sec + lane * (2 * NP.value) + i * 2) | 0x55550000u;
            if (lane * (8 * NP.value) + i * 8 >= N2) q[i] = 0x55555555u;
            e[i] = escbits(q[i]);
            n += __builtin_popcount(e[i]);
        }
        const u32 in2 = scan64(n), N3 = last(in2);
        const u32 off3 = sec + ((N2 + 7) >> 3) * 2, off4 = off3 + ((N3 + 7) >> 3) * 2;
        u32 q3 = L16(off3 + lane * 2) | 0x55550000u;
        if (lane * 8 >= N3) q3 = 0x55555555u;
        const u32 e3 = escbits(q3), m3 = __builtin_popcount(e3);
        const u32 p3 = off4 + scan64(m3) - m3;
        L64(v3 + lane * 8) = u32x2{expand(t3, p3, q3 & 0xFF), expand(t3, p3 + __builtin_popcount(e3 & 0xFF), (q3 >> 8) & 0xFF)};
        u32 pp = v3 + in2 - n;
        u32x2 z[NP.value];
#pragma unroll
        for (int i = 0; i < NP.value; ++i) {
            z[i] = u32x2{expand(t2, pp, q[i] & 0xFF), expand(t2, pp + __builtin_popcount(e[i] & 0xFF), (q[i] >> 8) & 0xFF)};
            pp += __builtin_popcount(e[i]);
        }
#pragma unroll
        for (int i = 0; i < NP.value; ++i) L64(v2 + lane * (8 * NP.value) + i * 8) = z[i];
    };
    u8* peer = a.ws[lane & 7];
    u32 npub = 0;

    lds_init();
    BAR();
    for (u32 job = 0; job < (bid >= 192 ? 2u : 1u); ++job) {
        const u32 id = (bid >= 192 && job == (bid >= 224 ? 0u : 1u)) ? xrun : tile;
        const u32 t = wave >> 1, h = wave & 1;
        if (id >= 256 && ((id - 256) >> 3) == rank) continue;
        if (job) {
            fetch_run(id);
            for (u32 o = lane * 16; o < WSCR; o += 1024) L128(v2 + o) = u32x4{0, 0, 0, 0};
        }
        if (id < 256) {
            u32 g[4][4];
#pragma unroll
            for (int i = 0; i < 4; ++i) {
                const u32 t0 = perm(f[i][1], f[i][0], 0x05010400u), t1 = perm(f[i][1], f[i][0], 0x07030602u);
                const u32 t2 = perm(f[i][3], f[i][2], 0x05010400u), t3 = perm(f[i][3], f[i][2], 0x07030602u);
                g[0][i] = perm(t2, t0, 0x05040100u); g[1][i] = perm(t2, t0, 0x07060302u);
                g[2][i] = perm(t3, t1, 0x05040100u); g[3][i] = perm(t3, t1, 0x07060302u);
            }
#pragma unroll
            for (int i = 0; i < 16; ++i) f[i >> 2][i & 3] = g[i >> 2][i & 3];
        }
        GET_CALL;
        asm volatile("" : "+v"(peer));
        const u32x4 f0 = {f[0][0], f[0][1], f[0][2], f[0][3]}, f1 = {f[1][0], f[1][1], f[1][2], f[1][3]};
        const u32x4 f2 = {f[2][0], f[2][1], f[2][2], f[2][3]}, f3 = {f[3][0], f[3][1], f[3][2], f[3][3]};
        const bool slab = id < 256 && (id >> 5) == rank;
        auto keep = [&]() {
            const bool pan = id >= 256 || slab;
            const u32 vo = pan ? stage_off(id, rank, par) : OFF_LW + (id >> 1) * 65536 + t * 16384 + (id & 1) * 8192 + h * 1024 + lane * 16 + 4096;
            const u32 c = pan ? panel_of(id) : 64 + (id >> 1);
            ST16(vo, f0, myws, -4096); ST16(vo, f1, myws, -2048); ST16(vo, f2, myws, 0); ST16(vo, f3, myws, 2048);
            WAITV(0);
            raise(c, arrive(c), call);
        };
        if (slab) {
            keep();
            continue;
        }
        u32 esum = 0;
#pragma unroll
        for (int i = 0; i < 16; ++i) esum = __builtin_amdgcn_udot4(f[i >> 2][i & 3] & 0x7C7C7C7Cu, 0x01010101u, esum, false);
        const u32 e0 = ((last(scan64(esum)) + 14746u) >> 14) & 31;
        const u32 runo = OFF_P + par * P_PAR + id * RUN;
        u32x4 Q, Mq;
        u32 S[2] = {0, 0};
        u32 e4[16];
        u32x2 lut[16];
        const u32 k1 = rep((33 - e0) & 31);
#pragma unroll
        for (int c = 0; c < 4; ++c) {
            u32 q = 0, m = 0;
#pragma unroll
            for (int i = 0; i < 4; ++i) {
                e4[4 * c + i] = f[c][i] & 0x7C7C7C7Cu;
                const u32 x8 = classify8(e4[4 * c + i], k1, 0u, T13);
                lut[4 * c + i] = L64(L_LUTC + x8);
                q |= i ? x8 << (8 * i - 3) : x8 >> 3;
                m |= (f[c][i] & 0x03030303u) << (2 * i);
                S[c >> 1] |= (f[c][i] & 0x80808080u) >> (4 * (c & 1) + i);
            }
            Q[c] = q; Mq[c] = m;
        }
        {
            const u32 vo = runo + VARCAP + wave * HALFB + lane * 16;
            ST16(vo, Q, myws, 0);
            ST16(vo, Mq, myws, 1024);
            const u32x4 s4 = {S[0], S[1], (u32)__shfl_xor((int)S[0], 32), (u32)__shfl_xor((int)S[1], 32)};
            if (lane < 32) ST16(vo, s4, myws, 2048);
        }
        u32 slot;
        bool to;
        if (id >= 256) {
            const u32 q = (id - 256) >> 3, qi = (q - lane - 1) & 7, si = (rank - lane - 1) & 7;
            const u32 j = ((id >> 1) & 3) * 12 + (qi - (qi > si)) * 2 + (id & 1);
            slot = lane == q ? 80 + (id & 7) - 2 * NPL : j < 12 ? j : j + 32;
            to = lane < 8 && lane != rank && (lane != q || (id & 7) >= 2 * NPL);
        } else {
            slot = 12 + (id & 31);
            to = lane == (id >> 5);
        }
        if (EARLY && id >= 256 && (id & 6) == 0) {
            WAITV(0);
            u32 arrived = 0;
            if (lane == 0) arrived = __hip_atomic_fetch_add((LAS u32*)(L + C_PUB1), 1u, RLX_WG);
            if (uni(arrived) == 7 && to && slot < 8)
                __hip_atomic_store((GAS u32*)(peer + OFF_F + (par * 8 + rank) * 2048 + (240 + slot) * 8), call, RLX_SYS);
            STAMP(3);
        }
        u32 n = 0;
#pragma unroll
        for (int c = 0; c < 4; ++c) n += __builtin_popcount(escbits(Q[c]));
        const u32 inc = scan64(n);
        const u32 N2 = last(inc);
        u32 len = 0;
        bool raw = N2 > CAP2 || FORCE_RAW & 1;
        if (!raw) {
            append(std::integral_constant<int, 16>{}, v2 + inc - n, e4, lut);
            const u32 k2 = rep((35 - e0) & 31), k3 = rep((38 - e0) & 31);
            len = pack_levels(std::integral_constant<int, 3>{}, N2, ev, [&](u32 u) { return classify8(u, k2, T2H, T2L); },
                              [&](u32 u) { return classify8(u, k3, 0u, T13); });
            raw = len > SECCAP;
        }
        if (raw) {
            len = 0;
            const u32 vo = OFF_RAW + par * RAW_PAR + (id * 8 + wave) * 4096 + lane * 16;
            ST16(vo, f0, myws, 0); ST16(vo, f1, myws, 1024); ST16(vo, f2, myws, 2048); ST16(vo, f3, myws, 3072);
        }
        const u32 len16 = (len + 15) >> 4;
        if (lane == 0) L32(C_HDR + wave * 4) = len16 | (raw ? 0x80000000u : (N2 << 9) | (e0 << 20));
        wbar();
        const u32 hv = lane < 8 ? L32(C_HDR + lane * 4) : 0u;
        const u32 hin = scan64(hv & 0x1FF);
        const u32 hoff = 2 + hin - (hv & 0x1FF);
        if (lane < len16) {
            const u32x4 sec = L128(ev + lane * 16);
            ST16(runo + __builtin_amdgcn_readlane(hoff, wave) * 16 + lane * 16, sec, myws, 0);
        }
        if (wave == 0 && lane < 8) ST4(runo + lane * 4, (hv & ~0x1FFu) | hoff, myws);
        WAITV(0);
        u32 arrived = 0;
        npub += 8;
        if (lane == 0) arrived = __hip_atomic_fetch_add((LAS u32*)(L + C_PUB), 1u, RLX_WG);
        if (uni(arrived) == npub - 1) {
            const u64 fl = ((u64)((2 + last(hin) + 7) >> 3) << 32) | call;
            if (to) __hip_atomic_store((GAS u64*)(peer + OFF_F + (par * 8 + rank) * 2048 + slot * 8), fl, RLX_SYS);
            STAMP(22 + job);
        }
        keep();
        if (bid >= 192 && job == 0) wbar();
    }

    GET_CALL;
    const float sc = a.sx[0] * a.sw[0];
    const u32 rg = wave >> 1, ch = wave & 1;
    auto out_ptr = [&](u32 row0, u32 col0) -> float* {
        return a.out + (size_t)(row0 + rg * 32 + (lane >> 4) * 4) * GN + col0 + ch * 64 + (lane & 15) * 4;
    };
    auto out_store = [&](float* o, int g, f32x4 v) {
        asm volatile("global_store_dwordx4 %0, %1, off sc1\n\ts_nop 1" :: "v"(o + (size_t)((g >> 2) * 16 + (g & 3)) * GN), "v"(v) : "memory");
    };

    if (is_link) {
        const u32 s = (rank + 1 + (bid >> 3)) & 7;
        const u8* base = a.ws[s];
        auto landed = [&](u32 mark) {
            while (LPOLL(C_LANDED) < mark) __builtin_amdgcn_s_sleep(1);
            CFENCE();
        };
        auto done = [&](u32 m) {
            WAITL();
            if (lane == 0) __hip_atomic_fetch_add((LAS u32*)(L + C_DONE + (m & 3) * 4), 1u, RLX_WG);
        };
        u32 m = 0, n, pbase = 8 * HALFB / 1024;
        u32 owed = 255;
        auto settle = [&]() {
            if (owed != 255) { WAITV(0); raise(owed, arrive(owed), call); owed = 255; }
        };
        auto next = [&]() {
            u32 rdy = LPOLL(C_RDY + (m & 7) * 4);
            if ((rdy >> 16) != m + 1) {
                settle();
                while (((rdy = LPOLL(C_RDY + (m & 7) * 4)) >> 16) != m + 1) __builtin_amdgcn_s_sleep(1);
            }
            n = (rdy >> 8) & 0xFF;
            if (n == 255) return;
            pbase += ((rdy & 0xFF) + 7) >> 3;
            if (LPOLL(C_LANDED) < pbase) settle();
            landed(pbase);
        };
        for (;; ++m) {
            next();
            if (n >= NRUN) break;
            const u32 id = run_id(n), vb = L_BIG + (m & 3) * RUN;
            u32 tick = 0, tickc = 255;
            const u32 hdr = uni(L32(vb + wave * 4));
            u32x4 r[4];
            if (hdr >> 31) {
                const u32 vo = OFF_RAW + par * RAW_PAR + (id * 8 + wave) * 4096 + lane * 16;
                asm volatile("global_load_dwordx4 %0, %4, %5 sc0 sc1\n\tglobal_load_dwordx4 %1, %4, %5 offset:1024 sc0 sc1\n\t"
                             "global_load_dwordx4 %2, %4, %5 offset:2048 sc0 sc1\n\tglobal_load_dwordx4 %3, %4, %5 offset:3072 sc0 sc1\n\t"
                             "s_waitcnt vmcnt(0)"
                             : "=&v"(r[0]), "=&v"(r[1]), "=&v"(r[2]), "=&v"(r[3]) : "v"(vo), "s"(base) : "memory");
            } else {
                const u32 e0 = (hdr >> 20) & 31, N2 = (hdr >> 9) & 0x7FF, sec = vb + (hdr & 0x1FF) * 16;
                const u32 t1 = (((e0 - 1) & 31) << 10) | (e0 << 18) | (((e0 + 1) & 31) << 26);
                const u32 t2 = (((e0 - 2) & 31) << 10) | (((e0 + 2) & 31) << 18) | (((e0 - 3) & 31) << 26);
                const u32 t3 = (((e0 - 6) & 31) << 10) | (((e0 - 5) & 31) << 18) | (((e0 - 4) & 31) << 26);
                unpack_levels(std::integral_constant<int, 3>{}, N2, sec, t2, t3);
                const u32 pneed = m ? pbase + (((wave + 1) * HALFB + 1023) >> 10) : 0;
                if (LPOLL(C_LANDED) < pneed) settle();
                if (owed != 255) { WAITV(0); tick = arrive(owed); tickc = owed; owed = 255; }
                landed(pneed);
                const u32 fx = vb + VARCAP + wave * HALFB;
                const u32x4 Q = L128(fx + lane * 16), Mq = L128(fx + 1024 + lane * 16);
                const u32x2 S = L64(fx + 2048 + (lane & 31) * 16 + (lane >> 5) * 8);
                u32 E[4], n = 0;
#pragma unroll
                for (int c = 0; c < 4; ++c) { E[c] = escbits(Q[c]); n += __builtin_popcount(E[c]); }
                u32 pp = v2 + scan64(n) - n;
#pragma unroll
                for (int c = 0; c < 4; ++c)
#pragma unroll
                    for (int i = 0; i < 4; ++i) {
                        r[c][i] = expand(t1, pp, (Q[c] >> (8 * i)) & 0xFF) | ((Mq[c] >> (2 * i)) & 0x03030303u)
                                  | ((S[c >> 1] << (4 * (c & 1) + i)) & 0x80808080u);
                        pp += __builtin_popcount(E[c] & (0xFFu << (8 * i)));
                    }
            }
            done(m);
            settle();
            if (tickc != 255) raise(tickc, tick, call);
            const u32 vo = stage_off(id, s, par);
            ST16(vo, r[0], myws, -4096); ST16(vo, r[1], myws, -2048); ST16(vo, r[2], myws, 0); ST16(vo, r[3], myws, 2048);
            owed = panel_of(id);
            if (m) pbase += 8 * HALFB / 1024;
        }
        settle();
        const __amdgpu_buffer_rsrc_t ro = __builtin_amdgcn_make_buffer_rsrc(a.out, 0, MS * GN * 4, 0x00020000);
        const u32 u = wave >> 1, rb = wave & 1;
        while (n != 255) {
            const u32 id = (n - NRUN), vb = L_BIG + (m & 3) * RUN;
            const u32 e0 = ((id >> 5) * 128 + (id & 1) * 64 + (u >> 1) * 32 + rb * 16) * GN + s * 2048 + ((id >> 1) & 15) * 128 + (u & 1) * 64;
            float* const o = a.out + e0 + (lane >> 4) * 4 * GN + (lane & 15) * 4;
            f32x4 P[4];
#pragma unroll
            for (int k = 0; k < 4; ++k)
                P[k] = __builtin_bit_cast(f32x4, __builtin_amdgcn_raw_buffer_load_b128(ro, ((lane >> 4) * 4 * GN + (lane & 15) * 4) * 4, (e0 + k * GN) * 4, 17));
            const u32 hdr = uni(L32(vb + u * 4));
            if (hdr >> 31) {
                done(m);
                const u32 vo = OFF_OR + par * OR_PAR + ((rank * 128 + id) * 8 + wave) * 4096 + lane * 16;
                f32x4 r[4];
                asm volatile("global_load_dwordx4 %0, %4, %5 sc0 sc1\n\tglobal_load_dwordx4 %1, %4, %5 offset:1024 sc0 sc1\n\t"
                             "global_load_dwordx4 %2, %4, %5 offset:2048 sc0 sc1\n\tglobal_load_dwordx4 %3, %4, %5 offset:3072 sc0 sc1\n\t"
                             "s_waitcnt vmcnt(0)"
                             : "=&v"(r[0]), "=&v"(r[1]), "=&v"(r[2]), "=&v"(r[3]) : "v"(vo), "s"(base) : "memory");
#pragma unroll
                for (int k = 0; k < 4; ++k)
                    out_store(o, k, f32x4{silu(r[k][0] + P[k][0], sc), silu(r[k][1] + P[k][1], sc), silu(r[k][2] + P[k][2], sc), silu(r[k][3] + P[k][3], sc)});
            } else {
                const u32 sec = vb + (hdr & 0xFF) * 16, N2 = (hdr >> 8) & 0x7FF, n2 = (hdr >> 19) & 15;
                u32x2 hard = {(rb ^ 1) << 4, 0};
                if (lane < n2) hard = L64(sec + lane * 8);
                const u32 he = e0 + (((hard[0] >> 9) & 3) * 4 + ((hard[0] >> 2) & 3)) * GN + ((hard[0] >> 5) & 15) * 4 + (hard[0] & 3);
                float hp = 0.f;
                if (n2) hp = __builtin_bit_cast(float, __builtin_amdgcn_raw_buffer_load_b32(ro, he * 4, 0, 17));
                unpack_levels(std::integral_constant<int, 2>{}, N2, sec + n2 * 8, 0x06030100u, 0x08070500u);
                landed(pbase + (((u + 1) * HEADB + 1023) >> 10));
                const u32 hd = vb + EXCCAP + u * HEADB;
                const u32x2 Q = L64(hd + (lane & 31) * 16 + (lane >> 5) * 8), Nb = L64(hd + 512 + lane * 16 + rb * 8);
                const u32 c0 = __builtin_popcount(escbits(Q[0])), c1 = c0 + __builtin_popcount(escbits(Q[1]));
                const u32 q = rb ? Q[1] : Q[0], e = escbits(q);
                u32 pp = v2 + scan64(c1) - c1 + (rb ? c0 : 0);
                u32 ft[8];
#pragma unroll
                for (int g = 0; g < 4; ++g) {
                    const u32 sy = expand(0x04020000u, pp, (q >> (8 * g)) & 0xFF);
                    pp += __builtin_popcount(e & (0xFFu << (8 * g)));
                    const u32 nb = Nb[g >> 1] >> (4 * (g & 1)), sg = nb & 0x08080808u;
                    const u32 T = (((sy << 2) & 0x78787878u) ^ ((sg << 5) - sg)) | (nb & 0x07070707u);
                    const u32 F = ((sy << 7) & 0x80808080u) | ((sy >> 1) & 0x70707070u);
                    ft[2 * g] = perm(T, F, 0x05010400u); ft[2 * g + 1] = perm(T, F, 0x07030602u);
                }
                landed(pbase + 4 * HEADB / 1024 + (wave + 1) * 2);
                const u32x4 d0 = L128(vb + EXCCAP + 4 * HEADB + wave * 2048 + lane * 16), d1 = L128(vb + EXCCAP + 4 * HEADB + wave * 2048 + 1024 + lane * 16);
                done(m);
                const float dn = __uint_as_float(hdr & 0x7F800000u);
                asm volatile("" : "+v"(P[0]), "+v"(P[1]), "+v"(P[2]), "+v"(P[3]), "+v"(hp));
#pragma unroll
                for (int g = 0; g < 4; ++g) {
                    const u32 m0 = g < 2 ? d0[2 * g] : d1[2 * g - 4], m1 = g < 2 ? d0[2 * g + 1] : d1[2 * g - 3];
                    const int k0 = perm(m0, ft[2 * g], 0x01050400u), k1 = perm(m0, ft[2 * g], 0x03070602u);
                    const int k2 = perm(m1, ft[2 * g + 1], 0x01050400u), k3 = perm(m1, ft[2 * g + 1], 0x03070602u);
                    out_store(o, g, f32x4{silu(__fmul_rn((float)k0, dn) + P[g][0], sc), silu(__fmul_rn((float)k1, dn) + P[g][1], sc),
                                          silu(__fmul_rn((float)k2, dn) + P[g][2], sc), silu(__fmul_rn((float)k3, dn) + P[g][3], sc)});
                }
                if (n2) {
                    WAITV(0);
                    if (((hard[0] >> 4) & 1) == rb)
                        asm volatile("global_store_dword %0, %1, off sc1" :: "v"(a.out + he), "v"(silu(__uint_as_float(hard[1]) + hp, sc)) : "memory");
                }
            }
            ++m;
            pbase += (OHALF - EXCCAP) / 1024;
            next();
        }
        if (LPOLL(C_DEAD)) {
            WAITV(0);
            for (u32 r = wave * 64 + lane; r < MS; r += 512)
                for (u32 c = 0; c < 2048; c += 64) a.out[(size_t)r * GN + s * 2048 + c] = __builtin_nanf("");
        }
    } else {
        for (u32 job = (bid >> 3) - 7, round = 0; job < NJOB; round += job >= 56) {
            FRESH_LANE();
            const bool part = job < 56;
            const u32 n = job - 56;
            const u32 layer = part ? job & 3 : n < 56 ? n / 14 : (n - 56) >> 1, qi = !part && n < 56 ? (n % 14) >> 1 : 7;
            const u32 ct = part ? ((rank + 1) * 16 + (job >> 2) * 8 + li) & 127 : 2 * li + (n & 1);
            const u32 q = (rank + 1 + qi) & 7;
            const u32 nk = part ? 4 : qi == 7 ? 32 : 28, skip = qi == 7 ? 32 : 4 * q;
            if (wave == 0) {
                const u32 sl = part ? (lane & 32) + rank * 4 + (lane & 3) : lane;
                const u32 fo = qi == 7 && (layer < NPL || ((sl >> 2) & 7) == rank) ? OFF_XF + par * 1024 + layer * 256 + sl * 4
                                                                                   : OFF_PF + (q * 4 + layer) * 128;
                bool ok = !LPOLL(C_DEAD);
                while (ok) {
                    const u32 fa = __hip_atomic_load((GAS u32*)(myws + fo), RLX_SYS);
                    const u32 fb = __hip_atomic_load((GAS u32*)(myws + OFF_PF + ((part ? 64 : 32) + ct) * 128), RLX_SYS);
                    if (__all(fa >= call && fb >= call)) break;
                    if (expired(call)) { give_up(call); ok = false; }
                    __builtin_amdgcn_s_sleep(GSLEEP);
                }
            }
            BAR();
            const bool dead = LPOLL(C_DEAD);
            if (wave == 0) STAMP(44 + min(round, 3u) * 3);
            f32x4 acc[2][4];
#pragma unroll
            for (int i = 0; i < 8; ++i) acc[i >> 2][i & 3] = f32x4{0.f, 0.f, 0.f, 0.f};
            for (u32 o = lane * 16; o < WSCR; o += 1024) L128(v2 + o) = u32x4{0, 0, 0, 0};
            if (!dead) {
                const u32 ga = (qi == 7 ? OFF_XO + (par * 4 + layer) * PSTR : OFF_SP + (q * 4 + layer) * PSTR) + (part ? rank * 65536 : 0) + wave * 2048 + lane * 16;
                const u32 gb = (part ? OFF_LW + ct * 65536 : OFF_SP + (32 + ct) * PSTR) + wave * 2048 + lane * 16;
                auto dma = [&](u32 k, int j) {
                    const u32 l = lbase + L_BIG + (k & 3) * 32768 + wave * 2048 + (j >> 1) * 16384 + (j & 1) * 1024;
                    DMA16((j >> 1 ? gb : ga) + (k < skip ? k : k + 4) * 16384 + (j & 1) * 1024, l, myws);
                };
                auto stage = [&](u32 k) { dma(k, 0); dma(k, 1); dma(k, 2); dma(k, 3); };
                u32x4 fr[2][12];
                auto step = [&](auto B, auto MM, auto RD, auto ST, u32 k) {
                    const u32 sa = L_BIG + (k & 3) * 32768 + rg * 4096 + lane * 16, sb = L_BIG + (k & 3) * 32768 + 16384 + ch * 8192 + lane * 16;
#pragma unroll
                    for (int i = 0; i < 8; ++i) {
                        if (MM.value) {
                            const u32x4 *pa = &fr[B.value ^ 1][(i & 1) * 2], *pb = &fr[B.value ^ 1][4 + (i >> 1) * 2];
                            const i32x8 af = {(int)pa[0][0], (int)pa[0][1], (int)pa[0][2], (int)pa[0][3], (int)pa[1][0], (int)pa[1][1], (int)pa[1][2], (int)pa[1][3]};
                            const i32x8 bf = {(int)pb[0][0], (int)pb[0][1], (int)pb[0][2], (int)pb[0][3], (int)pb[1][0], (int)pb[1][1], (int)pb[1][2], (int)pb[1][3]};
                            acc[i & 1][i >> 1] = __builtin_amdgcn_mfma_scale_f32_16x16x128_f8f6f4(af, bf, acc[i & 1][i >> 1], 1, 1, 0, 0x7F7F7F7F, 0, 0x7F7F7F7F);
                        }
                        __builtin_amdgcn_sched_barrier(0);
                        if (RD.value && i < 6) {
                            const u32 o = i < 2 ? sa + i * 2048 : sb + (i - 2) * 2048;
                            fr[B.value][2 * i] = L128(o);
                            fr[B.value][2 * i + 1] = L128(o + 1024);
                        }
                        if (ST.value && (i & 1)) dma(k + 3, i >> 1);
                        __builtin_amdgcn_sched_barrier(0);
                    }
                };
                constexpr std::integral_constant<int, 0> N{};
                constexpr std::integral_constant<int, 1> Y{};
                stage(0); stage(1); stage(2);
                WAITV(8); BAR();
                step(N, N, Y, Y, 0);
                for (u32 k = 1; k < nk - 3; k += 2) {
                    WAITV(8); BAR();
                    step(Y, Y, Y, Y, k);
                    WAITV(8); BAR();
                    step(N, Y, Y, Y, k + 1);
                }
                WAITV(8); BAR();
                step(Y, Y, Y, N, 1);
                WAITV(4); BAR();
                step(N, Y, Y, N, 2);
                WAITV(0); BAR();
                step(Y, Y, Y, N, 3);
                step(N, Y, N, N, 0);
            }
            FRESH_LANE();
            u32 ticket = 0;
            if (threadIdx.x == 0) ticket = __hip_atomic_fetch_add((GAS u32*)(myws + OFF_Q + li * 128), 1u, RLX_AG);
            if (wave == 0) STAMP(45 + min(round, 3u) * 3);
#define ROW(g) (f32x4{acc[g >> 2][0][g & 3], acc[g >> 2][1][g & 3], acc[g >> 2][2][g & 3], acc[g >> 2][3][g & 3]})
            if (part) {
                if (!dead) {
                    float* o = out_ptr(layer * 128, ct * 128);
#pragma unroll
                    for (int g = 0; g < 8; ++g) out_store(o, g, ROW(g));
                    WAITV(0);
                    BAR();
                    if (wave == 1) raise(48 + (ct >> 4), arrive(48 + (ct >> 4)), call);
                }
            } else if (qi == 7) {
                const bool fail = dead || uni(__hip_atomic_load((GAS u32*)(myws + OFF_TMO), RLX_AG)) == call;
                const float s2 = fail ? __builtin_nanf("") : sc;
                float* o = out_ptr(layer * 128, rank * 2048 + ct * 128);
#pragma unroll
                for (int g = 0; g < 8; ++g)
                    out_store(o, g, f32x4{silu(acc[g >> 2][0][g & 3], s2), silu(acc[g >> 2][1][g & 3], s2),
                                          silu(acc[g >> 2][2][g & 3], s2), silu(acc[g >> 2][3][g & 3], s2)});
                WAITV(0);
            } else if (!dead) {
                const u32 id = (layer * 16 + ct) * 2 + (wave >> 2), to = OFF_OP + par * OP_PAR + (q * 128 + id) * OHALF;
                u32 esum = 0;
#pragma unroll
                for (int i = 0; i < 8; ++i) esum += (__float_as_uint(acc[i >> 2][i & 3][(i + (i >> 2)) & 3]) >> 19) & 0xFFF;
                const u32 etop = min(max((last(scan64(esum)) + 37683u) >> 13, 40u), 250u);
                const float up = __uint_as_float((285 - etop) << 23), dn = __uint_as_float((etop - 31) << 23);
                u32 d[16], sy[8], Nb[4] = {0, 0, 0, 0}, Q[2] = {0, 0}, hardm = 0, off = 0;
                u32x2 lut[8];
                const u32 vm = to + EXCCAP + 4 * HEADB + (wave & 3) * 4096 + lane * 16;
#pragma unroll
                for (int g = 0; g < 8; ++g) {
                    u32 k[4];
#pragma unroll
                    for (int cc = 0; cc < 4; ++cc) {
                        const float v = acc[g >> 2][cc][g & 3];
                        k[cc] = (u32)cvt_i32(v * up);
                        off |= __float_as_uint((float)(int)k[cc] * dn) ^ __float_as_uint(v);
                    }
                    d[2 * g] = perm(k[1], k[0], 0x06050201u); d[2 * g + 1] = perm(k[3], k[2], 0x06050201u);
                    const u32 a01 = perm(k[1], k[0], 0x07030400u), a23 = perm(k[3], k[2], 0x07030400u);
                    const u32 F = perm(a23, a01, 0x05040100u), T = perm(a23, a01, 0x07060302u);
                    off |= F & 0x0F0F0F0Fu;
                    const u32 sg = (T >> 4) & 0x08080808u;
                    sy[g] = (((T ^ ((sg << 5) - sg)) >> 2) & 0x1E1E1E1Eu) | ((F >> 7) & 0x01010101u) | ((F << 1) & 0xE0E0E0E0u);
                    Nb[g >> 1] |= ((T & 0x07070707u) | sg) << (4 * (g & 1));
                    const u32 x8 = lookup8(sy[g], 0x00000018u, 0x00100008u);
                    lut[g] = L64(L_LUTC + x8);
                    Q[g >> 2] |= g & 3 ? x8 << (8 * (g & 3) - 3) : x8 >> 3;
                    if (g & 1) {
                        ST16(vm + (g >> 1) * 1024, (u32x4{d[2 * g - 2], d[2 * g - 1], d[2 * g], d[2 * g + 1]}), myws, 0);
                        asm volatile("" : "+v"(off), "+v"(sy[g - 1]), "+v"(sy[g]), "+v"(Nb[g >> 1]), "+v"(Q[g >> 2]));
                        __builtin_amdgcn_sched_barrier(0);
                    }
                }
                if (__any(off != 0)) {
                    float up2 = up;
                    asm volatile("" : "+v"(up2));
#pragma unroll
                    for (int i = 0; i < 32; ++i) {
                        const float v = acc[i >> 4][i & 3][(i >> 2) & 3];
                        hardm |= (u32)(__float_as_uint((float)(cvt_i32(v * up2) & ~15) * dn) != __float_as_uint(v)) << i;
                    }
                }
                {
                    const u32 vh = to + EXCCAP + (wave & 3) * HEADB + lane * 16;
                    ST16(vh, (u32x4{Nb[0], Nb[1], Nb[2], Nb[3]}), myws, 512);
                    const u32x4 q4 = {Q[0], Q[1], (u32)__shfl_xor((int)Q[0], 32), (u32)__shfl_xor((int)Q[1], 32)};
                    if (lane < 32) ST16(vh, q4, myws, 0);
                }
                const u32 cnt = (__builtin_popcount(escbits(Q[0])) + __builtin_popcount(escbits(Q[1]))) | (__builtin_popcount(hardm) << 16);
                const u32 inc = scan64(cnt), tot = last(inc);
                const u32 N2 = tot & 0xFFFF, n2 = tot >> 16;
                u32 len = ~0u;
                if (N2 <= OCAP2 && n2 <= HARDCAP) {
                    append(std::integral_constant<int, 8>{}, v2 + ((inc - cnt) & 0xFFFF), sy, lut);
                    len = pack_levels(std::integral_constant<int, 2>{}, N2, ev + n2 * 8, [&](u32 u) { return lookup8(u, 0x00180000u, 0x10000800u); },
                                      [&](u32 u) { return lookup8(u - 0x01010101u, 0x18100008u, 0u); });
                    if (len != ~0u) len += n2 * 8;
                    if (n2) {
                        const u32 dump = L_BIG + wave * 8192;
#pragma unroll
                        for (int g = 0; g < 8; ++g) *(LAS f32x4*)(L + dump + g * 1024 + lane * 16) = ROW(g);
                        u32 o = ev + ((inc - cnt) >> 16) * 8;
                        while (hardm) {
                            const u32 b = __builtin_ctz(hardm);
                            hardm &= hardm - 1;
                            L64(o) = u32x2{lane * 32 + b, L32(dump + (b >> 2) * 1024 + lane * 16 + (b & 3) * 4)};
                            o += 8;
                        }
                    }
                }
                const bool raw = len > OSECCAP || FORCE_RAW & 2;
                if (raw) {
                    len = 0;
                    const u32 vo = OFF_OR + par * OR_PAR + ((q * 128 + id) * 4 + (wave & 3)) * 8192 + lane * 16 + 4096;
                    ST16(vo, ROW(0), myws, -4096); ST16(vo, ROW(1), myws, -3072); ST16(vo, ROW(2), myws, -2048); ST16(vo, ROW(3), myws, -1024);
                    ST16(vo, ROW(4), myws, 0);     ST16(vo, ROW(5), myws, 1024);  ST16(vo, ROW(6), myws, 2048);  ST16(vo, ROW(7), myws, 3072);
                }
                const u32 len16 = (len + 15) >> 4;
                if (lane == 0) L32(C_HDR + wave * 4) = len16 | (raw ? 0x80000000u : (N2 << 8) | (n2 << 19) | ((etop - 31) << 23));
                BAR();
                const u32 hv = lane < 4 ? L32(C_HDR + (wave & 4) * 4 + lane * 4) : 0u;
                const u32 hin = scan64(hv & 0xFF);
                const u32 hoff = 1 + hin - (hv & 0xFF);
                if (lane < len16) {
                    const u32x4 sec = L128(ev + lane * 16);
                    ST16(to + __builtin_amdgcn_readlane(hoff, wave & 3) * 16 + lane * 16, sec, myws, 0);
                }
                if ((wave & 3) == 0 && lane < 4) ST4(to + lane * 4, (hv & ~0xFFu) | hoff, myws);
                WAITV(0);
                BAR();
                if ((wave & 3) == 1 && lane == 0) {
                    const u64 fl = ((u64)((1 + last(hin) + 7) >> 3) << 32) | call;
                    __hip_atomic_store((GAS u64*)(a.ws[q] + OFF_F + (par * 8 + rank) * 2048 + (NRUN + id) * 8), fl, RLX_SYS);
                }
            }
            if (threadIdx.x == 0) L32(C_NEXT) = 25 + ticket - NJOB * (call - 1);
            BAR();
            job = LPOLL(C_NEXT);
            if (wave == 0) STAMP(46 + min(round, 3u) * 3);
        }
    }
    if (threadIdx.x == 0) __hip_atomic_store((GAS u32*)(myws + OFF_CNT + bid * 128), call, RLX_AG);
}

extern "C" void kernel_launch(int rank, int nranks, ncclComm_t comm,
                              hipStream_t stream,
                              void* const* d_in, const int* in_sizes, int n_in,
                              void* d_out, int out_size,
                              const int* in_dtypes, int out_dtype,
                              void* const* d_out_all,
                              void* const* d_ws_all, size_t ws_bytes) {
    if (nranks != NR || (unsigned)rank >= (unsigned)NR || ws_bytes < WS_NEED || n_in != 4
        || in_sizes[0] != 4096 * KS || in_sizes[1] != KS * GN || out_size != MS * GN) {
        fprintf(stderr, "unexpected problem shape\n");
        return;
    }
    Args a;
    a.x = (const u8*)d_in[0];
    a.w = (const u8*)d_in[1];
    a.sx = (const float*)d_in[2];
    a.sw = (const float*)d_in[3];
    a.out = (float*)d_out;
    for (int p = 0; p < NR; ++p) a.ws[p] = (u8*)d_ws_all[p];
    a.self = a.ws[rank];
    a.rank = rank;
    a.pad = 0;
    fused_gemm_rs<<<NWG, NTHR, 0, stream>>>(a);
    HIP_CHECK(hipGetLastError());
}
```

```cpp
#include <hip/hip_runtime.h>
#include <rccl/rccl.h>
#include <cstdio>
#include <cstdint>
#include <type_traits>

#define HIP_CHECK(expr) do { hipError_t _e = (expr); if (_e != hipSuccess) { \
    fprintf(stderr, "HIP error %s at %s:%d\n", hipGetErrorString(_e), __FILE__, __LINE__); \
    return; } } while (0)

typedef unsigned u32;
typedef unsigned long long u64;
typedef unsigned char u8;
typedef u32 u32x2 __attribute__((ext_vector_type(2)));
typedef u32 u32x4 __attribute__((ext_vector_type(4)));
typedef int i32x8 __attribute__((ext_vector_type(8)));
typedef float f32x4 __attribute__((ext_vector_type(4)));
#define GAS __attribute__((address_space(1)))
#define LAS __attribute__((address_space(3)))

#define WINDOW 6
#define WINDOW2 14
#define BOOSTN 0
#define NPL 2
#define NRL 5
#define FORCE_RAW 0
#define ALIGNMASK 0
#define GSLEEP 8
#define XWIN 4
#define STAMPS 0

constexpr int NR = 8;
constexpr int GN = 16384;
constexpr int KS = 512;
constexpr int MS = 512;
constexpr int NWG = 256;
constexpr int NTHR = 704;
constexpr u32 NOWN = 8 - 2 * NPL, WDIR = 64 - 4 * NRL;
constexpr u32 S_W = 0, S_X = 64, S_RL = 128, S_O = 256, S_EARLY = 384, S_END = 511;

constexpr u32 VARCAP = 5120;
constexpr u32 HALFB  = 2560;
constexpr u32 RUN    = VARCAP + 8 * HALFB;
constexpr u32 CAP2 = 1536, CAP3 = 512, SECCAP = 608;
static_assert(32 + 8 * SECCAP <= VARCAP - 128, "sections fit; the last line of the variable part is never used");
constexpr u32 EXCCAP = 3072;
constexpr u32 HEADB  = 1536;
constexpr u32 OHALF  = EXCCAP + 4 * HEADB + 8 * 2048;
constexpr u32 OCAP2 = 1024, OSECCAP = 752, HARDCAP = 15;
static_assert(16 + 4 * OSECCAP <= EXCCAP && OHALF == RUN, "sections fit");

constexpr u32 OFF_CNT = 0;
constexpr u32 OFF_TMO = 32u << 10;
constexpr u32 OFF_Q   = 40u << 10;
constexpr u32 OFF_XF  = 48u << 10;
constexpr u32 OFF_CT  = 64u << 10;
constexpr u32 OFF_PF  = 96u << 10;
constexpr u32 OFF_F   = 128u << 10;
constexpr u32 OFF_DBG = 256u << 10;
constexpr u32 PSTR    = 32 * 16384 + 4352;
constexpr u32 OFF_XO  = 1u << 20;
constexpr u32 OFF_SP  = 6u << 20;
constexpr u32 OFF_P   = 40u << 20;
constexpr u32 P_PAR   = 11u << 20;
constexpr u32 OFF_RAW = 62u << 20;
constexpr u32 RAW_PAR = 10u << 20;
constexpr u32 OFF_OP  = 82u << 20;
constexpr u32 OP_PAR  = 25u << 20;
constexpr u32 OFF_OR  = 132u << 20;
constexpr u32 OR_PAR  = 32u << 20;
constexpr u32 OFF_LW  = 196u << 20;
constexpr u32 WS_NEED = OFF_LW + (8u << 20);
static_assert(OFF_XO + 8 * PSTR <= OFF_SP && OFF_SP + 64 * PSTR <= OFF_P && 448 * RUN <= P_PAR
              && 320 * 32768 <= RAW_PAR && 1024 * OHALF <= OP_PAR && NRL <= 8, "workspace");
constexpr u64 DEADLINE_TICKS = 50000000ull;

constexpr u32 L_LUTD = 0;
constexpr u32 L_LUTC = 1024;
constexpr u32 L_BIG  = 3072;
constexpr u32 L_ZERO = L_BIG + 131072;
constexpr u32 V2S = CAP2 + 16, V3S = CAP3 + 16, EVS = 1040;
constexpr u32 WSCR   = V2S + V3S + EVS;
constexpr u32 L_SCR  = L_ZERO;
constexpr u32 L_CTL  = L_SCR + 8 * WSCR;
constexpr u32 C_WB = L_CTL, C_LANDED = L_CTL + 4, C_DEAD = L_CTL + 8, C_PUB = L_CTL + 12, C_DONE = L_CTL + 16;
constexpr u32 C_HDR = L_CTL + 32, C_START = L_CTL + 64, C_NEXT = L_CTL + 68, C_PUB1 = L_CTL + 72, C_RL = L_CTL + 80;
constexpr u32 C_RDY = L_CTL + 96;
constexpr u32 L_END  = L_CTL + 192;
static_assert(L_BIG + 4 * RUN <= L_ZERO && WSCR % 16 == 0 && L_END % 16 == 0 && L_END <= 160 * 1024, "LDS");
static_assert(CAP2 / 4 + CAP3 / 4 + CAP3 + 16 <= EVS && 8 * HARDCAP + OCAP2 / 4 + CAP3 / 4 + CAP3 + 16 <= EVS, "section scratch");

struct Args {
    const u8* x;
    const u8* w;
    const float* sx;
    const float* sw;
    float* out;
    u8* self;
    u8* ws[NR];
    int rank;
    int pad;
};

#define ST16(voff, data, sbase, imm) \
    asm volatile("global_store_dwordx4 %0, %1, %2 offset:" #imm " sc0 sc1\n\ts_nop 1" \
                 :: "v"(voff), "v"(data), "s"(sbase) : "memory")
#define ST4(voff, data, sbase) \
    asm volatile("global_store_dword %0, %1, %2 sc0 sc1" :: "v"(voff), "v"(data), "s"(sbase) : "memory")
#define DMA16(voff, ldsaddr, sbase) do { u32 keep_; \
    asm volatile("s_mov_b32 %0, m0\n\ts_mov_b32 m0, %2\n\ts_nop 0\n\t" \
                 "global_load_lds_dwordx4 %1, %3 sc0 sc1\n\ts_mov_b32 m0, %0" \
                 : "=&s"(keep_) : "v"(voff), "s"(ldsaddr), "s"(sbase) : "memory"); } while (0)
#define WAITV(n) asm volatile("s_waitcnt vmcnt(%0)" :: "n"(n) : "memory")
#define WAITL() asm volatile("s_waitcnt lgkmcnt(0)" ::: "memory")
#define BAR() asm volatile("s_waitcnt lgkmcnt(0)\n\ts_barrier" ::: "memory")
#define RLX_WG __ATOMIC_RELAXED, __HIP_MEMORY_SCOPE_WORKGROUP
#define RLX_AG __ATOMIC_RELAXED, __HIP_MEMORY_SCOPE_AGENT
#define RLX_SYS __ATOMIC_RELAXED, __HIP_MEMORY_SCOPE_SYSTEM
#define CFENCE() asm volatile("" ::: "memory")

__device__ __forceinline__ u64 rt() { return __builtin_amdgcn_s_memrealtime(); }
__device__ __forceinline__ u32 uni(u32 v) { return __builtin_amdgcn_readfirstlane(v); }
__device__ __forceinline__ u32 perm(u32 hi, u32 lo, u32 sel) { return __builtin_amdgcn_perm(hi, lo, sel); }
__device__ __forceinline__ u32 scan64(u32 v) {
    v += __builtin_amdgcn_update_dpp(0u, v, 0x111, 0xf, 0xf, true);
    v += __builtin_amdgcn_update_dpp(0u, v, 0x112, 0xf, 0xf, true);
    v += __builtin_amdgcn_update_dpp(0u, v, 0x114, 0xf, 0xf, true);
    v += __builtin_amdgcn_update_dpp(0u, v, 0x118, 0xf, 0xf, true);
    v += __builtin_amdgcn_update_dpp(0u, v, 0x142, 0xa, 0xf, false);
    v += __builtin_amdgcn_update_dpp(0u, v, 0x143, 0xc, 0xf, false);
    return v;
}
__device__ __forceinline__ u32 last(u32 v) { return __builtin_amdgcn_readlane(v, 63); }
__device__ __forceinline__ u32 escbits(u32 q) { return ~(q | (q >> 1)) & 0x55555555u; }
__device__ __forceinline__ u32 lookup8(u32 sel, u32 thi, u32 tlo) {
    const u32 r = perm(thi, tlo, sel);
    return __builtin_amdgcn_udot4((r ^ (r >> 2)) & 0x18181818u, 0x40100401u, 0u, false);
}
__device__ __forceinline__ u32 classify8(u32 e4, u32 kadd, u32 thi, u32 tlo) {
    return lookup8(((e4 >> 2) + kadd) & 0x1F1F1F1Fu, thi, tlo);
}
constexpr u32 T13 = 0x00181008u, T2L = 0x00000818u, T2H = 0x00001000u;
__device__ __forceinline__ u32 ones(u32 sel) { return perm(0u, 0x00008000u, sel); }
__device__ __forceinline__ u32 rep(u32 b) { return b * 0x01010101u; }
__device__ __forceinline__ u32 blk(u32 r, u32 b) { return (r < 4 ? r : r - b) & 3; }
__device__ __forceinline__ u32 partner(u32 r, u32 b) { return r < 4 ? 4 + ((r + b) & 3) : (r - b) & 3; }
__device__ __forceinline__ float silu(float acc, float sc) {
    const float y = acc * sc;
    const float e = __expf(-__builtin_amdgcn_fmed3f(y, -60.f, 60.f));
    return y * __builtin_amdgcn_rcpf(1.f + e);
}
__device__ __forceinline__ int cvt_i32(float f) { int r; asm("v_cvt_i32_f32 %0, %1" : "=v"(r) : "v"(f)); return r; }

__global__ void __launch_bounds__(NTHR) fused_gemm_rs(Args a) {
    __shared__ __attribute__((aligned(16))) u8 lds_raw[L_END];
    LAS u8* const L = (LAS u8*)lds_raw;
#define L8(off) (*(LAS u8*)(L + (off)))
#define L16(off) (*(LAS unsigned short*)(L + (off)))
#define L32(off) (*(LAS u32*)(L + (off)))
#define L64(off) (*(LAS u32x2*)(L + (off)))
#define L128(off) (*(LAS u32x4*)(L + (off)))
#define LPOLL(off) uni(__hip_atomic_load((LAS u32*)(L + (off)), RLX_WG))
    const u32 bid = blockIdx.x;
    u32 lane = threadIdx.x & 63;
#define FRESH_LANE() do { asm volatile("" : "+v"(lane)); __builtin_assume(lane < 64); } while (0)
    const u32 wave = uni(threadIdx.x >> 6);
    const u32 rank = a.rank;
    const bool is_link = (bid >> 3) < 7;
    u8* const myws = a.self;
    const u32 lbase = (u32)(size_t)L;
    const u64 t_begin = rt();
#define STAMP(i)

    const u32 tile = ((bid & 7) | ((bid >> 1) & 8)) * 16 + (bid >> 5) * 2 + ((bid >> 3) & 1);
    const u32 xrun = 256 + ((bid >> 1) & 7) * 8 + (7 - (bid >> 5)) * 4 + ((bid >> 4) & 1) * 2 + (bid & 1);
    u32 f[4][4];
    auto fetch_run = [&](u32 id) {
        if (id >= 256) {
            const __amdgpu_buffer_rsrc_t rx = __builtin_amdgcn_make_buffer_rsrc((void*)a.x, 0, 4096 * KS, 0x00020000);
            const u32 vo = (lane & 15) * KS + (lane >> 4) * 32, so = (id - 256) * 64 * KS + (wave >> 1) * 128 + (wave & 1) * 16;
#pragma unroll
            for (int c = 0; c < 4; ++c) {
                const u32x4 v = __builtin_amdgcn_raw_buffer_load_b128(rx, vo, so + c * 16 * KS, 0);
                f[c][0] = v[0]; f[c][1] = v[1]; f[c][2] = v[2]; f[c][3] = v[3];
            }
        } else {
            const __amdgpu_buffer_rsrc_t rw = __builtin_amdgcn_make_buffer_rsrc((void*)a.w, 0, KS * GN, 0x00020000);
            const u32 vo = (lane >> 4) * 32 * GN + (lane & 15) * 4, so = ((wave >> 1) * 128 + (wave & 1) * 16) * GN + id * 64;
#pragma unroll
            for (int r = 0; r < 16; ++r) f[r >> 2][r & 3] = __builtin_amdgcn_raw_buffer_load_b32(rw, vo, so + r * GN, 0);
        }
        __builtin_amdgcn_sched_barrier(0);
    };
    if (wave < 8) fetch_run(bid >= 224 ? xrun : tile);

    const u32 cnt_v = __hip_atomic_load((GAS u32*)(myws + OFF_CNT + bid * 128), RLX_AG);
#define GET_CALL const u32 call = uni(cnt_v) + 1, par = call & 1
    u32 spins = 0;
    auto expired = [&](u32 call) -> bool {
        if (++spins & 15) return false;
        return uni(__hip_atomic_load((GAS u32*)(myws + OFF_TMO), RLX_AG)) == call || rt() - t_begin > DEADLINE_TICKS;
    };
    auto give_up = [&](u32 call) {
        if (lane == 0) {
            __hip_atomic_store((GAS u32*)(myws + OFF_TMO), call, RLX_AG);
            __hip_atomic_store((LAS u32*)(L + C_DEAD), 1u, RLX_WG);
        }
    };
    auto lds_init = [&]() {
        for (u32 o = L_ZERO + threadIdx.x * 16; o < L_END; o += 576 * 16) L128(o) = u32x4{0, 0, 0, 0};
        if (threadIdx.x < 256) {
            const u32 x = threadIdx.x;
            u32 d = 0, c = 0x0C0C0C0Cu, n = 0;
#pragma unroll
            for (int b = 0; b < 4; ++b) {
                const u32 code = (x >> (2 * b)) & 3;
                d |= (code ? 4 + code : n) << (8 * b);
                if (!code) c = (c & ~(0xFFu << (8 * n))) | (b << (8 * n));
                n += !code;
            }
            L32(L_LUTD + x * 4) = d;
            L64(L_LUTC + x * 8) = u32x2{c, n};
        }
    };
    const u32 li = bid & 7, from = (rank + 1 + (bid >> 3)) & 7;
    const bool cross = ((rank ^ from) & 4) != 0;
    const u32 pband = (rank < 4 ? from - rank : rank - from) & 3;
    const u32 nitem = cross ? 88 + NOWN + 3 * NRL + 96 : 24 + WDIR + NOWN + 128;
    auto slot_of = [&](u32 n) -> u32 {
        const u32 ns = cross ? 12 : 8, nh = ns / 2, fj = (from - rank) & 3;
        if (n < ns + 8) {
            if (n >= nh && n < nh + 8) return S_W + n - nh;
            const u32 m = n < nh ? n : n - nh - 8, j = 1 + (m >> 1);
            return S_X + ((rank & 4) | ((rank + j + (!cross && j >= fj)) & 3)) * 8 + (n < nh ? 0 : 2) + (m & 1);
        }
        if (n < ns + 24) return S_W + n - ns;
        if (n < 48) { const u32 m = n - ns - 24, j = m >> 2; return S_X + ((~rank & 4) | ((rank + j + (cross && j >= fj)) & 3)) * 8 + 4 + (m & 3); }
        const u32 nw = 24 + (cross ? 64 : WDIR);
        if (n < nw) {
            const u32 o = n - 24, wf = WDIR & ~7u;
            return S_W + (!cross || o < wf ? o : o < wf + 4 * NRL ? o - wf + WDIR : o - 4 * NRL);
        }
        if (n < nw + NOWN) return S_X + rank * 8 + 2 * NPL + n - nw;
        u32 m = n - nw - NOWN;
        if (!cross) return S_O + ((m & ~14u) | ((m & 6) << 1) | ((m & 8) >> 2));
        if (m < 3 * NRL) return S_RL + ((rank + 1 + m % 3) & 3) * 8 + m / 3;
        m -= 3 * NRL;
        const u32 b = (m % 12) >> 2;
        return S_O + (m / 12) * 16 + (b + (b >= pband)) * 4 + (m & 3);
    };
    auto run_id = [&](u32 sl) -> u32 {
        if (sl >= S_X && sl < S_RL) return 256 + sl - S_X;
        const u32 o = sl < S_X ? sl : (16 - NRL + (sl & 7)) * 4 + pband;
        return (o & 3) * 64 + blk(rank, o & 3) * 16 + (o >> 2);
    };
    auto run_src = [&](u32 sl) -> u32 { return sl < S_RL ? from : (rank & 4) | ((sl >> 3) & 3); };
    auto run_pos = [&](u32 sl) -> u32 { return sl < S_RL ? run_id(sl) : 320 + pband * 32 + sl - S_RL; };

    if (wave == 9) {
        const u32 q = (rank + 1 + bid) & 7, t = (bid >> 3) & 3, rg = bid >> 5;
        if (q != rank && rg >= 2 * NPL) {
            asm volatile("s_barrier" ::: "memory");
            return;
        }
        const __amdgpu_buffer_rsrc_t rx = __builtin_amdgcn_make_buffer_rsrc((void*)a.x, 0, 4096 * KS, 0x00020000);
        u32x4 v[4][2];
#pragma unroll
        for (int rb = 0; rb < 4; ++rb)
#pragma unroll
            for (int p = 0; p < 2; ++p)
                v[rb][p] = __builtin_amdgcn_raw_buffer_load_b128(rx, (lane & 15) * KS + (lane >> 4) * 32 + p * 16, (q * MS + rg * 64 + rb * 16) * KS + t * 128, 0);
        u8* dst = a.ws[q];
        asm volatile("s_barrier" ::: "memory");
        GET_CALL;
        asm volatile("" :: "v"(v[0][0]), "v"(v[0][1]), "v"(v[1][0]), "v"(v[1][1]),
                          "v"(v[2][0]), "v"(v[2][1]), "v"(v[3][0]), "v"(v[3][1]));
        const u32 vo = OFF_XO + (par * 4 + (rg >> 1)) * PSTR + (rank * 4 + t) * 16384 + (rg & 1) * 8192 + lane * 16 + 4096;
        STAMP(4);
#define XW() WAITV(XWIN - 1)
        ST16(vo, v[0][0], dst, -4096); XW(); ST16(vo, v[0][1], dst, -3072); XW();
        ST16(vo, v[1][0], dst, -2048); XW(); ST16(vo, v[1][1], dst, -1024); XW();
        ST16(vo, v[2][0], dst, 0);     XW(); ST16(vo, v[2][1], dst, 1024);  XW();
        ST16(vo, v[3][0], dst, 2048);  XW(); ST16(vo, v[3][1], dst, 3072);
        WAITV(0);
        if (lane == 0) __hip_atomic_store((GAS u32*)(dst + OFF_XF + par * 1024 + rg * 128 + (rank * 4 + t) * 4), call, RLX_SYS);
        STAMP(1);
        return;
    }
    if (wave == 10) {
        asm volatile("s_barrier" ::: "memory");
        GET_CALL;
        if (!is_link) return;
        __builtin_amdgcn_s_setprio(2);
        bool part_ok = false;
        for (u32 k = 0;; ++k) {
            u32 n = li + 8 * k;
            if (k > 1) {
                while (LPOLL(C_START) < k) __builtin_amdgcn_s_sleep(2);
                u32 tk = 0;
                if (lane == 0) tk = __hip_atomic_fetch_add((GAS u32*)(myws + OFF_Q + (8 + (bid >> 3)) * 128), 1u, RLX_AG);
                n = 16 + uni(tk) - (nitem - 8) * (call - 1);
            }
            u32 lines = 0, sl = S_END;
            if (n < nitem) {
                sl = slot_of(n);
                while (sl >= S_O && !part_ok) {
                    const u32 f = __hip_atomic_load((GAS u32*)(myws + OFF_PF + (64 + (lane & 15)) * 128), RLX_SYS);
                    part_ok = __all(f >= call || (lane & 3) == blk(rank, (lane >> 2) & 3));
                    if (!part_ok && expired(call)) break;
                }
                for (;;) {
                    const u64 f = __hip_atomic_load((GAS u64*)(myws + OFF_F + (par * 8 + from) * 4096 + sl * 8), RLX_SYS);
                    lines = uni((u32)(f >> 32)) & 0xFF;
                    if (uni((u32)f) >= call && (sl < S_O || part_ok)) break;
                    if (expired(call)) { give_up(call); sl = S_END; break; }
                    __builtin_amdgcn_s_sleep(2);
                }
            } else if (uni(__hip_atomic_load((GAS u32*)(myws + OFF_TMO), RLX_AG)) == call) {
                give_up(call);
            }
            while (LPOLL(C_DONE + (k & 3) * 4) < 8 * (k >> 2)) __builtin_amdgcn_s_sleep(1);
            CFENCE();
            lines = min(max(lines, 1u), (sl < S_O ? VARCAP : EXCCAP) / 128);
            const u32 boost = n >= nitem - BOOSTN;
            const u32 src = sl >= S_O ? OFF_OP + par * OP_PAR + (rank * 128 + ((sl - S_O) & 127)) * OHALF : OFF_P + par * P_PAR + run_pos(sl) * RUN;
            if (lane == 0)
                __hip_atomic_store((LAS u64*)(L + C_RDY + (k & 7) * 8), ((u64)src << 32) | ((k + 1) << 16) | (boost << 15) | (sl << 6) | lines, RLX_WG);
            if (sl == S_END) return;
        }
    }
    if (wave == 8) {
        lds_init();
        GET_CALL;
        BAR();
        if (!is_link) return;
        __builtin_amdgcn_s_setprio(3);
        const u8* base = a.ws[from];
        const u32 l16 = lane * 16, a_landed = lbase + C_LANDED;
        int issued = 0;
        auto publish = [&](int v) {
            asm volatile("s_mov_b64 exec, 1\n\tds_max_i32 %0, %1\n\ts_mov_b64 exec, -1" :: "v"(a_landed), "v"(v) : "memory");
        };
#define PULL(W) asm volatile("s_mov_b32 m0, %1\n\ts_mov_b64 exec, %3\n\ts_waitcnt vmcnt(%4)\n\t" \
                             "global_load_lds_dwordx4 %0, %2 sc0 sc1\n\ts_mov_b64 exec, -1" \
                             :: "v"(l16), "s"(lbase + loff), "s"(base + goff), "s"(mask), "n"(W - 1) : "memory")
        auto pull = [&](u32 goff, u32 loff, u64 mask, bool boost) {
            if (boost) { PULL(WINDOW2); publish(issued - (WINDOW2 - 1)); }
            else { PULL(WINDOW); publish(issued - (WINDOW - 1)); }
            ++issued;
        };
        bool early = false;
        for (;;) {
            early = uni(__hip_atomic_load((GAS u32*)(myws + OFF_F + (par * 8 + from) * 4096 + (slot_of(li) + S_EARLY) * 8), RLX_SYS)) >= call;
            if (early) break;
            if (expired(call)) { give_up(call); break; }
        }
        CFENCE();
        STAMP(2);
        if (!early) issued = 8 * HALFB / 1024;
        else
            for (u32 o = VARCAP; o < RUN; o += 1024) pull(OFF_P + par * P_PAR + run_pos(slot_of(li)) * RUN + o, L_BIG + o, ~0ull, false);
        for (u32 m = 0;; ++m) {
            u64 r64 = __hip_atomic_load((LAS u64*)(L + C_RDY + (m & 7) * 8), RLX_WG);
            if ((uni((u32)r64) >> 16) != m + 1) {
                WAITV(0);
                publish(issued);
                while (uni((u32)(r64 = __hip_atomic_load((LAS u64*)(L + C_RDY + (m & 7) * 8), RLX_WG))) >> 16 != m + 1) __builtin_amdgcn_s_sleep(1);
            }
            const u32 rdy = uni((u32)r64), g0 = uni((u32)(r64 >> 32));
            CFENCE();
            const u32 sl = (rdy >> 6) & 511;
            if (sl == S_END) break;
            const bool boost = (rdy >> 15) & 1;
            const u32 l0 = L_BIG + (m & 3) * RUN;
            for (u32 o = 0, ln = rdy & 63; ; o += 1024, ln -= 8) {
                pull(g0 + o, l0 + o, ln >= 8 ? ~0ull : (1ull << (8 * ln)) - 1, boost);
                if (ln <= 8) break;
            }
            if (lane == 0) __hip_atomic_store((LAS u32*)(L + C_START), m + 1, RLX_WG);
            if (m) for (u32 o = sl >= S_O ? EXCCAP : VARCAP; o < RUN; o += 1024) pull(g0 + o, l0 + o, ~0ull, boost);
        }
        WAITV(12); publish(issued - 12); WAITV(10); publish(issued - 10); WAITV(8); publish(issued - 8); WAITV(6); publish(issued - 6);
        WAITV(4); publish(issued - 4); WAITV(2); publish(issued - 2); WAITV(0); publish(issued);
        STAMP(21);
        return;
    }

    u32 phase = 0;
    auto wbar = [&]() {
        phase += 8;
        WAITL();
        if (lane == 0) __hip_atomic_fetch_add((LAS u32*)(L + C_WB), 1u, RLX_WG);
        while (LPOLL(C_WB) < phase) __builtin_amdgcn_s_sleep(1);
        CFENCE();
    };
    auto append = [&](auto G, u32 pos, const u32* e4, const u32x2* lut) {
        u32 carry = 0;
#pragma unroll
        for (int i = 0; i < G.value; ++i) {
            const u32 comp = perm(0u, e4[i], lut[i][0]), sh = pos << 3;
            __hip_atomic_fetch_or((LAS u32*)(L + (pos & ~3u)), (comp << (sh & 31)) | carry, RLX_WG);
            carry = __builtin_amdgcn_ubfe(comp, 0u - sh, sh);
            pos += lut[i][1];
        }
        __hip_atomic_fetch_or((LAS u32*)(L + (pos & ~3u)), carry, RLX_WG);
    };
    auto expand = [&](u32 tab, u32 pos, u32 x) -> u32 {
        const u32 sel = L32(L_LUTD + x * 4);
        const LAS u32* p = (const LAS u32*)(L + (pos & ~3u));
        return perm(tab, __builtin_amdgcn_alignbyte(p[1], p[0], ALIGNMASK ? pos & 3 : pos), sel);
    };
    auto panel_of = [&](u32 id) -> u32 { return id >= 256 ? (id - 256) >> 1 : 32 + ((id >> 1) & 7) * 4 + (id >> 6); };
    auto stage_off = [&](u32 id, u32 s, u32 par) -> u32 {
        return ((id - 256) >> 3 == rank ? OFF_XO + (par * 4 + ((id >> 1) & 3)) * PSTR : OFF_SP + panel_of(id) * PSTR) + (s * 4 + (wave >> 1)) * 16384 + (id & 1) * 8192 + (wave & 1) * 1024 + lane * 16 + 4096;
    };
    auto arrive = [&](u32 c) -> u32 {
        u32 t = ~0u;
        asm volatile("s_mov_b64 exec, 1\n\tglobal_atomic_add %0, %1, %2, %3 sc0\n\ts_mov_b64 exec, -1"
                     : "+v"(t) : "v"(OFF_CT + c * 128), "v"(1u), "s"(myws) : "memory");
        return t;
    };
    auto raise = [&](u32 c, u32 t, u32 call) {
        const u32 per_call = c < 32 ? 112 : c < 64 ? 128 : c < 128 ? 32 : 16;
        asm volatile("s_waitcnt vmcnt(0)" : "+v"(t) :: "memory");
        if (t == per_call * call - 1) asm volatile("global_store_dword %0, %1, %2 sc1" :: "v"(OFF_PF + c * 128), "v"(call), "s"(myws) : "memory");
    };
    const u32 v2 = L_SCR + wave * WSCR, v3 = v2 + V2S, ev = v3 + V3S;
    auto pack_levels = [&](auto NP, u32 N2, u32 e, auto cls2, auto cls3) -> u32 {
        u32 u[2 * NP.value], m = 0;
        u32x2 l2[2 * NP.value];
#pragma unroll
        for (int i = 0; i < NP.value; ++i) {
            const u32x2 v = L64(v2 + lane * (8 * NP.value) + i * 8);
            u[2 * i] = v[0]; u[2 * i + 1] = v[1];
        }
#pragma unroll
        for (int i = 0; i < NP.value; ++i) {
            u32 q8 = cls2(u[2 * i]) | (cls2(u[2 * i + 1]) << 8);
            const int valid = (int)N2 - (int)lane * (8 * NP.value) - i * 8;
            q8 |= (0x5555u * 8) << (2 * min(max(valid, 0), 8));
            if (valid > 0) L16(e + lane * (2 * NP.value) + i * 2) = (unsigned short)(q8 >> 3);
            l2[2 * i] = L64(L_LUTC + (q8 & 0x7F8));
            l2[2 * i + 1] = L64(L_LUTC + ((q8 >> 8) & 0x7F8));
        }
#pragma unroll
        for (int i = 0; i < 2 * NP.value; ++i) m += l2[i][1];
        const u32 in2 = scan64(m);
        const u32 N3 = last(in2);
        if (N3 > CAP3) return ~0u;
        if (m) append(std::integral_constant<int, 2 * NP.value>{}, v3 + in2 - m, u, l2);
        const u32 off3 = ((N2 + 7) >> 3) * 2, off4 = off3 + ((N3 + 7) >> 3) * 2;
        const u32x2 v = L64(v3 + lane * 8);
        const u32 w[2] = {v[0], v[1]};
        u32 q8 = cls3(w[0]) | (cls3(w[1]) << 8);
        const int valid = (int)N3 - (int)lane * 8;
        q8 |= (0x5555u * 8) << (2 * min(max(valid, 0), 8));
        if (valid > 0) L16(e + off3 + lane * 2) = (unsigned short)(q8 >> 3);
        const u32x2 l3[2] = {L64(L_LUTC + (q8 & 0x7F8)), L64(L_LUTC + ((q8 >> 8) & 0x7F8))};
        const u32 m3 = l3[0][1] + l3[1][1];
        const u32 in3 = scan64(m3);
        if (m3) append(std::integral_constant<int, 2>{}, e + off4 + in3 - m3, w, l3);
        return off4 + last(in3);
    };
    auto unpack_levels = [&](auto NP, u32 N2, u32 sec, u32 t2, u32 t3) {
        u32 q[NP.value], e[NP.value], n = 0;
#pragma unroll
        for (int i = 0; i < NP.value; ++i) {
            q[i] = L16(sec + lane * (2 * NP.value) + i * 2) | 0x55550000u;
            if (lane * (8 * NP.value) + i * 8 >= N2) q[i] = 0x55555555u;
            e[i] = escbits(q[i]);
            n += __builtin_popcount(e[i]);
        }
        const u32 in2 = scan64(n), N3 = last(in2);
        const u32 off3 = sec + ((N2 + 7) >> 3) * 2, off4 = off3 + ((N3 + 7) >> 3) * 2;
        u32 q3 = L16(off3 + lane * 2) | 0x55550000u;
        if (lane * 8 >= N3) q3 = 0x55555555u;
        const u32 e3 = escbits(q3), m3 = __builtin_popcount(e3);
        const u32 p3 = off4 + scan64(m3) - m3;
        L64(v3 + lane * 8) = u32x2{expand(t3, p3, q3 & 0xFF), expand(t3, p3 + __builtin_popcount(e3 & 0xFF), (q3 >> 8) & 0xFF)};
        u32 pp = v3 + in2 - n;
        u32x2 z[NP.value];
#pragma unroll
        for (int i = 0; i < NP.value; ++i) {
            z[i] = u32x2{expand(t2, pp, q[i] & 0xFF), expand(t2, pp + __builtin_popcount(e[i] & 0xFF), (q[i] >> 8) & 0xFF)};
            pp += __builtin_popcount(e[i]);
        }
#pragma unroll
        for (int i = 0; i < NP.value; ++i) L64(v2 + lane * (8 * NP.value) + i * 8) = z[i];
    };
    u8* peer = a.ws[lane & 7];
    u32 npub = 0;

    lds_init();
    BAR();
    for (u32 job = 0; job < (bid >= 192 ? 2u : 1u); ++job) {
        const u32 id = (bid >= 192 && job == (bid >= 224 ? 0u : 1u)) ? xrun : tile;
        const u32 t = wave >> 1, h = wave & 1;
        if (id >= 256 && ((id - 256) >> 3) == rank) continue;
        if (job) {
            fetch_run(id);
            for (u32 o = lane * 16; o < WSCR; o += 1024) L128(v2 + o) = u32x4{0, 0, 0, 0};
        }
        if (id < 256) {
            u32 g[4][4];
#pragma unroll
            for (int i = 0; i < 4; ++i) {
                const u32 t0 = perm(f[i][1], f[i][0], 0x05010400u), t1 = perm(f[i][1], f[i][0], 0x07030602u);
                const u32 t2 = perm(f[i][3], f[i][2], 0x05010400u), t3 = perm(f[i][3], f[i][2], 0x07030602u);
                g[0][i] = perm(t2, t0, 0x05040100u); g[1][i] = perm(t2, t0, 0x07060302u);
                g[2][i] = perm(t3, t1, 0x05040100u); g[3][i] = perm(t3, t1, 0x07060302u);
            }
#pragma unroll
            for (int i = 0; i < 16; ++i) f[i >> 2][i & 3] = g[i >> 2][i & 3];
        }
        GET_CALL;
        asm volatile("" : "+v"(peer));
        const u32x4 f0 = {f[0][0], f[0][1], f[0][2], f[0][3]}, f1 = {f[1][0], f[1][1], f[1][2], f[1][3]};
        const u32x4 f2 = {f[2][0], f[2][1], f[2][2], f[2][3]}, f3 = {f[3][0], f[3][1], f[3][2], f[3][3]};
        auto keep = [&]() {
            if (id >= 256 && (((id >> 3) ^ rank ^ id) & 4)) return;
            const bool pan = id >= 256 || ((id >> 4) & 3) == blk(rank, id >> 6);
            const u32 vo = pan ? stage_off(id, rank, par) : OFF_LW + (id >> 1) * 65536 + t * 16384 + (id & 1) * 8192 + h * 1024 + lane * 16 + 4096;
            const u32 c = pan ? panel_of(id) : 128 + (id >> 1);
            ST16(vo, f0, myws, -4096); ST16(vo, f1, myws, -2048); ST16(vo, f2, myws, 0); ST16(vo, f3, myws, 2048);
            WAITV(0);
            raise(c, arrive(c), call);
        };
        u32 esum = 0;
#pragma unroll
        for (int i = 0; i < 16; ++i) esum = __builtin_amdgcn_udot4(f[i >> 2][i & 3] & 0x7C7C7C7Cu, 0x01010101u, esum, false);
        const u32 e0 = ((last(scan64(esum)) + 14746u) >> 14) & 31;
        const u32 runo = OFF_P + par * P_PAR + id * RUN;
        u32x4 Q, Mq;
        u32 S[2] = {0, 0};
        u32 e4[16];
        u32x2 lut[16];
        const u32 k1 = rep((33 - e0) & 31);
#pragma unroll
        for (int c = 0; c < 4; ++c) {
            u32 q = 0, m = 0;
#pragma unroll
            for (int i = 0; i < 4; ++i) {
                e4[4 * c + i] = f[c][i] & 0x7C7C7C7Cu;
                const u32 x8 = classify8(e4[4 * c + i], k1, 0u, T13);
                lut[4 * c + i] = L64(L_LUTC + x8);
                q |= i ? x8 << (8 * i - 3) : x8 >> 3;
                m |= (f[c][i] & 0x03030303u) << (2 * i);
                S[c >> 1] |= (f[c][i] & 0x80808080u) >> (4 * (c & 1) + i);
            }
            Q[c] = q; Mq[c] = m;
        }
        {
            const u32 vo = runo + VARCAP + wave * HALFB + lane * 16;
            ST16(vo, Q, myws, 0);
            ST16(vo, Mq, myws, 1024);
            const u32x4 s4 = {S[0], S[1], (u32)__shfl_xor((int)S[0], 32), (u32)__shfl_xor((int)S[1], 32)};
            if (lane < 32) ST16(vo, s4, myws, 2048);
        }
        u32 slot;
        bool to;
        if (id >= 256) {
            const u32 q = (id - 256) >> 3;
            slot = S_X + id - 256;
            to = lane < 8 && lane != rank && (lane == q ? (id & 7) >= 2 * NPL : !((q ^ lane ^ id) & 4));
        } else {
            slot = S_W + (id & 15) * 4 + (id >> 6);
            to = lane < 8 && lane != rank && ((id >> 4) & 3) == blk(lane, id >> 6) && (slot < WDIR || ((lane ^ rank) & 4));
        }
        if (id >= 256 ? (id & 6) == 0 : slot < 4) {
            WAITV(0);
            u32 arrived = 0;
            if (lane == 0) arrived = __hip_atomic_fetch_add((LAS u32*)(L + C_PUB1), 1u, RLX_WG);
            if (uni(arrived) == 7 && to)
                __hip_atomic_store((GAS u32*)(peer + OFF_F + (par * 8 + rank) * 4096 + (slot + S_EARLY) * 8), call, RLX_SYS);
            STAMP(3);
        }
        u32 n = 0;
#pragma unroll
        for (int c = 0; c < 4; ++c) n += __builtin_popcount(escbits(Q[c]));
        const u32 inc = scan64(n);
        const u32 N2 = last(inc);
        u32 len = 0;
        bool raw = N2 > CAP2 || FORCE_RAW & 1;
        if (!raw) {
            append(std::integral_constant<int, 16>{}, v2 + inc - n, e4, lut);
            const u32 k2 = rep((35 - e0) & 31), k3 = rep((38 - e0) & 31);
            len = pack_levels(std::integral_constant<int, 3>{}, N2, ev, [&](u32 u) { return classify8(u, k2, T2H, T2L); },
                              [&](u32 u) { return classify8(u, k3, 0u, T13); });
            raw = len > SECCAP;
        }
        if (raw) {
            len = 0;
            const u32 vo = OFF_RAW + par * RAW_PAR + (id * 8 + wave) * 4096 + lane * 16;
            ST16(vo, f0, myws, 0); ST16(vo, f1, myws, 1024); ST16(vo, f2, myws, 2048); ST16(vo, f3, myws, 3072);
        }
        const u32 len16 = (len + 15) >> 4;
        if (lane == 0) L32(C_HDR + wave * 4) = len16 | (raw ? 0x80000000u : (N2 << 9) | (e0 << 20));
        wbar();
        const u32 hv = lane < 8 ? L32(C_HDR + lane * 4) : 0u;
        const u32 hin = scan64(hv & 0x1FF);
        const u32 hoff = 2 + hin - (hv & 0x1FF);
        if (lane < len16) {
            const u32x4 sec = L128(ev + lane * 16);
            ST16(runo + __builtin_amdgcn_readlane(hoff, wave) * 16 + lane * 16, sec, myws, 0);
        }
        if (wave == 0 && lane < 8) ST4(runo + lane * 4, (hv & ~0x1FFu) | hoff, myws);
        WAITV(0);
        u32 arrived = 0;
        npub += 8;
        if (lane == 0) arrived = __hip_atomic_fetch_add((LAS u32*)(L + C_PUB), 1u, RLX_WG);
        if (uni(arrived) == npub - 1) {
            const u64 fl = ((u64)((2 + last(hin) + 7) >> 3) << 32) | call;
            if (to) __hip_atomic_store((GAS u64*)(peer + OFF_F + (par * 8 + rank) * 4096 + slot * 8), fl, RLX_SYS);
            STAMP(22 + job);
        }
        keep();
        if (bid >= 192 && job == 0) wbar();
    }

    GET_CALL;
    const float sc = a.sx[0] * a.sw[0];
    const u32 rg = wave >> 1, ch = wave & 1;
    auto out_store = [&](u32 e, u32 r, f32x4 v) {
        asm volatile("global_store_dwordx4 %0, %1, %2 sc1\n\ts_nop 1"
                     :: "v"((lane >> 4) * (16 * GN) + (lane & 15) * 16), "v"(v), "s"(a.out + e + r * GN) : "memory");
    };

    if (is_link) {
        auto landed = [&](u32 mark) {
            while (LPOLL(C_LANDED) < mark) __builtin_amdgcn_s_sleep(1);
            CFENCE();
        };
        auto done = [&](u32 m) {
            WAITL();
            asm volatile("s_mov_b64 exec, 1\n\tds_add_u32 %0, %1\n\ts_mov_b64 exec, -1" :: "v"(lbase + C_DONE + (m & 3) * 4), "v"(1u) : "memory");
        };
        u32 m = 0, sl, lines, pbase = 8 * HALFB / 1024;
        u32 owed = 255;
        auto settle = [&]() {
            if (owed != 255) { WAITV(0); raise(owed, arrive(owed), call); owed = 255; }
        };
        auto next = [&]() {
            u32 rdy = LPOLL(C_RDY + (m & 7) * 8);
            if ((rdy >> 16) != m + 1) {
                settle();
                while (((rdy = LPOLL(C_RDY + (m & 7) * 8)) >> 16) != m + 1) __builtin_amdgcn_s_sleep(1);
            }
            sl = (rdy >> 6) & 511;
            lines = rdy & 63;
            if (sl == S_END) return;
            pbase += (lines + 7) >> 3;
            if (LPOLL(C_LANDED) < pbase) settle();
            landed(pbase);
        };
        for (;; ++m) {
            next();
            if (sl >= S_O) break;
            const u32 id = run_id(sl), s = run_src(sl), vb = L_BIG + (m & 3) * RUN;
            const u8* base = a.ws[s];
            u32 tick = 0, tickc = 255;
            const bool relay = sl >= WDIR && sl < S_X && (sl & 3) != pband;
            const u32 rsl = (from & 3) * 8 + (sl >> 2) - (16 - NRL);
            auto pass_on = [&]() {
                const u32 ro = OFF_P + par * P_PAR + (320 + (sl & 3) * 32 + rsl) * RUN;
                const u32 o0 = wave * 640 + lane * 16, o1 = VARCAP + wave * HALFB + lane * 16;
                if (lane < 40 && o0 < lines * 128) { const u32x4 v = L128(vb + o0); ST16(ro + o0, v, myws, 0); }
                const u32x4 v0 = L128(vb + o1), v1 = L128(vb + o1 + 1024);
                ST16(ro + o1, v0, myws, 0); ST16(ro + o1, v1, myws, 1024);
                if (lane < 32) { const u32x4 v = L128(vb + o1 + 2048); ST16(ro + o1, v, myws, 2048); }
            };
            const u32 hdr = uni(L32(vb + wave * 4));
            u32x4 r[4];
            if (hdr >> 31) {
                const u32 vo = OFF_RAW + par * RAW_PAR + (id * 8 + wave) * 4096 + lane * 16;
                asm volatile("global_load_dwordx4 %0, %4, %5 sc0 sc1\n\tglobal_load_dwordx4 %1, %4, %5 offset:1024 sc0 sc1\n\t"
                             "global_load_dwordx4 %2, %4, %5 offset:2048 sc0 sc1\n\tglobal_load_dwordx4 %3, %4, %5 offset:3072 sc0 sc1\n\t"
                             "s_waitcnt vmcnt(0)"
                             : "=&v"(r[0]), "=&v"(r[1]), "=&v"(r[2]), "=&v"(r[3]) : "v"(vo), "s"(base) : "memory");
            } else {
                const u32 e0 = (hdr >> 20) & 31, N2 = (hdr >> 9) & 0x7FF, sec = vb + (hdr & 0x1FF) * 16;
                const u32 t1 = (((e0 - 1) & 31) << 10) | (e0 << 18) | (((e0 + 1) & 31) << 26);
                const u32 t2 = (((e0 - 2) & 31) << 10) | (((e0 + 2) & 31) << 18) | (((e0 - 3) & 31) << 26);
                const u32 t3 = (((e0 - 6) & 31) << 10) | (((e0 - 5) & 31) << 18) | (((e0 - 4) & 31) << 26);
                unpack_levels(std::integral_constant<int, 3>{}, N2, sec, t2, t3);
                const u32 pneed = m ? pbase + (((wave + 1) * HALFB + 1023) >> 10) : 0;
                if (LPOLL(C_LANDED) < pneed) settle();
                if (owed != 255) { WAITV(0); tick = arrive(owed); tickc = owed; owed = 255; }
                landed(pneed);
                if (relay) pass_on();
                const u32 fx = vb + VARCAP + wave * HALFB;
                const u32x4 Q = L128(fx + lane * 16), Mq = L128(fx + 1024 + lane * 16);
                const u32x2 S = L64(fx + 2048 + (lane & 31) * 16 + (lane >> 5) * 8);
                u32 E[4], n = 0;
#pragma unroll
                for (int c = 0; c < 4; ++c) { E[c] = escbits(Q[c]); n += __builtin_popcount(E[c]); }
                u32 pp = v2 + scan64(n) - n;
#pragma unroll
                for (int c = 0; c < 4; ++c)
#pragma unroll
                    for (int i = 0; i < 4; ++i) {
                        r[c][i] = expand(t1, pp, (Q[c] >> (8 * i)) & 0xFF) | ((Mq[c] >> (2 * i)) & 0x03030303u)
                                  | ((S[c >> 1] << (4 * (c & 1) + i)) & 0x80808080u);
                        pp += __builtin_popcount(E[c] & (0xFFu << (8 * i)));
                    }
            }
            if (relay) {
                if (hdr >> 31) pass_on();
                WAITV(0);
                u32 arrived = 0;
                if (lane == 0) arrived = __hip_atomic_fetch_add((LAS u32*)(L + C_RL + (m & 3) * 4), 1u, RLX_WG);
                if ((uni(arrived) & 7) == 7 && lane == 0)
                    __hip_atomic_store((GAS u64*)(a.ws[partner(rank, sl & 3)] + OFF_F + (par * 8 + rank) * 4096 + (S_RL + rsl) * 8), ((u64)lines << 32) | call, RLX_SYS);
            }
            done(m);
            settle();
            if (tickc != 255) raise(tickc, tick, call);
            const u32 vo = stage_off(id, s, par);
            ST16(vo, r[0], myws, -4096); ST16(vo, r[1], myws, -2048); ST16(vo, r[2], myws, 0); ST16(vo, r[3], myws, 2048);
            owed = panel_of(id);
            if (m) pbase += 8 * HALFB / 1024;
        }
        settle();
        const u32 u = wave >> 1, rb = wave & 1;
        const u8* base = a.ws[from];
        while (sl != S_END) {
            const u32 id = sl - S_O, vb = L_BIG + (m & 3) * RUN, band = (id >> 2) & 3;
            const u32 e0 = ((cross ? 256 : 0) + (id & 3) * 64 + (u >> 1) * 32 + rb * 16) * GN
                           + (band * 32 + blk(from, band) * 8 + (id >> 4)) * 128 + (u & 1) * 64;
            f32x4 P[4];
#pragma unroll
            for (int k = 0; k < 4; ++k)
                asm volatile("global_load_dwordx4 %0, %1, %2 sc0 sc1"
                             : "=v"(P[k]) : "v"((lane >> 4) * (16 * GN) + (lane & 15) * 16), "s"(a.out + e0 + k * GN) : "memory");
            landed(pbase);
            const u32 hdr = uni(L32(vb + u * 4));
            const u32 n2 = hdr >> 31 ? 0 : (hdr >> 19) & 15;
            u32x2 hard = {(rb ^ 1) << 4, 0};
            u32 he = 0;
            float hp = 0.f;
            f32x4 r[4];
            if (hdr >> 31) {
                done(m);
                const u32 vo = OFF_OR + par * OR_PAR + ((rank * 128 + id) * 8 + wave) * 4096 + lane * 16;
                asm volatile("global_load_dwordx4 %0, %4, %5 sc0 sc1\n\tglobal_load_dwordx4 %1, %4, %5 offset:1024 sc0 sc1\n\t"
                             "global_load_dwordx4 %2, %4, %5 offset:2048 sc0 sc1\n\tglobal_load_dwordx4 %3, %4, %5 offset:3072 sc0 sc1\n\t"
                             "s_waitcnt vmcnt(0)"
                             : "=&v"(r[0]), "=&v"(r[1]), "=&v"(r[2]), "=&v"(r[3]) : "v"(vo), "s"(base) : "memory");
            } else {
                const u32 sec = vb + (hdr & 0xFF) * 16, N2 = (hdr >> 8) & 0x7FF;
                if (lane < n2) hard = L64(sec + lane * 8);
                if (n2) {
                    he = ((((hard[0] >> 9) & 3) * 4 + ((hard[0] >> 2) & 3)) * GN + ((hard[0] >> 5) & 15) * 4 + (hard[0] & 3)) * 4;
                    asm volatile("global_load_dword %0, %1, %2 sc0 sc1\n\ts_waitcnt vmcnt(0)" : "=&v"(hp) : "v"(he), "s"(a.out + e0) : "memory");
                }
                unpack_levels(std::integral_constant<int, 2>{}, N2, sec + n2 * 8, 0x06030100u, 0x08070500u);
                landed(pbase + (((u + 1) * HEADB + 1023) >> 10));
                const u32 hd = vb + EXCCAP + u * HEADB;
                const u32x2 Q = L64(hd + (lane & 31) * 16 + (lane >> 5) * 8), Nb = L64(hd + 512 + lane * 16 + rb * 8);
                const u32 c0 = __builtin_popcount(escbits(Q[0])), c1 = c0 + __builtin_popcount(escbits(Q[1]));
                const u32 q = rb ? Q[1] : Q[0], e = escbits(q);
                u32 pp = v2 + scan64(c1) - c1 + (rb ? c0 : 0);
                u32 ft[8];
#pragma unroll
                for (int g = 0; g < 4; ++g) {
                    const u32 sy = expand(0x04020000u, pp, (q >> (8 * g)) & 0xFF);
                    pp += __builtin_popcount(e & (0xFFu << (8 * g)));
                    const u32 nb = Nb[g >> 1] >> (4 * (g & 1));
                    const u32 T = (((sy << 2) & 0x78787878u) | (nb & 0x07070707u)) ^ ones(nb & 0x08080808u);
                    const u32 F = ((sy << 7) & 0x80808080u) | ((sy >> 1) & 0x70707070u);
                    ft[2 * g] = perm(T, F, 0x05010400u); ft[2 * g + 1] = perm(T, F, 0x07030602u);
                }
                asm volatile("" : "+v"(ft[0]), "+v"(ft[1]), "+v"(ft[2]), "+v"(ft[3]), "+v"(ft[4]), "+v"(ft[5]), "+v"(ft[6]), "+v"(ft[7]));
                landed(pbase + 4 * HEADB / 1024 + (wave + 1) * 2);
                const u32x4 d0 = L128(vb + EXCCAP + 4 * HEADB + wave * 2048 + lane * 16), d1 = L128(vb + EXCCAP + 4 * HEADB + wave * 2048 + 1024 + lane * 16);
                done(m);
                const float dn = __uint_as_float(hdr & 0x7F800000u);
#pragma unroll
                for (int g = 0; g < 4; ++g) {
                    const u32 m0 = g < 2 ? d0[2 * g] : d1[2 * g - 4], m1 = g < 2 ? d0[2 * g + 1] : d1[2 * g - 3];
                    const int k0 = perm(m0, ft[2 * g], 0x01050400u), k1 = perm(m0, ft[2 * g], 0x03070602u);
                    const int k2 = perm(m1, ft[2 * g + 1], 0x01050400u), k3 = perm(m1, ft[2 * g + 1], 0x03070602u);
                    r[g] = f32x4{(float)k0, (float)k1, (float)k2, (float)k3} * dn;
                }
            }
            asm volatile("s_waitcnt vmcnt(0)" : "+v"(P[0]), "+v"(P[1]), "+v"(P[2]), "+v"(P[3]) :: "memory");
#pragma unroll
            for (int g = 0; g < 4; ++g)
                out_store(e0, g, f32x4{silu(r[g][0] + P[g][0], sc), silu(r[g][1] + P[g][1], sc), silu(r[g][2] + P[g][2], sc), silu(r[g][3] + P[g][3], sc)});
            if (n2) {
                WAITV(0);
                if (((hard[0] >> 4) & 1) == rb)
                    asm volatile("global_store_dword %0, %1, %2 sc1" :: "v"(he), "v"(silu(__uint_as_float(hard[1]) + hp, sc)), "s"(a.out + e0) : "memory");
            }
            ++m;
            pbase += (OHALF - EXCCAP) / 1024;
            u32 rdy;
            while (((rdy = LPOLL(C_RDY + (m & 7) * 8)) >> 16) != m + 1) __builtin_amdgcn_s_sleep(1);
            sl = (rdy >> 6) & 511;
            pbase += ((rdy & 63) + 7) >> 3;
        }
        if (LPOLL(C_DEAD)) {
            WAITV(0);
            for (u32 r = wave * 64 + lane; r < MS; r += 512)
                for (u32 c = 0; c < GN; c += 64) a.out[(size_t)r * GN + c] = __builtin_nanf("");
        }
    } else {
        const u32 npart = li < 2 ? 24 : 56, njob = npart + 48;
        for (u32 job = (bid >> 3) - 7, round = 0; job < njob + 128; round += job >= npart) {
            FRESH_LANE();
            const bool part = job < npart, own = part || job >= njob;
            const u32 n = part ? job + (li < 2 ? 24 * li : 56 * li - 64) : own ? job - njob : (job - npart) % 24;
            const u32 band = part ? n / 96 : n & 3;
            u32 q = rank, layer = part ? n & 3 : (n >> 2) & 3, block = blk(rank, band), ti = part ? (n >> 2) & 7 : n >> 4;
            if (part) {
                block = (block + 1 + (n >> 5) % 3) & 3;
            } else if (!own) {
                const u32 bat = ((job < npart + 24 ? 0x76531420u : 0xFEDCBA98u) >> (4 * li)) & 15, j = (n >> 2) % 3;
                q = bat & 1 ? (~rank & 4) | ((partner(rank, band) + 1 + j) & 3) : (rank & 4) | ((rank + 1 + j) & 3);
                layer = (bat & 1) * 2 + (n >= 12);
                ti = bat >> 1;
            }
            const u32 ct = ti * 4 + band, gct = band * 32 + block * 8 + ti;
            u32 nk = part ? 4 : own ? 32 : 28;
            asm volatile("" : "+s"(nk));
            const u32 skip = own ? 32 : 4 * q;
            if (wave == 0) {
                const u32 sl = part ? (lane & 32) + rank * 4 + (lane & 3) : lane;
                const u32 fo = own && (layer < NPL || ((sl >> 2) & 7) == rank) ? OFF_XF + par * 1024 + layer * 256 + sl * 4
                                                                                   : OFF_PF + (q * 4 + layer) * 128;
                bool ok = !LPOLL(C_DEAD);
                while (ok) {
                    const u32 fa = __hip_atomic_load((GAS u32*)(myws + fo), RLX_SYS);
                    const u32 fb = __hip_atomic_load((GAS u32*)(myws + OFF_PF + (part ? 128 + gct : 32 + ct) * 128), RLX_SYS);
                    if (__all(fa >= call && fb >= call)) break;
                    if (expired(call)) { give_up(call); ok = false; }
                    __builtin_amdgcn_s_sleep(GSLEEP);
                }
            }
            BAR();
            const bool dead = LPOLL(C_DEAD);
            if (wave == 0) STAMP(44 + min(round, 3u) * 3);
            f32x4 acc[2][4];
#pragma unroll
            for (int i = 0; i < 8; ++i) acc[i >> 2][i & 3] = f32x4{0.f, 0.f, 0.f, 0.f};
            for (u32 o = lane * 16; o < WSCR; o += 1024) L128(v2 + o) = u32x4{0, 0, 0, 0};
            if (!dead) {
                const u32 ga = (own ? OFF_XO + (par * 4 + layer) * PSTR : OFF_SP + (q * 4 + layer) * PSTR) + (part ? rank * 65536 : 0) + wave * 2048 + lane * 16;
                const u32 gb = (part ? OFF_LW + gct * 65536 : OFF_SP + (32 + ct) * PSTR) + wave * 2048 + lane * 16;
                auto dma = [&](u32 k, int j) {
                    const u32 l = lbase + L_BIG + (k & 3) * 32768 + wave * 2048 + (j >> 1) * 16384 + (j & 1) * 1024;
                    DMA16((j >> 1 ? gb : ga) + (k < skip ? k : k + 4) * 16384 + (j & 1) * 1024, l, myws);
                };
                auto stage = [&](u32 k) { dma(k, 0); dma(k, 1); dma(k, 2); dma(k, 3); };
                u32x4 fr[2][12];
                auto step = [&](auto B, auto MM, auto RD, auto ST, u32 k) {
                    const u32 sa = L_BIG + (k & 3) * 32768 + rg * 4096 + lane * 16, sb = L_BIG + (k & 3) * 32768 + 16384 + ch * 8192 + lane * 16;
#pragma unroll
                    for (int i = 0; i < 8; ++i) {
                        if (MM.value) {
                            const u32x4 *pa = &fr[B.value ^ 1][(i & 1) * 2], *pb = &fr[B.value ^ 1][4 + (i >> 1) * 2];
                            const i32x8 af = {(int)pa[0][0], (int)pa[0][1], (int)pa[0][2], (int)pa[0][3], (int)pa[1][0], (int)pa[1][1], (int)pa[1][2], (int)pa[1][3]};
                            const i32x8 bf = {(int)pb[0][0], (int)pb[0][1], (int)pb[0][2], (int)pb[0][3], (int)pb[1][0], (int)pb[1][1], (int)pb[1][2], (int)pb[1][3]};
                            acc[i & 1][i >> 1] = __builtin_amdgcn_mfma_scale_f32_16x16x128_f8f6f4(af, bf, acc[i & 1][i >> 1], 1, 1, 0, 0x7F7F7F7F, 0, 0x7F7F7F7F);
                        }
                        __builtin_amdgcn_sched_barrier(0);
                        if (RD.value && i < 6) {
                            const u32 o = i < 2 ? sa + i * 2048 : sb + (i - 2) * 2048;
                            fr[B.value][2 * i] = L128(o);
                            fr[B.value][2 * i + 1] = L128(o + 1024);
                        }
                        if (ST.value && (i & 1)) dma(k + 3, i >> 1);
                        __builtin_amdgcn_sched_barrier(0);
                    }
                };
                constexpr std::integral_constant<int, 0> N{};
                constexpr std::integral_constant<int, 1> Y{};
                stage(0); stage(1); stage(2);
                WAITV(8); BAR();
                step(N, N, Y, Y, 0);
                for (u32 k = 1; k < nk - 3; k += 2) {
                    WAITV(8); BAR();
                    step(Y, Y, Y, Y, k);
                    WAITV(8); BAR();
                    step(N, Y, Y, Y, k + 1);
                }
                WAITV(8); BAR();
                step(Y, Y, Y, N, 1);
                WAITV(4); BAR();
                step(N, Y, Y, N, 2);
                WAITV(0); BAR();
                step(Y, Y, Y, N, 3);
                step(N, Y, N, N, 0);
            }
            FRESH_LANE();
            u32 ticket = 0;
            if (threadIdx.x == 0) ticket = __hip_atomic_fetch_add((GAS u32*)(myws + OFF_Q + (job < njob ? li : 15) * 128), 1u, RLX_AG);
            if (wave == 0) STAMP(45 + min(round, 3u) * 3);
            const u32 eo = (layer * 128 + rg * 32) * GN + gct * 128 + ch * 64;
#define ROW(g) (f32x4{acc[g >> 2][0][g & 3], acc[g >> 2][1][g & 3], acc[g >> 2][2][g & 3], acc[g >> 2][3][g & 3]})
            if (part) {
                if (!dead) {
#pragma unroll
                    for (int g = 0; g < 8; ++g) out_store(eo, (g >> 2) * 16 + (g & 3), ROW(g));
                    WAITV(0);
                    BAR();
                    if (wave == 1) raise(64 + (gct >> 3), arrive(64 + (gct >> 3)), call);
                }
            } else if (own) {
                const bool fail = dead || uni(__hip_atomic_load((GAS u32*)(myws + OFF_TMO), RLX_AG)) == call;
                const float s2 = fail ? __builtin_nanf("") : sc;
#pragma unroll
                for (int g = 0; g < 8; ++g)
                    out_store(eo, (g >> 2) * 16 + (g & 3), f32x4{silu(acc[g >> 2][0][g & 3], s2), silu(acc[g >> 2][1][g & 3], s2),
                                          silu(acc[g >> 2][2][g & 3], s2), silu(acc[g >> 2][3][g & 3], s2)});
                WAITV(0);
            } else if (!dead) {
                const u32 id = ct * 4 + (layer & 1) * 2 + (wave >> 2), to = OFF_OP + par * OP_PAR + (q * 128 + id) * OHALF;
                u32 esum = 0;
#pragma unroll
                for (int i = 0; i < 8; ++i) esum += (__float_as_uint(acc[i >> 2][i & 3][(i + (i >> 2)) & 3]) >> 19) & 0xFFF;
                const u32 etop = min(max((last(scan64(esum)) + 37683u) >> 13, 40u), 250u);
                const float up = __uint_as_float((285 - etop) << 23), dn = __uint_as_float((etop - 31) << 23);
                u32 d[16], sy[8], Nb[4] = {0, 0, 0, 0}, Q[2] = {0, 0}, hardm = 0, off = 0;
                u32x2 lut[8];
                const u32 vm = to + EXCCAP + 4 * HEADB + (wave & 3) * 4096 + lane * 16;
#pragma unroll
                for (int g = 0; g < 8; ++g) {
                    u32 k[4];
#pragma unroll
                    for (int cc = 0; cc < 4; ++cc) {
                        const float v = acc[g >> 2][cc][g & 3];
                        k[cc] = (u32)cvt_i32(v * up);
                        off |= __float_as_uint((float)(int)k[cc] * dn) ^ __float_as_uint(v);
                    }
                    d[2 * g] = perm(k[1], k[0], 0x06050201u); d[2 * g + 1] = perm(k[3], k[2], 0x06050201u);
                    const u32 a01 = perm(k[1], k[0], 0x07030400u), a23 = perm(k[3], k[2], 0x07030400u);
                    const u32 F = perm(a23, a01, 0x05040100u), T = perm(a23, a01, 0x07060302u);
                    off |= F & 0x0F0F0F0Fu;
                    const u32 sg = (T >> 4) & 0x08080808u, Tm = T ^ ones(sg);
                    sy[g] = ((Tm >> 2) & 0x1E1E1E1Eu) | ((F >> 7) & 0x01010101u) | ((F << 1) & 0xE0E0E0E0u);
                    Nb[g >> 1] |= ((Tm & 0x07070707u) | sg) << (4 * (g & 1));
                    const u32 x8 = lookup8(sy[g], 0x00000018u, 0x00100008u);
                    lut[g] = L64(L_LUTC + x8);
                    Q[g >> 2] |= g & 3 ? x8 << (8 * (g & 3) - 3) : x8 >> 3;
                    if (g & 1) {
                        ST16(vm + (g >> 1) * 1024, (u32x4{d[2 * g - 2], d[2 * g - 1], d[2 * g], d[2 * g + 1]}), myws, 0);
                        asm volatile("" : "+v"(off), "+v"(sy[g - 1]), "+v"(sy[g]), "+v"(Nb[g >> 1]), "+v"(Q[g >> 2]));
                        __builtin_amdgcn_sched_barrier(0);
                    }
                }
                if (__any(off != 0)) {
                    float up2 = up;
                    asm volatile("" : "+v"(up2));
#pragma unroll
                    for (int i = 0; i < 32; ++i) {
                        const float v = acc[i >> 4][i & 3][(i >> 2) & 3];
                        hardm |= (u32)(__float_as_uint((float)(cvt_i32(v * up2) & ~15) * dn) != __float_as_uint(v)) << i;
                    }
                }
                {
                    const u32 vh = to + EXCCAP + (wave & 3) * HEADB + lane * 16;
                    ST16(vh, (u32x4{Nb[0], Nb[1], Nb[2], Nb[3]}), myws, 512);
                    const u32x4 q4 = {Q[0], Q[1], (u32)__shfl_xor((int)Q[0], 32), (u32)__shfl_xor((int)Q[1], 32)};
                    if (lane < 32) ST16(vh, q4, myws, 0);
                }
                const u32 cnt = (__builtin_popcount(escbits(Q[0])) + __builtin_popcount(escbits(Q[1]))) | (__builtin_popcount(hardm) << 16);
                const u32 inc = scan64(cnt), tot = last(inc);
                const u32 N2 = tot & 0xFFFF, n2 = tot >> 16;
                u32 len = ~0u;
                if (N2 <= OCAP2 && n2 <= HARDCAP) {
                    append(std::integral_constant<int, 8>{}, v2 + ((inc - cnt) & 0xFFFF), sy, lut);
                    len = pack_levels(std::integral_constant<int, 2>{}, N2, ev + n2 * 8, [&](u32 u) { return lookup8(u, 0x00180000u, 0x10000800u); },
                                      [&](u32 u) { return lookup8(u - 0x01010101u, 0x18100008u, 0u); });
                    if (len != ~0u) len += n2 * 8;
                    if (n2) {
                        const u32 dump = L_BIG + wave * 8192;
#pragma unroll
                        for (int g = 0; g < 8; ++g) *(LAS f32x4*)(L + dump + g * 1024 + lane * 16) = ROW(g);
                        u32 o = ev + ((inc - cnt) >> 16) * 8;
                        while (hardm) {
                            const u32 b = __builtin_ctz(hardm);
                            hardm &= hardm - 1;
                            L64(o) = u32x2{lane * 32 + b, L32(dump + (b >> 2) * 1024 + lane * 16 + (b & 3) * 4)};
                            o += 8;
                        }
                    }
                }
                const bool raw = len > OSECCAP || FORCE_RAW & 2;
                if (raw) {
                    len = 0;
                    const u32 vo = OFF_OR + par * OR_PAR + ((q * 128 + id) * 4 + (wave & 3)) * 8192 + lane * 16 + 4096;
                    ST16(vo, ROW(0), myws, -4096); ST16(vo, ROW(1), myws, -3072); ST16(vo, ROW(2), myws, -2048); ST16(vo, ROW(3), myws, -1024);
                    ST16(vo, ROW(4), myws, 0);     ST16(vo, ROW(5), myws, 1024);  ST16(vo, ROW(6), myws, 2048);  ST16(vo, ROW(7), myws, 3072);
                }
                const u32 len16 = (len + 15) >> 4;
                if (lane == 0) L32(C_HDR + wave * 4) = len16 | (raw ? 0x80000000u : (N2 << 8) | (n2 << 19) | ((etop - 31) << 23));
                BAR();
                const u32 hv = lane < 4 ? L32(C_HDR + (wave & 4) * 4 + lane * 4) : 0u;
                const u32 hin = scan64(hv & 0xFF);
                const u32 hoff = 1 + hin - (hv & 0xFF);
                if (lane < len16) {
                    const u32x4 sec = L128(ev + lane * 16);
                    ST16(to + __builtin_amdgcn_readlane(hoff, wave & 3) * 16 + lane * 16, sec, myws, 0);
                }
                if ((wave & 3) == 0 && lane < 4) ST4(to + lane * 4, (hv & ~0xFFu) | hoff, myws);
                WAITV(0);
                BAR();
                if ((wave & 3) == 1 && lane == 0) {
                    const u64 fl = ((u64)((1 + last(hin) + 7) >> 3) << 32) | call;
                    __hip_atomic_store((GAS u64*)(a.ws[q] + OFF_F + (par * 8 + rank) * 4096 + (S_O + id) * 8), fl, RLX_SYS);
                }
            }
            if (threadIdx.x == 0) {
                u32 nx = njob + ticket - 328 * (call - 1);
                if (job < njob) {
                    nx = 25 + ticket - njob * (call - 1);
                    if (nx >= njob) nx = njob + __hip_atomic_fetch_add((GAS u32*)(myws + OFF_Q + 15 * 128), 1u, RLX_AG) - 328 * (call - 1);
                }
                L32(C_NEXT) = nx;
            }
            BAR();
            job = LPOLL(C_NEXT);
            if (wave == 0) STAMP(46 + min(round, 3u) * 3);
        }
    }
    if (threadIdx.x == 0) __hip_atomic_store((GAS u32*)(myws + OFF_CNT + bid * 128), call, RLX_AG);
}

extern "C" void kernel_launch(int rank, int nranks, ncclComm_t comm,
                              hipStream_t stream,
                              void* const* d_in, const int* in_sizes, int n_in,
                              void* d_out, int out_size,
                              const int* in_dtypes, int out_dtype,
                              void* const* d_out_all,
                              void* const* d_ws_all, size_t ws_bytes) {
    if (nranks != NR || (unsigned)rank >= (unsigned)NR || ws_bytes < WS_NEED || n_in != 4
        || in_sizes[0] != 4096 * KS || in_sizes[1] != KS * GN || out_size != MS * GN) {
        fprintf(stderr, "unexpected problem shape\n");
        return;
    }
    Args a;
    a.x = (const u8*)d_in[0];
    a.w = (const u8*)d_in[1];
    a.sx = (const float*)d_in[2];
    a.sw = (const float*)d_in[3];
    a.out = (float*)d_out;
    for (int p = 0; p < NR; ++p) a.ws[p] = (u8*)d_ws_all[p];
    a.self = a.ws[rank];
    a.rank = rank;
    a.pad = 0;
    fused_gemm_rs<<<NWG, NTHR, 0, stream>>>(a);
    HIP_CHECK(hipGetLastError());
}
```

```cpp
#include <hip/hip_runtime.h>
#include <rccl/rccl.h>
#include <cstdio>
#include <cstdint>
#include <type_traits>

#define HIP_CHECK(expr) do { hipError_t _e = (expr); if (_e != hipSuccess) { \
    fprintf(stderr, "HIP error %s at %s:%d\n", hipGetErrorString(_e), __FILE__, __LINE__); \
    return; } } while (0)

typedef unsigned u32;
typedef unsigned long long u64;
typedef unsigned char u8;
typedef u32 u32x2 __attribute__((ext_vector_type(2)));
typedef u32 u32x4 __attribute__((ext_vector_type(4)));
typedef int i32x8 __attribute__((ext_vector_type(8)));
typedef float f32x4 __attribute__((ext_vector_type(4)));
#define GAS __attribute__((address_space(1)))
#define LAS __attribute__((address_space(3)))

#define FORCE_RAW 0
#define STAMPS 0

constexpr int WINDOW = 6;
constexpr int XWIN = 4;
constexpr u32 NPL = 2;
constexpr u32 NRL = 5;
constexpr int NR = 8;
constexpr int GN = 16384;
constexpr int KS = 512;
constexpr int MS = 512;
constexpr int NWG = 256;
constexpr int NTHR = 704;
constexpr u32 NOWN = 8 - 2 * NPL, WDIR = 64 - 4 * NRL;
constexpr u32 S_W = 0, S_X = 64, S_RL = 128, S_O = 256, S_EARLY = 384, S_END = 511;

constexpr u32 VARCAP = 5120;
constexpr u32 HALFB  = 2560;
constexpr u32 RUN    = VARCAP + 8 * HALFB;
constexpr u32 CAP2 = 1536, CAP3 = 512, SECCAP = 624;
static_assert(32 + 8 * SECCAP <= VARCAP, "sections fit");
constexpr u32 EXCCAP = 3072;
constexpr u32 HEADB  = 1536;
constexpr u32 OHALF  = EXCCAP + 4 * HEADB + 8 * 2048;
constexpr u32 OCAP2 = 1024, OSECCAP = 752, HARDCAP = 15;
static_assert(16 + 4 * OSECCAP <= EXCCAP && OHALF == RUN, "sections fit");

constexpr u32 OFF_CNT = 0;
constexpr u32 OFF_TMO = 32u << 10;
constexpr u32 OFF_Q   = 40u << 10;
constexpr u32 OFF_XF  = 48u << 10;
constexpr u32 OFF_CT  = 64u << 10;
constexpr u32 OFF_PF  = 96u << 10;
constexpr u32 OFF_F   = 204u << 20;
constexpr u32 OFF_DBG = 256u << 10;
constexpr u32 PSTR    = 32 * 16384 + 4352;
constexpr u32 OFF_XO  = 1u << 20;
constexpr u32 OFF_SP  = 6u << 20;
constexpr u32 OFF_P   = 40u << 20;
constexpr u32 P_PAR   = 11u << 20;
constexpr u32 OFF_RAW = 62u << 20;
constexpr u32 RAW_PAR = 10u << 20;
constexpr u32 OFF_OP  = 82u << 20;
constexpr u32 OP_PAR  = 25u << 20;
constexpr u32 OFF_OR  = 132u << 20;
constexpr u32 OR_PAR  = 32u << 20;
constexpr u32 OFF_LW  = 196u << 20;
constexpr u32 WS_NEED = OFF_F + (1u << 20);
static_assert(OFF_XO + 8 * PSTR <= OFF_SP && OFF_SP + 64 * PSTR <= OFF_P && 448 * RUN <= P_PAR
              && 320 * 32768 <= RAW_PAR && 1024 * OHALF <= OP_PAR && NRL <= 8, "workspace");
constexpr u64 DEADLINE_TICKS = 50000000ull;

constexpr u32 L_LUTD = 0;
constexpr u32 L_LUTC = 1024;
constexpr u32 L_BIG  = 3072;
constexpr u32 L_ZERO = L_BIG + 131072;
constexpr u32 V2S = CAP2 + 16, V3S = CAP3 + 16, EVS = 1040;
constexpr u32 WSCR   = V2S + V3S + EVS;
constexpr u32 L_SCR  = L_ZERO;
constexpr u32 L_CTL  = L_SCR + 8 * WSCR;
constexpr u32 C_WB = L_CTL, C_LANDED = L_CTL + 4, C_DEAD = L_CTL + 8, C_PUB = L_CTL + 12, C_DONE = L_CTL + 16;
constexpr u32 C_HDR = L_CTL + 32, C_START = L_CTL + 64, C_NEXT = L_CTL + 68, C_PUB1 = L_CTL + 72, C_VPOS = L_CTL + 76, C_RL = L_CTL + 80;
constexpr u32 C_RDY = L_CTL + 96;
constexpr u32 L_END  = L_CTL + 192;
static_assert(L_BIG + 4 * RUN <= L_ZERO && WSCR % 16 == 0 && L_END % 16 == 0 && L_END <= 160 * 1024, "LDS");
static_assert(CAP2 / 4 + CAP3 / 4 + CAP3 + 16 <= EVS && 8 * HARDCAP + OCAP2 / 4 + CAP3 / 4 + CAP3 + 16 <= EVS, "section scratch");

struct Args {
    const u8* x;
    const u8* w;
    const float* sx;
    const float* sw;
    float* out;
    u8* self;
    u8* ws[NR];
    int rank;
    int pad;
};

#define ST16(voff, data, sbase, imm) \
    asm volatile("global_store_dwordx4 %0, %1, %2 offset:" #imm " sc0 sc1\n\ts_nop 1" \
                 :: "v"(voff), "v"(data), "s"(sbase) : "memory")
#define ST4(voff, data, sbase) \
    asm volatile("global_store_dword %0, %1, %2 sc0 sc1" :: "v"(voff), "v"(data), "s"(sbase) : "memory")
#define DMA16(voff, ldsaddr, sbase) do { u32 keep_; \
    asm volatile("s_mov_b32 %0, m0\n\ts_mov_b32 m0, %2\n\ts_nop 0\n\t" \
                 "global_load_lds_dwordx4 %1, %3 sc0 sc1\n\ts_mov_b32 m0, %0" \
                 : "=&s"(keep_) : "v"(voff), "s"(ldsaddr), "s"(sbase) : "memory"); } while (0)
#define WAITV(n) asm volatile("s_waitcnt vmcnt(%0)" :: "n"(n) : "memory")
#define WAITL() asm volatile("s_waitcnt lgkmcnt(0)" ::: "memory")
#define BAR() asm volatile("s_waitcnt lgkmcnt(0)\n\ts_barrier" ::: "memory")
#define RLX_WG __ATOMIC_RELAXED, __HIP_MEMORY_SCOPE_WORKGROUP
#define RLX_AG __ATOMIC_RELAXED, __HIP_MEMORY_SCOPE_AGENT
#define RLX_SYS __ATOMIC_RELAXED, __HIP_MEMORY_SCOPE_SYSTEM
#define CFENCE() asm volatile("" ::: "memory")

__device__ __forceinline__ u64 rt() { return __builtin_amdgcn_s_memrealtime(); }
__device__ __forceinline__ u32 uni(u32 v) { return __builtin_amdgcn_readfirstlane(v); }
__device__ __forceinline__ u32 perm(u32 hi, u32 lo, u32 sel) { return __builtin_amdgcn_perm(hi, lo, sel); }
__device__ __forceinline__ u32 scan64(u32 v) {
    v += __builtin_amdgcn_update_dpp(0u, v, 0x111, 0xf, 0xf, true);
    v += __builtin_amdgcn_update_dpp(0u, v, 0x112, 0xf, 0xf, true);
    v += __builtin_amdgcn_update_dpp(0u, v, 0x114, 0xf, 0xf, true);
    v += __builtin_amdgcn_update_dpp(0u, v, 0x118, 0xf, 0xf, true);
    v += __builtin_amdgcn_update_dpp(0u, v, 0x142, 0xa, 0xf, false);
    v += __builtin_amdgcn_update_dpp(0u, v, 0x143, 0xc, 0xf, false);
    return v;
}
__device__ __forceinline__ u32 last(u32 v) { return __builtin_amdgcn_readlane(v, 63); }
__device__ __forceinline__ u32 escbits(u32 q) { return ~(q | (q >> 1)) & 0x55555555u; }
__device__ __forceinline__ u32 lookup8(u32 sel, u32 thi, u32 tlo) {
    const u32 r = perm(thi, tlo, sel);
    return __builtin_amdgcn_udot4((r ^ (r >> 2)) & 0x18181818u, 0x40100401u, 0u, false);
}
__device__ __forceinline__ u32 classify8(u32 e4, u32 kadd, u32 thi, u32 tlo) {
    return lookup8(((e4 >> 2) + kadd) & 0x1F1F1F1Fu, thi, tlo);
}
constexpr u32 T13 = 0x00181008u, T2L = 0x00000818u, T2H = 0x00001000u;
__device__ __forceinline__ u32 ones(u32 sel) { return perm(0u, 0x00008000u, sel); }
__device__ __forceinline__ u32 bfi(u32 m, u32 a, u32 b) { return (a & m) | (b & ~m); }
__device__ __forceinline__ u32 rep(u32 b) { return b * 0x01010101u; }
__device__ __forceinline__ u32 blk(u32 r, u32 b) { return (r < 4 ? r : r - b) & 3; }
__device__ __forceinline__ u32 partner(u32 r, u32 b) { return r < 4 ? 4 + ((r + b) & 3) : (r - b) & 3; }
__device__ __forceinline__ float silu(float acc, float sc) {
    const float y = acc * sc;
    const float e = __expf(-__builtin_amdgcn_fmed3f(y, -60.f, 60.f));
    return y * __builtin_amdgcn_rcpf(1.f + e);
}
__device__ __forceinline__ int cvt_i32(float f) { int r; asm("v_cvt_i32_f32 %0, %1" : "=v"(r) : "v"(f)); return r; }

__global__ void __launch_bounds__(NTHR) fused_gemm_rs(Args a) {
    __shared__ __attribute__((aligned(16))) u8 lds_raw[L_END];
    LAS u8* const L = (LAS u8*)lds_raw;
#define L16(off) (*(LAS unsigned short*)(L + (off)))
#define L32(off) (*(LAS u32*)(L + (off)))
#define L64(off) (*(LAS u32x2*)(L + (off)))
#define L128(off) (*(LAS u32x4*)(L + (off)))
#define LPOLL(off) uni(__hip_atomic_load((LAS u32*)(L + (off)), RLX_WG))
    const u32 bid = blockIdx.x;
    u32 lane = threadIdx.x & 63;
#define FRESH_LANE() do { asm volatile("" : "+v"(lane)); __builtin_assume(lane < 64); } while (0)
    const u32 wave = uni(threadIdx.x >> 6);
    const u32 rank = a.rank;
    const bool is_link = (bid >> 3) < 7;
    u8* const myws = a.self;
    const u32 lbase = (u32)(size_t)L;
    const u64 t_begin = rt();
#define STAMP(i)

    const u32 tile = ((bid & 7) | ((bid >> 1) & 8)) * 16 + ((0x72104365u >> (4 * (bid >> 5))) & 7) * 2 + ((bid >> 3) & 1);
    const u32 xrun = 256 + ((bid >> 1) & 7) * 8 + (7 - (bid >> 5)) * 4 + ((bid >> 4) & 1) * 2 + (bid & 1);
    u32 f[4][4];
    auto fetch_run = [&](u32 id) {
        if (id >= 256) {
            const __amdgpu_buffer_rsrc_t rx = __builtin_amdgcn_make_buffer_rsrc((void*)a.x, 0, 4096 * KS, 0x00020000);
            const u32 vo = (lane & 15) * KS + (lane >> 4) * 32, so = (id - 256) * 64 * KS + (wave >> 1) * 128 + (wave & 1) * 16;
#pragma unroll
            for (int c = 0; c < 4; ++c) {
                const u32x4 v = __builtin_amdgcn_raw_buffer_load_b128(rx, vo, so + c * 16 * KS, 0);
                f[c][0] = v[0]; f[c][1] = v[1]; f[c][2] = v[2]; f[c][3] = v[3];
            }
        } else {
            const __amdgpu_buffer_rsrc_t rw = __builtin_amdgcn_make_buffer_rsrc((void*)a.w, 0, KS * GN, 0x00020000);
            const u32 vo = (lane >> 4) * 32 * GN + (lane & 15) * 4, so = ((wave >> 1) * 128 + (wave & 1) * 16) * GN + id * 64;
#pragma unroll
            for (int r = 0; r < 16; ++r) f[r >> 2][r & 3] = __builtin_amdgcn_raw_buffer_load_b32(rw, vo, so + r * GN, 0);
        }
        __builtin_amdgcn_sched_barrier(0);
    };
    if (wave < 8) fetch_run(bid >= 224 ? xrun : tile);

    const u32 cnt_v = __hip_atomic_load((GAS u32*)(myws + OFF_CNT + bid * 128), RLX_AG);
#define GET_CALL const u32 call = uni(cnt_v) + 1, par = call & 1
    u32 spins = 0;
    auto expired = [&](u32 call) -> bool {
        if (++spins & 15) return false;
        return uni(__hip_atomic_load((GAS u32*)(myws + OFF_TMO), RLX_AG)) == call || rt() - t_begin > DEADLINE_TICKS;
    };
    auto give_up = [&](u32 call) {
        if (lane == 0) {
            __hip_atomic_store((GAS u32*)(myws + OFF_TMO), call, RLX_AG);
            __hip_atomic_store((LAS u32*)(L + C_DEAD), 1u, RLX_WG);
        }
    };
    auto lds_init = [&]() {
        for (u32 o = L_ZERO + threadIdx.x * 16; o < L_END; o += 576 * 16) L128(o) = u32x4{0, 0, 0, 0};
        if (threadIdx.x < 256) {
            const u32 x = threadIdx.x;
            u32 d = 0, c = 0x0C0C0C0Cu, n = 0;
#pragma unroll
            for (int b = 0; b < 4; ++b) {
                const u32 code = (x >> (2 * b)) & 3;
                d |= (code ? 4 + code : n) << (8 * b);
                if (!code) c = (c & ~(0xFFu << (8 * n))) | (b << (8 * n));
                n += !code;
            }
            L32(L_LUTD + x * 4) = d;
            L64(L_LUTC + x * 8) = u32x2{c, n};
        }
    };
    const u32 li = bid & 7, from = (rank + 1 + (bid >> 3)) & 7;
    const bool cross = ((rank ^ from) & 4) != 0;
    const u32 pband = (rank < 4 ? from - rank : rank - from) & 3;
    const u32 nitem = cross ? 88 + NOWN + 3 * NRL + 96 : 24 + WDIR + NOWN + 128;
    auto slot_of = [&](u32 n) -> u32 {
        const u32 ns = cross ? 12 : 8, nh = ns / 2, fj = (from - rank) & 3;
        if (n < ns + 8) {
            if (n >= nh && n < nh + 8) return S_W + n - nh;
            const u32 m = n < nh ? n : n - nh - 8, j = 1 + (m >> 1);
            return S_X + ((rank & 4) | ((rank + j + (!cross && j >= fj)) & 3)) * 8 + (n < nh ? 0 : 2) + (m & 1);
        }
        if (n < ns + 24) return S_W + n - ns;
        if (n < 48) { const u32 m = n - ns - 24, j = m >> 2; return S_X + ((~rank & 4) | ((rank + j + (cross && j >= fj)) & 3)) * 8 + 4 + (m & 3); }
        const u32 nw = 24 + (cross ? 64 : WDIR);
        if (n < nw) {
            const u32 o = n - 24, wf = WDIR & ~7u;
            return S_W + (!cross || o < wf ? o : o < wf + 4 * NRL ? o - wf + WDIR : o - 4 * NRL);
        }
        u32 m = n - nw;
        if (m < NOWN) return S_X + rank * 8 + 2 * NPL + m;
        m -= NOWN;
        if (!cross) return S_O + ((m & ~14u) | ((m & 6) << 1) | ((m & 8) >> 2));
        if (m < 3 * NRL) return S_RL + ((rank + 1 + m % 3) & 3) * 8 + m / 3;
        m -= 3 * NRL;
        const u32 b = (m % 12) >> 2;
        return S_O + (m / 12) * 16 + (b + (b >= pband)) * 4 + (m & 3);
    };
    auto run_id = [&](u32 sl) -> u32 {
        if (sl >= S_X && sl < S_RL) return 256 + sl - S_X;
        const u32 o = sl < S_X ? sl : (16 - NRL + (sl & 7)) * 4 + pband;
        return (o & 3) * 64 + blk(rank, o & 3) * 16 + (o >> 2);
    };
    auto run_src = [&](u32 sl) -> u32 { return sl < S_RL ? from : (rank & 4) | ((sl >> 3) & 3); };
    auto run_pos = [&](u32 sl) -> u32 { return sl < S_RL ? run_id(sl) : 320 + pband * 32 + sl - S_RL; };

    if (wave == 9) {
        const u32 q = (rank + 1 + bid) & 7, t = (bid >> 3) & 3, rg = bid >> 5;
        if (q != rank && rg >= 2 * NPL) {
            asm volatile("s_barrier" ::: "memory");
            return;
        }
        const __amdgpu_buffer_rsrc_t rx = __builtin_amdgcn_make_buffer_rsrc((void*)a.x, 0, 4096 * KS, 0x00020000);
        u32x4 v[4][2];
#pragma unroll
        for (int rb = 0; rb < 4; ++rb)
#pragma unroll
            for (int p = 0; p < 2; ++p)
                v[rb][p] = __builtin_amdgcn_raw_buffer_load_b128(rx, (lane & 15) * KS + (lane >> 4) * 32 + p * 16, (q * MS + rg * 64 + rb * 16) * KS + t * 128, 0);
        u8* dst = a.ws[q];
        asm volatile("s_barrier" ::: "memory");
        GET_CALL;
        asm volatile("" :: "v"(v[0][0]), "v"(v[0][1]), "v"(v[1][0]), "v"(v[1][1]),
                          "v"(v[2][0]), "v"(v[2][1]), "v"(v[3][0]), "v"(v[3][1]));
        const u32 vo = OFF_XO + (par * 4 + (rg >> 1)) * PSTR + (rank * 4 + t) * 16384 + (rg & 1) * 8192 + lane * 16 + 4096;
        STAMP(4);
#define XW() WAITV(XWIN - 1)
        ST16(vo, v[0][0], dst, -4096); XW(); ST16(vo, v[0][1], dst, -3072); XW();
        ST16(vo, v[1][0], dst, -2048); XW(); ST16(vo, v[1][1], dst, -1024); XW();
        ST16(vo, v[2][0], dst, 0);     XW(); ST16(vo, v[2][1], dst, 1024);  XW();
        ST16(vo, v[3][0], dst, 2048);  XW(); ST16(vo, v[3][1], dst, 3072);
        WAITV(0);
        if (lane == 0) __hip_atomic_store((GAS u32*)(dst + OFF_XF + par * 1024 + rg * 128 + (rank * 4 + t) * 4), call, RLX_SYS);
        STAMP(1);
        return;
    }
    if (wave == 10) {
        asm volatile("s_barrier" ::: "memory");
        GET_CALL;
        if (!is_link) return;
        __builtin_amdgcn_s_setprio(2);
        bool part_ok = false;
        u64 f = 0;
        for (u32 k = 0;; ++k) {
            u32 n = li + 8 * k;
            if (k > 1) {
                while (LPOLL(C_START) < k) __builtin_amdgcn_s_sleep(2);
                u32 tk = 0;
                if (lane == 0) tk = __hip_atomic_fetch_add((GAS u32*)(myws + OFF_Q + (8 + (bid >> 3)) * 128), 1u, RLX_AG);
                n = 16 + uni(tk) - (nitem - 8) * (call - 1);
            }
            u32 lines = 0, sl = S_END;
            if (n < nitem) {
                sl = slot_of(n);
                while (sl >= S_O && !part_ok) {
                    const u32 pf = __hip_atomic_load((GAS u32*)(myws + OFF_PF + (64 + (lane & 15)) * 128), RLX_SYS);
                    part_ok = __all(pf >= call || (lane & 3) == blk(rank, (lane >> 2) & 3));
                    if (!part_ok && expired(call)) break;
                }
                const u32 psl = k == 0 && lane == 1 ? slot_of(li + 8) : sl;
                for (;;) {
                    if (k == 1 && __builtin_amdgcn_readlane((u32)f, 1) >= call) {
                        lines = __builtin_amdgcn_readlane((u32)(f >> 32), 1) & 0xFF;
                        break;
                    }
                    f = __hip_atomic_load((GAS u64*)(myws + OFF_F + ((par * 8 + from) * 512 + psl) * 128), RLX_SYS);
                    lines = uni((u32)(f >> 32)) & 0xFF;
                    if (uni((u32)f) >= call && (sl < S_O || part_ok)) break;
                    if (expired(call)) { give_up(call); sl = S_END; break; }
                    __builtin_amdgcn_s_sleep(2);
                }
            } else if (uni(__hip_atomic_load((GAS u32*)(myws + OFF_TMO), RLX_AG)) == call) {
                give_up(call);
            }
            while (LPOLL(C_DONE + (k & 3) * 4) < 8 * (k >> 2)) __builtin_amdgcn_s_sleep(1);
            CFENCE();
            lines = min(max(lines, 1u), (sl < S_O ? VARCAP : EXCCAP) / 128);
            const u32 src = sl >= S_O ? OFF_OP + par * OP_PAR + (rank * 128 + ((sl - S_O) & 127)) * OHALF : OFF_P + par * P_PAR + run_pos(sl) * RUN;
            if (lane == 0)
                __hip_atomic_store((LAS u64*)(L + C_RDY + (k & 7) * 8), ((u64)src << 32) | ((k + 1) << 16) | (sl << 6) | lines, RLX_WG);
            if (sl == S_END) return;
        }
    }
    if (wave == 8) {
        lds_init();
        GET_CALL;
        BAR();
        if (!is_link) return;
        __builtin_amdgcn_s_setprio(3);
        const u8* base = a.ws[from];
        const u32 l16 = lane * 16, a_landed = lbase + C_LANDED;
        int issued = 0;
        auto publish = [&](int v) {
            asm volatile("s_mov_b64 exec, 1\n\tds_max_i32 %0, %1\n\ts_mov_b64 exec, -1" :: "v"(a_landed), "v"(v) : "memory");
        };
        auto pull = [&](u32 goff, u32 loff, u64 mask) {
            asm volatile("s_mov_b32 m0, %1\n\ts_mov_b64 exec, %3\n\ts_waitcnt vmcnt(%4)\n\t"
                         "global_load_lds_dwordx4 %0, %2 sc0 sc1\n\ts_mov_b64 exec, -1"
                         :: "v"(l16), "s"(lbase + loff), "s"(base + goff), "s"(mask), "n"(WINDOW - 1) : "memory");
            publish(issued - (WINDOW - 1));
            ++issued;
        };
        bool early = false;
        for (;;) {
            early = uni(__hip_atomic_load((GAS u32*)(myws + OFF_F + ((par * 8 + from) * 512 + slot_of(li) + S_EARLY) * 128), RLX_SYS)) >= call;
            if (early) break;
            if (expired(call)) { give_up(call); break; }
        }
        CFENCE();
        STAMP(2);
        u32 o0 = VARCAP;
        while (early && o0 < RUN && !(LPOLL(C_RDY) >> 16)) {
            pull(OFF_P + par * P_PAR + run_pos(slot_of(li)) * RUN + o0, L_BIG + o0, ~0ull);
            o0 += 1024;
        }
        if (lane == 0) __hip_atomic_store((LAS u32*)(L + C_VPOS), (u32)issued + 1, RLX_WG);
        for (u32 m = 0;; ++m) {
            u64 r64;
            auto dealt = [&]() { return uni((u32)(r64 = __hip_atomic_load((LAS u64*)(L + C_RDY + (m & 7) * 8), RLX_WG))) >> 16 == m + 1; };
            if (!dealt()) {
#define STEP(w) if (!dealt()) { WAITV(w); publish(issued - w); }
                STEP(5) STEP(4) STEP(3) STEP(2) STEP(1) STEP(0)
                while (!dealt()) __builtin_amdgcn_s_sleep(1);
            }
            const u32 rdy = uni((u32)r64), g0 = uni((u32)(r64 >> 32));
            CFENCE();
            const u32 sl = (rdy >> 6) & 511;
            if (sl == S_END) break;
            const u32 l0 = L_BIG + (m & 3) * RUN;
            for (u32 o = 0, ln = rdy & 63; ; o += 1024, ln -= 8) {
                pull(g0 + o, l0 + o, ln >= 8 ? ~0ull : (1ull << (8 * ln)) - 1);
                if (ln <= 8) break;
            }
            if (lane == 0) __hip_atomic_store((LAS u32*)(L + C_START), m + 1, RLX_WG);
            for (u32 o = m == 0 ? o0 : sl >= S_O ? EXCCAP : VARCAP; o < RUN; o += 1024) pull(g0 + o, l0 + o, ~0ull);
        }
        WAITV(4); publish(issued - 4); WAITV(2); publish(issued - 2); WAITV(0); publish(issued);
        STAMP(21);
        return;
    }

    u32 phase = 0;
    auto wbar = [&]() {
        phase += 8;
        WAITL();
        if (lane == 0) __hip_atomic_fetch_add((LAS u32*)(L + C_WB), 1u, RLX_WG);
        while (LPOLL(C_WB) < phase) __builtin_amdgcn_s_sleep(1);
        CFENCE();
    };
    auto append = [&](auto G, u32 pos, const u32* e4, const u32x2* lut) {
        u32 carry = 0;
#pragma unroll
        for (int i = 0; i < G.value; ++i) {
            const u32 comp = perm(0u, e4[i], lut[i][0]), sh = pos << 3;
            __hip_atomic_fetch_or((LAS u32*)(L + (pos & ~3u)), (comp << (sh & 31)) | carry, RLX_WG);
            carry = __builtin_amdgcn_ubfe(comp, 0u - sh, sh);
            pos += lut[i][1];
        }
        __hip_atomic_fetch_or((LAS u32*)(L + (pos & ~3u)), carry, RLX_WG);
    };
    auto expand = [&](u32 tab, u32 pos, u32 x) -> u32 {
        const u32 sel = L32(L_LUTD + x * 4);
        const LAS u32* p = (const LAS u32*)(L + (pos & ~3u));
        return perm(tab, __builtin_amdgcn_alignbyte(p[1], p[0], pos), sel);
    };
    auto panel_of = [&](u32 id) -> u32 { return id >= 256 ? (id - 256) >> 1 : 32 + ((id >> 1) & 7) * 4 + (id >> 6); };
    auto stage_off = [&](u32 id, u32 s, u32 par) -> u32 {
        return ((id - 256) >> 3 == rank ? OFF_XO + (par * 4 + ((id >> 1) & 3)) * PSTR : OFF_SP + panel_of(id) * PSTR) + (s * 4 + (wave >> 1)) * 16384 + (id & 1) * 8192 + (wave & 1) * 1024 + lane * 16 + 4096;
    };
    auto arrive = [&](u32 c) -> u32 {
        u32 t = ~0u;
        asm volatile("s_mov_b64 exec, 1\n\tglobal_atomic_add %0, %1, %2, %3 sc0\n\ts_mov_b64 exec, -1"
                     : "+v"(t) : "v"(OFF_CT + c * 128), "v"(1u), "s"(myws) : "memory");
        return t;
    };
    auto raise = [&](u32 c, u32 t, u32 call) {
        const u32 per_call = c < 32 ? 112 : c < 64 ? 128 : c < 128 ? 32 : 16;
        asm volatile("s_waitcnt vmcnt(0)" : "+v"(t) :: "memory");
        if (t == per_call * call - 1) asm volatile("global_store_dword %0, %1, %2 sc1" :: "v"(OFF_PF + c * 128), "v"(call), "s"(myws) : "memory");
    };
    const u32 v2 = L_SCR + wave * WSCR, v3 = v2 + V2S, ev = v3 + V3S;
    auto pack_levels = [&](auto NP, u32 N2, u32 e, auto cls2, auto cls3) -> u32 {
        u32 u[2 * NP.value], m = 0;
        u32x2 l2[2 * NP.value];
#pragma unroll
        for (int i = 0; i < NP.value; ++i) {
            const u32x2 v = L64(v2 + lane * (8 * NP.value) + i * 8);
            u[2 * i] = v[0]; u[2 * i + 1] = v[1];
        }
#pragma unroll
        for (int i = 0; i < NP.value; ++i) {
            u32 q8 = cls2(u[2 * i]) | (cls2(u[2 * i + 1]) << 8);
            const int valid = (int)N2 - (int)lane * (8 * NP.value) - i * 8;
            q8 |= (0x5555u * 8) << (2 * min(max(valid, 0), 8));
            if (valid > 0) L16(e + lane * (2 * NP.value) + i * 2) = (unsigned short)(q8 >> 3);
            l2[2 * i] = L64(L_LUTC + (q8 & 0x7F8));
            l2[2 * i + 1] = L64(L_LUTC + ((q8 >> 8) & 0x7F8));
        }
#pragma unroll
        for (int i = 0; i < 2 * NP.value; ++i) m += l2[i][1];
        const u32 in2 = scan64(m);
        const u32 N3 = last(in2);
        if (N3 > CAP3) return ~0u;
        if (m) append(std::integral_constant<int, 2 * NP.value>{}, v3 + in2 - m, u, l2);
        const u32 off3 = ((N2 + 7) >> 3) * 2, off4 = off3 + ((N3 + 7) >> 3) * 2;
        const u32x2 v = L64(v3 + lane * 8);
        const u32 w[2] = {v[0], v[1]};
        u32 q8 = cls3(w[0]) | (cls3(w[1]) << 8);
        const int valid = (int)N3 - (int)lane * 8;
        q8 |= (0x5555u * 8) << (2 * min(max(valid, 0), 8));
        if (valid > 0) L16(e + off3 + lane * 2) = (unsigned short)(q8 >> 3);
        const u32x2 l3[2] = {L64(L_LUTC + (q8 & 0x7F8)), L64(L_LUTC + ((q8 >> 8) & 0x7F8))};
        const u32 m3 = l3[0][1] + l3[1][1];
        const u32 in3 = scan64(m3);
        if (m3) append(std::integral_constant<int, 2>{}, e + off4 + in3 - m3, w, l3);
        return off4 + last(in3);
    };
    auto unpack_levels = [&](auto NP, u32 N2, u32 sec, u32 t2, u32 t3) {
        u32 q[NP.value], e[NP.value], n = 0;
#pragma unroll
        for (int i = 0; i < NP.value; ++i) {
            q[i] = L16(sec + lane * (2 * NP.value) + i * 2) | 0x55550000u;
            if (lane * (8 * NP.value) + i * 8 >= N2) q[i] = 0x55555555u;
            e[i] = escbits(q[i]);
            n += __builtin_popcount(e[i]);
        }
        const u32 in2 = scan64(n), N3 = last(in2);
        const u32 off3 = sec + ((N2 + 7) >> 3) * 2, off4 = off3 + ((N3 + 7) >> 3) * 2;
        u32 q3 = L16(off3 + lane * 2) | 0x55550000u;
        if (lane * 8 >= N3) q3 = 0x55555555u;
        const u32 e3 = escbits(q3), m3 = __builtin_popcount(e3);
        const u32 p3 = off4 + scan64(m3) - m3;
        L64(v3 + lane * 8) = u32x2{expand(t3, p3, q3 & 0xFF), expand(t3, p3 + __builtin_popcount(e3 & 0xFF), (q3 >> 8) & 0xFF)};
        u32 pp = v3 + in2 - n;
        u32x2 z[NP.value];
#pragma unroll
        for (int i = 0; i < NP.value; ++i) {
            z[i] = u32x2{expand(t2, pp, q[i] & 0xFF), expand(t2, pp + __builtin_popcount(e[i] & 0xFF), (q[i] >> 8) & 0xFF)};
            pp += __builtin_popcount(e[i]);
        }
#pragma unroll
        for (int i = 0; i < NP.value; ++i) L64(v2 + lane * (8 * NP.value) + i * 8) = z[i];
    };
    u8* peer = a.ws[lane & 7];
    u32 npub = 0, owed = ~0u;

    lds_init();
    BAR();
    for (u32 job = 0; job < (bid >= 192 ? 2u : 1u); ++job) {
        const u32 id = (bid >= 192 && job == (bid >= 224 ? 0u : 1u)) ? xrun : tile;
        const u32 t = wave >> 1, h = wave & 1;
        if (id >= 256 && ((id - 256) >> 3) == rank) continue;
        if (job) {
            fetch_run(id);
            for (u32 o = lane * 16; o < WSCR; o += 1024) L128(v2 + o) = u32x4{0, 0, 0, 0};
        }
        if (id < 256) {
            u32 g[4][4];
#pragma unroll
            for (int i = 0; i < 4; ++i) {
                const u32 t0 = perm(f[i][1], f[i][0], 0x05010400u), t1 = perm(f[i][1], f[i][0], 0x07030602u);
                const u32 t2 = perm(f[i][3], f[i][2], 0x05010400u), t3 = perm(f[i][3], f[i][2], 0x07030602u);
                g[0][i] = perm(t2, t0, 0x05040100u); g[1][i] = perm(t2, t0, 0x07060302u);
                g[2][i] = perm(t3, t1, 0x05040100u); g[3][i] = perm(t3, t1, 0x07060302u);
            }
#pragma unroll
            for (int i = 0; i < 16; ++i) f[i >> 2][i & 3] = g[i >> 2][i & 3];
        }
        GET_CALL;
        asm volatile("" : "+v"(peer));
        const u32x4 f0 = {f[0][0], f[0][1], f[0][2], f[0][3]}, f1 = {f[1][0], f[1][1], f[1][2], f[1][3]};
        const u32x4 f2 = {f[2][0], f[2][1], f[2][2], f[2][3]}, f3 = {f[3][0], f[3][1], f[3][2], f[3][3]};
        auto keep = [&]() {
            if (id >= 256 && (((id >> 3) ^ rank ^ id) & 4)) return;
            const bool pan = id >= 256 || ((id >> 4) & 3) == blk(rank, id >> 6);
            const u32 vo = pan ? stage_off(id, rank, par) : OFF_LW + (id >> 1) * 65536 + t * 16384 + (id & 1) * 8192 + h * 1024 + lane * 16 + 4096;
            const u32 c = pan ? panel_of(id) : 128 + (id >> 1);
            ST16(vo, f0, myws, -4096); ST16(vo, f1, myws, -2048); ST16(vo, f2, myws, 0); ST16(vo, f3, myws, 2048);
            if (is_link) { owed = c; return; }
            WAITV(0);
            raise(c, arrive(c), call);
        };
        u32 esum = 0;
#pragma unroll
        for (int i = 0; i < 4; ++i) esum = __builtin_amdgcn_udot4(f[i][i] & 0x7C7C7C7Cu, 0x01010101u, esum, false);
        const u32 e0 = ((last(scan64(esum)) + 3686u) >> 12) & 31;
        const u32 runo = OFF_P + par * P_PAR + id * RUN;
        u32x4 Q, Mq;
        u32 S[2];
        u32 e4[16];
        u32x2 lut[16];
        const u32 k1 = rep((33 - e0) & 31);
#pragma unroll
        for (int c = 0; c < 4; ++c) {
            u32 q = 0;
#pragma unroll
            for (int i = 0; i < 4; ++i) {
                e4[4 * c + i] = f[c][i] & 0x7C7C7C7Cu;
                const u32 x8 = classify8(e4[4 * c + i], k1, 0u, T13);
                lut[4 * c + i] = L64(L_LUTC + x8);
                q |= i ? x8 << (8 * i - 3) : x8 >> 3;
            }
            Q[c] = q;
        }
#pragma unroll
        for (int j = 0; j < 6; ++j) {
            const u32 x = j < 4 ? f[3][j] << 5 : bfi(0x40404040u, f[3][2 * j - 8] >> 1, f[3][2 * j - 7] >> 2);
            const u32 w = bfi(0x60606060u, x, bfi(0x1C1C1C1Cu, __builtin_rotateleft32(f[j >> 1][2 * (j & 1) + 1], 3), f[j >> 1][2 * (j & 1)]));
            if (j < 4) Mq[j] = w; else S[j - 4] = w;
        }
        {
            const u32 vo = runo + VARCAP + wave * HALFB + lane * 16;
            ST16(vo, Q, myws, 0);
            ST16(vo, Mq, myws, 1024);
            const u32x4 s4 = {S[0], S[1], (u32)__shfl_xor((int)S[0], 32), (u32)__shfl_xor((int)S[1], 32)};
            if (lane < 32) ST16(vo, s4, myws, 2048);
        }
        u32 slot;
        bool to;
        if (id >= 256) {
            const u32 q = (id - 256) >> 3;
            slot = S_X + id - 256;
            to = lane < 8 && lane != rank && (lane == q ? (id & 7) >= 2 * NPL : !((q ^ lane ^ id) & 4));
        } else {
            slot = S_W + (id & 15) * 4 + (id >> 6);
            to = lane < 8 && lane != rank && ((id >> 4) & 3) == blk(lane, id >> 6) && (slot < WDIR || ((lane ^ rank) & 4));
        }
        if (id >= 256 ? (id & 6) == 0 : slot < 4) {
            WAITV(0);
            u32 arrived = 0;
            if (lane == 0) arrived = __hip_atomic_fetch_add((LAS u32*)(L + C_PUB1), 1u, RLX_WG);
            if (uni(arrived) == 7 && to)
                __hip_atomic_store((GAS u32*)(peer + OFF_F + ((par * 8 + rank) * 512 + (slot + S_EARLY)) * 128), call, RLX_SYS);
            STAMP(3);
        }
        u32 n = 0;
#pragma unroll
        for (int c = 0; c < 4; ++c) n += __builtin_popcount(escbits(Q[c]));
        const u32 inc = scan64(n);
        const u32 N2 = last(inc);
        u32 len = 0;
        bool raw = N2 > CAP2 || FORCE_RAW & 1;
        if (!raw) {
            append(std::integral_constant<int, 16>{}, v2 + inc - n, e4, lut);
            const u32 k2 = rep((35 - e0) & 31), k3 = rep((38 - e0) & 31);
            len = pack_levels(std::integral_constant<int, 3>{}, N2, ev, [&](u32 u) { return classify8(u, k2, T2H, T2L); },
                              [&](u32 u) { return classify8(u, k3, 0u, T13); });
            raw = len > SECCAP;
        }
        if (raw) {
            len = 0;
            const u32 vo = OFF_RAW + par * RAW_PAR + (id * 8 + wave) * 4096 + lane * 16;
            ST16(vo, f0, myws, 0); ST16(vo, f1, myws, 1024); ST16(vo, f2, myws, 2048); ST16(vo, f3, myws, 3072);
        }
        const u32 len16 = (len + 15) >> 4;
        if (lane == 0) L32(C_HDR + wave * 4) = len16 | (raw ? 0x80000000u : (N2 << 9) | (e0 << 20));
        wbar();
        const u32 hv = lane < 8 ? L32(C_HDR + lane * 4) : 0u;
        const u32 hin = scan64(hv & 0x1FF);
        const u32 hoff = 2 + hin - (hv & 0x1FF);
        if (lane < len16) {
            const u32x4 sec = L128(ev + lane * 16);
            ST16(runo + __builtin_amdgcn_readlane(hoff, wave) * 16 + lane * 16, sec, myws, 0);
        }
        if (wave == 0 && lane < 8) ST4(runo + lane * 4, (hv & ~0x1FFu) | hoff, myws);
        WAITV(0);
        u32 arrived = 0;
        npub += 8;
        if (lane == 0) arrived = __hip_atomic_fetch_add((LAS u32*)(L + C_PUB), 1u, RLX_WG);
        if (uni(arrived) == npub - 1) {
            const u64 fl = ((u64)((2 + last(hin) + 7) >> 3) << 32) | call;
            if (to) __hip_atomic_store((GAS u64*)(peer + OFF_F + ((par * 8 + rank) * 512 + slot) * 128), fl, RLX_SYS);
            STAMP(22 + job);
        }
        keep();
        if (bid >= 192 && job == 0) wbar();
    }

    GET_CALL;
    const float sc = a.sx[0] * a.sw[0];
    const u32 rg = wave >> 1, ch = wave & 1;
    auto out_store = [&](u32 e, u32 r, f32x4 v) {
        asm volatile("global_store_dwordx4 %0, %1, %2 sc1\n\ts_nop 1"
                     :: "v"((lane >> 4) * (16 * GN) + (lane & 15) * 16), "v"(v), "s"(a.out + e + r * GN) : "memory");
    };

    if (is_link) {
        auto landed = [&](u32 mark) {
            while (LPOLL(C_LANDED) < mark) __builtin_amdgcn_s_sleep(1);
            CFENCE();
        };
        auto done = [&](u32 m) {
            WAITL();
            asm volatile("s_mov_b64 exec, 1\n\tds_add_u32 %0, %1\n\ts_mov_b64 exec, -1" :: "v"(lbase + C_DONE + (m & 3) * 4), "v"(1u) : "memory");
        };
        u32 m = 0, sl, lines, pbase = 0;
        auto settle = [&]() {
            if (owed != ~0u) { WAITV(0); raise(owed, arrive(owed), call); owed = ~0u; }
        };
        auto next = [&]() {
            u32 rdy = LPOLL(C_RDY + (m & 7) * 8);
            if ((rdy >> 16) != m + 1) {
                settle();
                while (((rdy = LPOLL(C_RDY + (m & 7) * 8)) >> 16) != m + 1) __builtin_amdgcn_s_sleep(1);
            }
            sl = (rdy >> 6) & 511;
            lines = rdy & 63;
            if (sl == S_END) return;
            pbase += (lines + 7) >> 3;
            u32 ahead = 1;
            if (m == 0) while (!(ahead = LPOLL(C_VPOS))) __builtin_amdgcn_s_sleep(1);
            if (LPOLL(C_LANDED) < pbase + ahead - 1) settle();
            landed(pbase + ahead - 1);
        };
        for (;; ++m) {
            next();
            if (sl >= S_O) break;
            const u32 id = run_id(sl), s = run_src(sl), vb = L_BIG + (m & 3) * RUN;
            const u8* base = a.ws[s];
            u32 tick = 0, tickc = ~0u;
            const bool relay = sl >= WDIR && sl < S_X && (sl & 3) != pband;
            const u32 rsl = (from & 3) * 8 + (sl >> 2) - (16 - NRL);
            auto pass_on = [&]() {
                const u32 ro = OFF_P + par * P_PAR + (320 + (sl & 3) * 32 + rsl) * RUN;
                const u32 o0 = wave * 640 + lane * 16, o1 = VARCAP + wave * HALFB + lane * 16;
                if (lane < 40 && o0 < lines * 128) { const u32x4 v = L128(vb + o0); ST16(ro + o0, v, myws, 0); }
                const u32x4 v0 = L128(vb + o1), v1 = L128(vb + o1 + 1024);
                ST16(ro + o1, v0, myws, 0); ST16(ro + o1, v1, myws, 1024);
                if (lane < 32) { const u32x4 v = L128(vb + o1 + 2048); ST16(ro + o1, v, myws, 2048); }
            };
            const u32 hdr = uni(L32(vb + wave * 4));
            u32x4 r[4];
            if (hdr >> 31) {
                const u32 vo = OFF_RAW + par * RAW_PAR + (id * 8 + wave) * 4096 + lane * 16;
                asm volatile("global_load_dwordx4 %0, %4, %5 sc0 sc1\n\tglobal_load_dwordx4 %1, %4, %5 offset:1024 sc0 sc1\n\t"
                             "global_load_dwordx4 %2, %4, %5 offset:2048 sc0 sc1\n\tglobal_load_dwordx4 %3, %4, %5 offset:3072 sc0 sc1\n\t"
                             "s_waitcnt vmcnt(0)"
                             : "=&v"(r[0]), "=&v"(r[1]), "=&v"(r[2]), "=&v"(r[3]) : "v"(vo), "s"(base) : "memory");
            } else {
                const u32 e0 = (hdr >> 20) & 31, N2 = (hdr >> 9) & 0x7FF, sec = vb + (hdr & 0x1FF) * 16;
                const u32 t1 = (((e0 - 1) & 31) << 10) | (e0 << 18) | (((e0 + 1) & 31) << 26);
                const u32 t2 = (((e0 - 2) & 31) << 10) | (((e0 + 2) & 31) << 18) | (((e0 - 3) & 31) << 26);
                const u32 t3 = (((e0 - 6) & 31) << 10) | (((e0 - 5) & 31) << 18) | (((e0 - 4) & 31) << 26);
                unpack_levels(std::integral_constant<int, 3>{}, N2, sec, t2, t3);
                const u32 pneed = pbase + (((wave + 1) * HALFB + 1023) >> 10);
                if (LPOLL(C_LANDED) < pneed) settle();
                if (owed != ~0u) { WAITV(0); tick = arrive(owed); tickc = owed; owed = ~0u; }
                landed(pneed);
                if (relay) pass_on();
                const u32 fx = vb + VARCAP + wave * HALFB;
                const u32x4 Q = L128(fx + lane * 16), Mq = L128(fx + 1024 + lane * 16);
                const u32x2 S = L64(fx + 2048 + (lane & 31) * 16 + (lane >> 5) * 8);
                u32 E[4], n = 0;
#pragma unroll
                for (int c = 0; c < 4; ++c) { E[c] = escbits(Q[c]); n += __builtin_popcount(E[c]); }
                u32 pp = v2 + scan64(n) - n;
#pragma unroll
                for (int c = 0; c < 4; ++c)
#pragma unroll
                    for (int i = 0; i < 4; ++i) {
                        const u32 w = c < 2 ? Mq[2 * c + (i >> 1)] : S[i >> 1];
                        const u32 sm = c < 3 ? (i & 1 ? __builtin_rotateright32(w, 3) : w) : bfi(0x80808080u, w << (1 + (i & 1)), Mq[i] >> 5);
                        r[c][i] = expand(t1, pp, (Q[c] >> (8 * i)) & 0xFF) | (sm & 0x83838383u);
                        pp += __builtin_popcount(E[c] & (0xFFu << (8 * i)));
                    }
            }
            if (relay) {
                if (hdr >> 31) pass_on();
                WAITV(0);
                u32 arrived = 0;
                if (lane == 0) arrived = __hip_atomic_fetch_add((LAS u32*)(L + C_RL + (m & 3) * 4), 1u, RLX_WG);
                if ((uni(arrived) & 7) == 7 && lane == 0)
                    __hip_atomic_store((GAS u64*)(a.ws[partner(rank, sl & 3)] + OFF_F + ((par * 8 + rank) * 512 + (S_RL + rsl)) * 128), ((u64)lines << 32) | call, RLX_SYS);
            }
            done(m);
            settle();
            if (tickc != ~0u) raise(tickc, tick, call);
            const u32 vo = stage_off(id, s, par);
            ST16(vo, r[0], myws, -4096); ST16(vo, r[1], myws, -2048); ST16(vo, r[2], myws, 0); ST16(vo, r[3], myws, 2048);
            owed = panel_of(id);
            pbase += 8 * HALFB / 1024;
        }
        settle();
        const u32 u = wave >> 1, rb = wave & 1;
        const u8* base = a.ws[from];
        while (sl != S_END) {
            const u32 id = sl - S_O, vb = L_BIG + (m & 3) * RUN, band = (id >> 2) & 3;
            const u32 e0 = ((cross ? 256 : 0) + (id & 3) * 64 + (u >> 1) * 32 + rb * 16) * GN
                           + (band * 32 + blk(from, band) * 8 + (id >> 4)) * 128 + (u & 1) * 64;
            f32x4 P[4];
#pragma unroll
            for (int k = 0; k < 4; ++k)
                asm volatile("global_load_dwordx4 %0, %1, %2 sc0 sc1"
                             : "=v"(P[k]) : "v"((lane >> 4) * (16 * GN) + (lane & 15) * 16), "s"(a.out + e0 + k * GN) : "memory");
            landed(pbase);
            const u32 hdr = uni(L32(vb + u * 4));
            const u32 n2 = hdr >> 31 ? 0 : (hdr >> 19) & 15;
            u32x2 hard = {(rb ^ 1) << 4, 0};
            u32 he = 0;
            float hp = 0.f;
            f32x4 r[4];
            if (hdr >> 31) {
                done(m);
                const u32 vo = OFF_OR + par * OR_PAR + ((rank * 128 + id) * 8 + wave) * 4096 + lane * 16;
                asm volatile("global_load_dwordx4 %0, %4, %5 sc0 sc1\n\tglobal_load_dwordx4 %1, %4, %5 offset:1024 sc0 sc1\n\t"
                             "global_load_dwordx4 %2, %4, %5 offset:2048 sc0 sc1\n\tglobal_load_dwordx4 %3, %4, %5 offset:3072 sc0 sc1\n\t"
                             "s_waitcnt vmcnt(0)"
                             : "=&v"(r[0]), "=&v"(r[1]), "=&v"(r[2]), "=&v"(r[3]) : "v"(vo), "s"(base) : "memory");
            } else {
                const u32 sec = vb + (hdr & 0xFF) * 16, N2 = (hdr >> 8) & 0x7FF;
                if (lane < n2) hard = L64(sec + lane * 8);
                if (n2) {
                    he = ((((hard[0] >> 9) & 3) * 4 + ((hard[0] >> 2) & 3)) * GN + ((hard[0] >> 5) & 15) * 4 + (hard[0] & 3)) * 4;
                    asm volatile("global_load_dword %0, %1, %2 sc0 sc1\n\ts_waitcnt vmcnt(0)" : "=&v"(hp) : "v"(he), "s"(a.out + e0) : "memory");
                }
                unpack_levels(std::integral_constant<int, 2>{}, N2, sec + n2 * 8, 0x06030100u, 0x08070500u);
                landed(pbase + (((u + 1) * HEADB + 1023) >> 10));
                const u32 hd = vb + EXCCAP + u * HEADB;
                const u32x2 Q = L64(hd + (lane & 31) * 16 + (lane >> 5) * 8), Nb = L64(hd + 512 + lane * 16 + rb * 8);
                const u32 c0 = __builtin_popcount(escbits(Q[0])), c1 = c0 + __builtin_popcount(escbits(Q[1]));
                const u32 q = rb ? Q[1] : Q[0], e = escbits(q);
                u32 pp = v2 + scan64(c1) - c1 + (rb ? c0 : 0);
                u32 ft[8];
#pragma unroll
                for (int g = 0; g < 4; ++g) {
                    const u32 sy = expand(0x04020000u, pp, (q >> (8 * g)) & 0xFF);
                    pp += __builtin_popcount(e & (0xFFu << (8 * g)));
                    const u32 nb = Nb[g >> 1] >> (4 * (g & 1));
                    const u32 T = (((sy << 2) & 0x78787878u) | (nb & 0x07070707u)) ^ ones(nb & 0x08080808u);
                    const u32 F = ((sy << 7) & 0x80808080u) | ((sy >> 1) & 0x70707070u);
                    ft[2 * g] = perm(T, F, 0x05010400u); ft[2 * g + 1] = perm(T, F, 0x07030602u);
                }
                asm volatile("" : "+v"(ft[0]), "+v"(ft[1]), "+v"(ft[2]), "+v"(ft[3]), "+v"(ft[4]), "+v"(ft[5]), "+v"(ft[6]), "+v"(ft[7]));
                landed(pbase + 4 * HEADB / 1024 + (wave + 1) * 2);
                const u32x4 d0 = L128(vb + EXCCAP + 4 * HEADB + wave * 2048 + lane * 16), d1 = L128(vb + EXCCAP + 4 * HEADB + wave * 2048 + 1024 + lane * 16);
                done(m);
                const float dn = __uint_as_float(hdr & 0x7F800000u);
#pragma unroll
                for (int g = 0; g < 4; ++g) {
                    const u32 m0 = g < 2 ? d0[2 * g] : d1[2 * g - 4], m1 = g < 2 ? d0[2 * g + 1] : d1[2 * g - 3];
                    const int k0 = perm(m0, ft[2 * g], 0x01050400u), k1 = perm(m0, ft[2 * g], 0x03070602u);
                    const int k2 = perm(m1, ft[2 * g + 1], 0x01050400u), k3 = perm(m1, ft[2 * g + 1], 0x03070602u);
                    r[g] = f32x4{(float)k0, (float)k1, (float)k2, (float)k3} * dn;
                }
            }
            asm volatile("s_waitcnt vmcnt(0)" : "+v"(P[0]), "+v"(P[1]), "+v"(P[2]), "+v"(P[3]) :: "memory");
#pragma unroll
            for (int g = 0; g < 4; ++g)
                out_store(e0, g, f32x4{silu(r[g][0] + P[g][0], sc), silu(r[g][1] + P[g][1], sc), silu(r[g][2] + P[g][2], sc), silu(r[g][3] + P[g][3], sc)});
            if (n2) {
                WAITV(0);
                if (((hard[0] >> 4) & 1) == rb)
                    asm volatile("global_store_dword %0, %1, %2 sc1" :: "v"(he), "v"(silu(__uint_as_float(hard[1]) + hp, sc)), "s"(a.out + e0) : "memory");
            }
            ++m;
            pbase += (OHALF - EXCCAP) / 1024;
            u32 rdy;
            while (((rdy = LPOLL(C_RDY + (m & 7) * 8)) >> 16) != m + 1) __builtin_amdgcn_s_sleep(1);
            sl = (rdy >> 6) & 511;
            pbase += ((rdy & 63) + 7) >> 3;
        }
        if (LPOLL(C_DEAD)) {
            WAITV(0);
            for (u32 r = wave * 64 + lane; r < MS; r += 512)
                for (u32 c = 0; c < GN; c += 64) a.out[(size_t)r * GN + c] = __builtin_nanf("");
        }
    } else {
        const u32 npart = li < 2 ? 24 : 56, njob = npart + 48;
        for (u32 job = (bid >> 3) - 7, round = 0; job < njob + 128; round += job >= npart) {
            FRESH_LANE();
            const bool part = job < npart, own = part || job >= njob;
            const u32 n = part ? job + (li < 2 ? 24 * li : 56 * li - 64) : own ? job - njob : (job - npart) % 24;
            const u32 band = part ? n / 96 : n & 3;
            u32 q = rank, layer = part ? n & 3 : (n >> 2) & 3, block = blk(rank, band), ti = part ? (n >> 2) & 7 : n >> 4;
            if (part) {
                block = (block + 1 + (n >> 5) % 3) & 3;
            } else if (!own) {
                const u32 bat = ((job < npart + 24 ? 0x76531420u : 0xFEDCBA98u) >> (4 * li)) & 15, j = (n >> 2) % 3;
                q = bat & 1 ? (~rank & 4) | ((partner(rank, band) + 1 + j) & 3) : (rank & 4) | ((rank + 1 + j) & 3);
                layer = (bat & 1) * 2 + (n >= 12);
                ti = bat >> 1;
            }
            const u32 ct = ti * 4 + band, gct = band * 32 + block * 8 + ti;
            u32 nk = part ? 4 : own ? 32 : 28;
            asm volatile("" : "+s"(nk));
            const u32 skip = own ? 32 : 4 * q;
            if (wave == 0) {
                const u32 sl = part ? (lane & 32) + rank * 4 + (lane & 3) : lane;
                const u32 fo = own && (layer < NPL || ((sl >> 2) & 7) == rank) ? OFF_XF + par * 1024 + layer * 256 + sl * 4
                                                                                   : OFF_PF + (q * 4 + layer) * 128;
                bool ok = !LPOLL(C_DEAD);
                while (ok) {
                    const u32 fa = __hip_atomic_load((GAS u32*)(myws + fo), RLX_SYS);
                    const u32 fb = __hip_atomic_load((GAS u32*)(myws + OFF_PF + (part ? 128 + gct : 32 + ct) * 128), RLX_SYS);
                    if (__all(fa >= call && fb >= call)) break;
                    if (expired(call)) { give_up(call); ok = false; }
                    __builtin_amdgcn_s_sleep(3);
                }
            }
            BAR();
            const bool dead = LPOLL(C_DEAD);
            if (wave == 0) STAMP(44 + min(round, 3u) * 3);
            f32x4 acc[2][4];
#pragma unroll
            for (int i = 0; i < 8; ++i) acc[i >> 2][i & 3] = f32x4{0.f, 0.f, 0.f, 0.f};
            for (u32 o = lane * 16; o < WSCR; o += 1024) L128(v2 + o) = u32x4{0, 0, 0, 0};
            if (!dead) {
                const u32 ga = (own ? OFF_XO + (par * 4 + layer) * PSTR : OFF_SP + (q * 4 + layer) * PSTR) + (part ? rank * 65536 : 0) + wave * 2048 + lane * 16;
                const u32 gb = (part ? OFF_LW + gct * 65536 : OFF_SP + (32 + ct) * PSTR) + wave * 2048 + lane * 16;
                auto dma = [&](u32 k, int j) {
                    const u32 l = lbase + L_BIG + (k & 3) * 32768 + wave * 2048 + (j >> 1) * 16384 + (j & 1) * 1024;
                    DMA16((j >> 1 ? gb : ga) + (k < skip ? k : k + 4) * 16384 + (j & 1) * 1024, l, myws);
                };
                auto stage = [&](u32 k) { dma(k, 0); dma(k, 1); dma(k, 2); dma(k, 3); };
                u32x4 fr[2][12];
                auto step = [&](auto B, auto MM, auto RD, auto ST, u32 k) {
                    const u32 sa = L_BIG + (k & 3) * 32768 + rg * 4096 + lane * 16, sb = L_BIG + (k & 3) * 32768 + 16384 + ch * 8192 + lane * 16;
#pragma unroll
                    for (int i = 0; i < 8; ++i) {
                        if (MM.value) {
                            const u32x4 *pa = &fr[B.value ^ 1][(i & 1) * 2], *pb = &fr[B.value ^ 1][4 + (i >> 1) * 2];
                            const i32x8 af = {(int)pa[0][0], (int)pa[0][1], (int)pa[0][2], (int)pa[0][3], (int)pa[1][0], (int)pa[1][1], (int)pa[1][2], (int)pa[1][3]};
                            const i32x8 bf = {(int)pb[0][0], (int)pb[0][1], (int)pb[0][2], (int)pb[0][3], (int)pb[1][0], (int)pb[1][1], (int)pb[1][2], (int)pb[1][3]};
                            acc[i & 1][i >> 1] = __builtin_amdgcn_mfma_scale_f32_16x16x128_f8f6f4(af, bf, acc[i & 1][i >> 1], 1, 1, 0, 0x7F7F7F7F, 0, 0x7F7F7F7F);
                        }
                        __builtin_amdgcn_sched_barrier(0);
                        if (RD.value && i < 6) {
                            const u32 o = i < 2 ? sa + i * 2048 : sb + (i - 2) * 2048;
                            fr[B.value][2 * i] = L128(o);
                            fr[B.value][2 * i + 1] = L128(o + 1024);
                        }
                        if (ST.value && (i & 1)) dma(k + 3, i >> 1);
                        __builtin_amdgcn_sched_barrier(0);
                    }
                };
                constexpr std::integral_constant<int, 0> N{};
                constexpr std::integral_constant<int, 1> Y{};
                stage(0); stage(1); stage(2);
                WAITV(8); BAR();
                step(N, N, Y, Y, 0);
                for (u32 k = 1; k < nk - 3; k += 2) {
                    WAITV(8); BAR();
                    step(Y, Y, Y, Y, k);
                    WAITV(8); BAR();
                    step(N, Y, Y, Y, k + 1);
                }
                WAITV(8); BAR();
                step(Y, Y, Y, N, 1);
                WAITV(4); BAR();
                step(N, Y, Y, N, 2);
                WAITV(0); BAR();
                step(Y, Y, Y, N, 3);
                step(N, Y, N, N, 0);
            }
            FRESH_LANE();
            u32 ticket = 0;
            asm volatile("s_cmp_eq_u32 %4, 0\n\ts_cselect_b64 exec, 1, 0\n\tglobal_atomic_add %0, %1, %2, %3 sc0\n\ts_mov_b64 exec, -1"
                         : "+v"(ticket) : "v"(OFF_Q + (job < njob ? li : 15) * 128), "v"(1u), "s"(myws), "s"(wave) : "memory", "scc");
            if (wave == 0) STAMP(45 + min(round, 3u) * 3);
            const u32 eo = (layer * 128 + rg * 32) * GN + gct * 128 + ch * 64;
#define ROW(g) (f32x4{acc[g >> 2][0][g & 3], acc[g >> 2][1][g & 3], acc[g >> 2][2][g & 3], acc[g >> 2][3][g & 3]})
            if (part) {
                if (!dead) {
#pragma unroll
                    for (int g = 0; g < 8; ++g) out_store(eo, (g >> 2) * 16 + (g & 3), ROW(g));
                    WAITV(0);
                    BAR();
                    if (wave == 1) raise(64 + (gct >> 3), arrive(64 + (gct >> 3)), call);
                }
            } else if (own) {
                const bool fail = dead || uni(__hip_atomic_load((GAS u32*)(myws + OFF_TMO), RLX_AG)) == call;
                const float s2 = fail ? __builtin_nanf("") : sc;
#pragma unroll
                for (int g = 0; g < 8; ++g)
                    out_store(eo, (g >> 2) * 16 + (g & 3), f32x4{silu(acc[g >> 2][0][g & 3], s2), silu(acc[g >> 2][1][g & 3], s2),
                                          silu(acc[g >> 2][2][g & 3], s2), silu(acc[g >> 2][3][g & 3], s2)});
                WAITV(0);
            } else if (!dead) {
                const u32 id = ct * 4 + (layer & 1) * 2 + (wave >> 2), to = OFF_OP + par * OP_PAR + (q * 128 + id) * OHALF;
                u32 esum = 0;
#pragma unroll
                for (int i = 0; i < 8; ++i) esum += (__float_as_uint(acc[i >> 2][i & 3][(i + (i >> 2)) & 3]) >> 19) & 0xFFF;
                const u32 etop = min(max((last(scan64(esum)) + 36536u) >> 13, 40u), 158u);
                const float up = __uint_as_float((285 - etop) << 23), dn = __uint_as_float((etop - 31) << 23);
                u32 d[16], sy[8], Nb[4] = {0, 0, 0, 0}, Q[2] = {0, 0}, hardm = 0, off = 0;
                u32x2 lut[8];
                const u32 vm = to + EXCCAP + 4 * HEADB + (wave & 3) * 4096 + lane * 16;
#pragma unroll
                for (int g = 0; g < 8; ++g) {
                    u32 k[4];
#pragma unroll
                    for (int cc = 0; cc < 4; ++cc) {
                        const float v = acc[g >> 2][cc][g & 3] * up;
                        k[cc] = (u32)cvt_i32(v);
                        off |= __float_as_uint((float)(int)k[cc]) ^ __float_as_uint(v);
                    }
                    d[2 * g] = perm(k[1], k[0], 0x06050201u); d[2 * g + 1] = perm(k[3], k[2], 0x06050201u);
                    const u32 a01 = perm(k[1], k[0], 0x07030400u), a23 = perm(k[3], k[2], 0x07030400u);
                    const u32 F = perm(a23, a01, 0x05040100u), T = perm(a23, a01, 0x07060302u);
                    off |= F & 0x0F0F0F0Fu;
                    const u32 sg = (T >> 4) & 0x08080808u, Tm = T ^ ones(sg);
                    sy[g] = ((Tm >> 2) & 0x1E1E1E1Eu) | ((F >> 7) & 0x01010101u) | ((F << 1) & 0xE0E0E0E0u);
                    Nb[g >> 1] |= ((Tm & 0x07070707u) | sg) << (4 * (g & 1));
                    const u32 x8 = lookup8(sy[g], 0x00000018u, 0x00100008u);
                    lut[g] = L64(L_LUTC + x8);
                    Q[g >> 2] |= g & 3 ? x8 << (8 * (g & 3) - 3) : x8 >> 3;
                    if (g & 1) {
                        ST16(vm + (g >> 1) * 1024, (u32x4{d[2 * g - 2], d[2 * g - 1], d[2 * g], d[2 * g + 1]}), myws, 0);
                        asm volatile("" : "+v"(off), "+v"(sy[g - 1]), "+v"(sy[g]), "+v"(Nb[g >> 1]), "+v"(Q[g >> 2]));
                        __builtin_amdgcn_sched_barrier(0);
                    }
                }
                if (__any(off != 0)) {
                    float up2 = up;
                    asm volatile("" : "+v"(up2));
#pragma unroll
                    for (int i = 0; i < 32; ++i) {
                        const float v = acc[i >> 4][i & 3][(i >> 2) & 3];
                        hardm |= (u32)(__float_as_uint((float)(cvt_i32(v * up2) & ~15) * dn) != __float_as_uint(v)) << i;
                    }
                }
                {
                    const u32 vh = to + EXCCAP + (wave & 3) * HEADB + lane * 16;
                    ST16(vh, (u32x4{Nb[0], Nb[1], Nb[2], Nb[3]}), myws, 512);
                    const u32x4 q4 = {Q[0], Q[1], (u32)__shfl_xor((int)Q[0], 32), (u32)__shfl_xor((int)Q[1], 32)};
                    if (lane < 32) ST16(vh, q4, myws, 0);
                }
                const u32 cnt = (__builtin_popcount(escbits(Q[0])) + __builtin_popcount(escbits(Q[1]))) | (__builtin_popcount(hardm) << 16);
                const u32 inc = scan64(cnt), tot = last(inc);
                const u32 N2 = tot & 0xFFFF, n2 = tot >> 16;
                u32 len = ~0u;
                if (N2 <= OCAP2 && n2 <= HARDCAP) {
                    append(std::integral_constant<int, 8>{}, v2 + ((inc - cnt) & 0xFFFF), sy, lut);
                    len = pack_levels(std::integral_constant<int, 2>{}, N2, ev + n2 * 8, [&](u32 u) { return lookup8(u, 0x00180000u, 0x10000800u); },
                                      [&](u32 u) { return lookup8(u - 0x01010101u, 0x18100008u, 0u); });
                    if (len != ~0u) len += n2 * 8;
                    if (n2) {
                        const u32 dump = L_BIG + wave * 8192;
#pragma unroll
                        for (int g = 0; g < 8; ++g) *(LAS f32x4*)(L + dump + g * 1024 + lane * 16) = ROW(g);
                        u32 o = ev + ((inc - cnt) >> 16) * 8;
                        while (hardm) {
                            const u32 b = __builtin_ctz(hardm);
                            hardm &= hardm - 1;
                            L64(o) = u32x2{lane * 32 + b, L32(dump + (b >> 2) * 1024 + lane * 16 + (b & 3) * 4)};
                            o += 8;
                        }
                    }
                }
                const bool raw = len > OSECCAP || FORCE_RAW & 2;
                if (raw) {
                    len = 0;
                    const u32 vo = OFF_OR + par * OR_PAR + ((q * 128 + id) * 4 + (wave & 3)) * 8192 + lane * 16 + 4096;
                    ST16(vo, ROW(0), myws, -4096); ST16(vo, ROW(1), myws, -3072); ST16(vo, ROW(2), myws, -2048); ST16(vo, ROW(3), myws, -1024);
                    ST16(vo, ROW(4), myws, 0);     ST16(vo, ROW(5), myws, 1024);  ST16(vo, ROW(6), myws, 2048);  ST16(vo, ROW(7), myws, 3072);
                }
                const u32 len16 = (len + 15) >> 4;
                if (lane == 0) L32(C_HDR + wave * 4) = len16 | (raw ? 0x80000000u : (N2 << 8) | (n2 << 19) | ((etop - 31) << 23));
                BAR();
                const u32 hv = lane < 4 ? L32(C_HDR + (wave & 4) * 4 + lane * 4) : 0u;
                const u32 hin = scan64(hv & 0xFF);
                const u32 hoff = 1 + hin - (hv & 0xFF);
                if (lane < len16) {
                    const u32x4 sec = L128(ev + lane * 16);
                    ST16(to + __builtin_amdgcn_readlane(hoff, wave & 3) * 16 + lane * 16, sec, myws, 0);
                }
                if ((wave & 3) == 0 && lane < 4) ST4(to + lane * 4, (hv & ~0xFFu) | hoff, myws);
                WAITV(0);
                BAR();
                if ((wave & 3) == 1 && lane == 0) {
                    const u64 fl = ((u64)((1 + last(hin) + 7) >> 3) << 32) | call;
                    __hip_atomic_store((GAS u64*)(a.ws[q] + OFF_F + ((par * 8 + rank) * 512 + (S_O + id)) * 128), fl, RLX_SYS);
                }
            }
            if (dead) WAITV(0);
            asm volatile("" : "+v"(ticket) :: "memory");
            if (threadIdx.x == 0) {
                u32 nx = njob + ticket - 328 * (call - 1);
                if (job < njob) {
                    nx = 25 + ticket - njob * (call - 1);
                    if (nx >= njob) nx = njob + __hip_atomic_fetch_add((GAS u32*)(myws + OFF_Q + 15 * 128), 1u, RLX_AG) - 328 * (call - 1);
                }
                L32(C_NEXT) = nx;
            }
            BAR();
            job = LPOLL(C_NEXT);
            if (wave == 0) STAMP(46 + min(round, 3u) * 3);
        }
    }
    if (threadIdx.x == 0) __hip_atomic_store((GAS u32*)(myws + OFF_CNT + bid * 128), call, RLX_AG);
}

extern "C" void kernel_launch(int rank, int nranks, ncclComm_t comm,
                              hipStream_t stream,
                              void* const* d_in, const int* in_sizes, int n_in,
                              void* d_out, int out_size,
                              const int* in_dtypes, int out_dtype,
                              void* const* d_out_all,
                              void* const* d_ws_all, size_t ws_bytes) {
    if (nranks != NR || (unsigned)rank >= (unsigned)NR || ws_bytes < WS_NEED || n_in != 4
        || in_sizes[0] != 4096 * KS || in_sizes[1] != KS * GN || out_size != MS * GN) {
        fprintf(stderr, "unexpected problem shape\n");
        return;
    }
    Args a;
    a.x = (const u8*)d_in[0];
    a.w = (const u8*)d_in[1];
    a.sx = (const float*)d_in[2];
    a.sw = (const float*)d_in[3];
    a.out = (float*)d_out;
    for (int p = 0; p < NR; ++p) a.ws[p] = (u8*)d_ws_all[p];
    a.self = a.ws[rank];
    a.rank = rank;
    a.pad = 0;
    fused_gemm_rs<<<NWG, NTHR, 0, stream>>>(a);
    HIP_CHECK(hipGetLastError());
}
```

```cpp
#include <hip/hip_runtime.h>
#include <rccl/rccl.h>
#include <cstdio>
#include <cstdint>
#include <type_traits>

#define HIP_CHECK(expr) do { hipError_t _e = (expr); if (_e != hipSuccess) { \
    fprintf(stderr, "HIP error %s at %s:%d\n", hipGetErrorString(_e), __FILE__, __LINE__); \
    return; } } while (0)

typedef unsigned u32;
typedef unsigned long long u64;
typedef unsigned char u8;
typedef u32 u32x2 __attribute__((ext_vector_type(2)));
typedef u32 u32x4 __attribute__((ext_vector_type(4)));
typedef int i32x8 __attribute__((ext_vector_type(8)));
typedef float f32x4 __attribute__((ext_vector_type(4)));
#define GAS __attribute__((address_space(1)))
#define LAS __attribute__((address_space(3)))

#define FORCE_RAW 0
#define GSLEEP 8
#define PROG 0xFFFF
#define LDELAY 12
#define TPERM 0x72104365u
#define RPERM 0x76543210u
#define NP0 0
#define STAMPS 0

constexpr int WINDOW = 6;
#define XWIN XWIN_
#define NPL_ 2
#define XWIN_ 4
constexpr u32 NPL = NPL_;
constexpr u32 NRL = 5;
constexpr int NR = 8;
constexpr int GN = 16384;
constexpr int KS = 512;
constexpr int MS = 512;
constexpr int NWG = 256;
constexpr int NTHR = 704;
constexpr u32 NOWN = 8 - 2 * NPL, WDIR = 64 - 4 * NRL;
constexpr u32 S_W = 0, S_X = 64, S_RL = 128, S_O = 256, S_EARLY = 384, S_END = 511;

constexpr u32 VARCAP = 5120;
constexpr u32 HALFB  = 2560;
constexpr u32 RUN    = VARCAP + 8 * HALFB;
constexpr u32 CAP2 = 1536, CAP3 = 512, SECCAP = 624;
static_assert(32 + 8 * SECCAP <= VARCAP, "sections fit");
constexpr u32 EXCCAP = 3072;
constexpr u32 HEADB  = 1536;
constexpr u32 OHALF  = EXCCAP + 4 * HEADB + 8 * 2048;
constexpr u32 OCAP2 = 1024, OSECCAP = 752, HARDCAP = 15;
static_assert(16 + 4 * OSECCAP <= EXCCAP && OHALF == RUN, "sections fit");

constexpr u32 OFF_CNT = 0;
constexpr u32 OFF_TMO = 32u << 10;
constexpr u32 OFF_Q   = 40u << 10;
constexpr u32 OFF_XF  = 48u << 10;
constexpr u32 OFF_CT  = 64u << 10;
constexpr u32 OFF_PF  = 96u << 10;
constexpr u32 OFF_F   = 204u << 20;
constexpr u32 OFF_DBG = 256u << 10;
constexpr u32 PSTR    = 32 * 16384 + 4352;
constexpr u32 OFF_XO  = 1u << 20;
constexpr u32 OFF_SP  = 6u << 20;
constexpr u32 OFF_P   = 40u << 20;
constexpr u32 P_PAR   = 11u << 20;
constexpr u32 OFF_RAW = 62u << 20;
constexpr u32 RAW_PAR = 10u << 20;
constexpr u32 OFF_OP  = 82u << 20;
constexpr u32 OP_PAR  = 25u << 20;
constexpr u32 OFF_OR  = 132u << 20;
constexpr u32 OR_PAR  = 32u << 20;
constexpr u32 OFF_LW  = 196u << 20;
constexpr u32 WS_NEED = OFF_F + (1u << 20);
static_assert(OFF_XO + 8 * PSTR <= OFF_SP && OFF_SP + 64 * PSTR <= OFF_P && 448 * RUN <= P_PAR
              && 320 * 32768 <= RAW_PAR && 1024 * OHALF <= OP_PAR && NRL <= 8, "workspace");
constexpr u64 DEADLINE_TICKS = 50000000ull;

constexpr u32 L_LUTD = 0;
constexpr u32 L_LUTC = 1024;
constexpr u32 L_BIG  = 3072;
constexpr u32 L_ZERO = L_BIG + 131072;
constexpr u32 V2S = CAP2 + 16, V3S = CAP3 + 16, EVS = 1040;
constexpr u32 WSCR   = V2S + V3S + EVS;
constexpr u32 L_SCR  = L_ZERO;
constexpr u32 L_CTL  = L_SCR + 8 * WSCR;
constexpr u32 C_WB = L_CTL, C_LANDED = L_CTL + 4, C_DEAD = L_CTL + 8, C_PUB = L_CTL + 12, C_DONE = L_CTL + 16;
constexpr u32 C_HDR = L_CTL + 32, C_START = L_CTL + 64, C_NEXT = L_CTL + 68, C_PUB1 = L_CTL + 72, C_VPOS = L_CTL + 76, C_RL = L_CTL + 80, C_ST = L_CTL + 192, C_ORD = L_CTL + 232, C_E1 = L_CTL + 240;
constexpr u32 C_RDY = L_CTL + 96;
constexpr u32 L_END  = L_CTL + 256;
static_assert(L_BIG + 4 * RUN <= L_ZERO && WSCR % 16 == 0 && L_END % 16 == 0 && L_END <= 160 * 1024, "LDS");
static_assert(CAP2 / 4 + CAP3 / 4 + CAP3 + 16 <= EVS && 8 * HARDCAP + OCAP2 / 4 + CAP3 / 4 + CAP3 + 16 <= EVS, "section scratch");

struct Args {
    const u8* x;
    const u8* w;
    const float* sx;
    const float* sw;
    float* out;
    u8* self;
    u8* ws[NR];
    int rank;
    int pad;
};

#define ST16(voff, data, sbase, imm) \
    asm volatile("global_store_dwordx4 %0, %1, %2 offset:" #imm " sc0 sc1\n\ts_nop 1" \
                 :: "v"(voff), "v"(data), "s"(sbase) : "memory")
#define ST4(voff, data, sbase) \
    asm volatile("global_store_dword %0, %1, %2 sc0 sc1" :: "v"(voff), "v"(data), "s"(sbase) : "memory")
#define DMA16(voff, ldsaddr, sbase) do { u32 keep_; \
    asm volatile("s_mov_b32 %0, m0\n\ts_mov_b32 m0, %2\n\ts_nop 0\n\t" \
                 "global_load_lds_dwordx4 %1, %3 sc0 sc1\n\ts_mov_b32 m0, %0" \
                 : "=&s"(keep_) : "v"(voff), "s"(ldsaddr), "s"(sbase) : "memory"); } while (0)
#define WAITV(n) asm volatile("s_waitcnt vmcnt(%0)" :: "n"(n) : "memory")
#define WAITL() asm volatile("s_waitcnt lgkmcnt(0)" ::: "memory")
#define BAR() asm volatile("s_waitcnt lgkmcnt(0)\n\ts_barrier" ::: "memory")
#define RLX_WG __ATOMIC_RELAXED, __HIP_MEMORY_SCOPE_WORKGROUP
#define RLX_AG __ATOMIC_RELAXED, __HIP_MEMORY_SCOPE_AGENT
#define RLX_SYS __ATOMIC_RELAXED, __HIP_MEMORY_SCOPE_SYSTEM
#define CFENCE() asm volatile("" ::: "memory")

__device__ __forceinline__ u64 rt() { return __builtin_amdgcn_s_memrealtime(); }
__device__ __forceinline__ u32 uni(u32 v) { return __builtin_amdgcn_readfirstlane(v); }
__device__ __forceinline__ u32 perm(u32 hi, u32 lo, u32 sel) { return __builtin_amdgcn_perm(hi, lo, sel); }
__device__ __forceinline__ u32 scan64(u32 v) {
    v += __builtin_amdgcn_update_dpp(0u, v, 0x111, 0xf, 0xf, true);
    v += __builtin_amdgcn_update_dpp(0u, v, 0x112, 0xf, 0xf, true);
    v += __builtin_amdgcn_update_dpp(0u, v, 0x114, 0xf, 0xf, true);
    v += __builtin_amdgcn_update_dpp(0u, v, 0x118, 0xf, 0xf, true);
    v += __builtin_amdgcn_update_dpp(0u, v, 0x142, 0xa, 0xf, false);
    v += __builtin_amdgcn_update_dpp(0u, v, 0x143, 0xc, 0xf, false);
    return v;
}
__device__ __forceinline__ u32 last(u32 v) { return __builtin_amdgcn_readlane(v, 63); }
__device__ __forceinline__ u32 escbits(u32 q) { return ~(q | (q >> 1)) & 0x55555555u; }
__device__ __forceinline__ u32 lookup8(u32 sel, u32 thi, u32 tlo) {
    const u32 r = perm(thi, tlo, sel);
    return __builtin_amdgcn_udot4((r ^ (r >> 2)) & 0x18181818u, 0x40100401u, 0u, false);
}
__device__ __forceinline__ u32 classify8(u32 e4, u32 kadd, u32 thi, u32 tlo) {
    return lookup8(((e4 >> 2) + kadd) & 0x1F1F1F1Fu, thi, tlo);
}
constexpr u32 T13 = 0x00181008u, T2L = 0x00000818u, T2H = 0x00001000u;
__device__ __forceinline__ u32 ones(u32 sel) { return perm(0u, 0x00008000u, sel); }
__device__ __forceinline__ u32 bfi(u32 m, u32 a, u32 b) { return (a & m) | (b & ~m); }
__device__ __forceinline__ u32 rep(u32 b) { return b * 0x01010101u; }
__device__ __forceinline__ u32 blk(u32 r, u32 b) { return (r < 4 ? r : r - b) & 3; }
__device__ __forceinline__ u32 partner(u32 r, u32 b) { return r < 4 ? 4 + ((r + b) & 3) : (r - b) & 3; }
__device__ __forceinline__ float silu(float acc, float sc) {
    const float y = acc * sc;
    const float e = __expf(-__builtin_amdgcn_fmed3f(y, -60.f, 60.f));
    return y * __builtin_amdgcn_rcpf(1.f + e);
}
__device__ __forceinline__ int cvt_i32(float f) { int r; asm("v_cvt_i32_f32 %0, %1" : "=v"(r) : "v"(f)); return r; }

__global__ void __launch_bounds__(NTHR) fused_gemm_rs(Args a) {
    __shared__ __attribute__((aligned(16))) u8 lds_raw[L_END];
    LAS u8* const L = (LAS u8*)lds_raw;
#define L16(off) (*(LAS unsigned short*)(L + (off)))
#define L32(off) (*(LAS u32*)(L + (off)))
#define L64(off) (*(LAS u32x2*)(L + (off)))
#define L128(off) (*(LAS u32x4*)(L + (off)))
#define LPOLL(off) uni(__hip_atomic_load((LAS u32*)(L + (off)), RLX_WG))
    const u32 bid = blockIdx.x;
    u32 lane = threadIdx.x & 63;
#define FRESH_LANE() do { asm volatile("" : "+v"(lane)); __builtin_assume(lane < 64); } while (0)
    const u32 wave = uni(threadIdx.x >> 6);
    const u32 rank = a.rank;
    const bool is_link = (bid >> 3) < 7;
    u8* const myws = a.self;
    const u32 lbase = (u32)(size_t)L;
    const u64 t_begin = rt();
#define STAMP(i)

    const u32 tile = ((bid & 7) | ((bid >> 1) & 8)) * 16 + ((TPERM >> (4 * (bid >> 5))) & 7) * 2 + ((bid >> 3) & 1);
    const u32 xrun = 256 + ((bid >> 1) & 7) * 8 + (7 - (bid >> 5)) * 4 + ((bid >> 4) & 1) * 2 + (bid & 1);
    u32 f[4][4];
    auto fetch_run = [&](u32 id) {
        if (id >= 256) {
            const __amdgpu_buffer_rsrc_t rx = __builtin_amdgcn_make_buffer_rsrc((void*)a.x, 0, 4096 * KS, 0x00020000);
            const u32 vo = (lane & 15) * KS + (lane >> 4) * 32, so = (id - 256) * 64 * KS + (wave >> 1) * 128 + (wave & 1) * 16;
#pragma unroll
            for (int c = 0; c < 4; ++c) {
                const u32x4 v = __builtin_amdgcn_raw_buffer_load_b128(rx, vo, so + c * 16 * KS, 0);
                f[c][0] = v[0]; f[c][1] = v[1]; f[c][2] = v[2]; f[c][3] = v[3];
            }
        } else {
            const __amdgpu_buffer_rsrc_t rw = __builtin_amdgcn_make_buffer_rsrc((void*)a.w, 0, KS * GN, 0x00020000);
            const u32 vo = (lane >> 4) * 32 * GN + (lane & 15) * 4, so = ((wave >> 1) * 128 + (wave & 1) * 16) * GN + id * 64;
#pragma unroll
            for (int r = 0; r < 16; ++r) f[r >> 2][r & 3] = __builtin_amdgcn_raw_buffer_load_b32(rw, vo, so + r * GN, 0);
        }
        __builtin_amdgcn_sched_barrier(0);
    };
    const bool urgent = bid >= 224 || !(tile & 14);
    if (wave < 8) {
        if (!urgent) {
            BAR();
            if (LDELAY) __builtin_amdgcn_s_sleep(LDELAY);
        }
        fetch_run(bid >= 224 ? xrun : tile);
    }

    const u32 cnt_v = __hip_atomic_load((GAS u32*)(myws + OFF_CNT + bid * 128), RLX_AG);
#define GET_CALL const u32 call = uni(cnt_v) + 1, par = call & 1
    u32 spins = 0;
    auto expired = [&](u32 call) -> bool {
        if (++spins & 15) return false;
        return uni(__hip_atomic_load((GAS u32*)(myws + OFF_TMO), RLX_AG)) == call || rt() - t_begin > DEADLINE_TICKS;
    };
    auto give_up = [&](u32 call) {
        if (lane == 0) {
            __hip_atomic_store((GAS u32*)(myws + OFF_TMO), call, RLX_AG);
            __hip_atomic_store((LAS u32*)(L + C_DEAD), 1u, RLX_WG);
        }
    };
    auto lds_init = [&](u32 tid) {
        for (u32 o = L_ZERO + tid * 16; o < L_END; o += 128 * 16) L128(o) = u32x4{0, 0, 0, 0};
        for (u32 x = tid; x < 256; x += 128) {
            u32 d = 0, c = 0x0C0C0C0Cu, n = 0;
#pragma unroll
            for (int b = 0; b < 4; ++b) {
                const u32 code = (x >> (2 * b)) & 3;
                d |= (code ? 4 + code : n) << (8 * b);
                if (!code) c = (c & ~(0xFFu << (8 * n))) | (b << (8 * n));
                n += !code;
            }
            L32(L_LUTD + x * 4) = d;
            L64(L_LUTC + x * 8) = u32x2{c, n};
        }
        BAR();
        __builtin_amdgcn_sched_barrier(0);
    };
    const u32 li = bid & 7, from = (rank + 1 + (bid >> 3)) & 7;
    const bool cross = ((rank ^ from) & 4) != 0;
    const u32 pband = (rank < 4 ? from - rank : rank - from) & 3;
    const u32 nitem = cross ? 88 + NOWN + 3 * NRL + 96 : 24 + WDIR + NOWN + 128;
    auto slot_of = [&](u32 n) -> u32 {
        const u32 ns = cross ? 12 : 8, nh = ns / 2, fj = (from - rank) & 3;
        if (n < ns + 8) {
            if (n >= nh && n < nh + 8) return S_W + n - nh;
            const u32 m = n < nh ? n : n - nh - 8, j = 1 + (m >> 1);
            return S_X + ((rank & 4) | ((rank + j + (!cross && j >= fj)) & 3)) * 8 + (n < nh ? 0 : 2) + (m & 1);
        }
        if (n < ns + 24) return S_W + n - ns;
        if (n < 48) { const u32 m = n - ns - 24, j = m >> 2; return S_X + ((~rank & 4) | ((rank + j + (cross && j >= fj)) & 3)) * 8 + 4 + (m & 3); }
        const u32 nw = 24 + (cross ? 64 : WDIR);
        if (n < nw) {
            const u32 o = n - 24, wf = WDIR & ~7u;
            return S_W + (!cross || o < wf ? o : o < wf + 4 * NRL ? o - wf + WDIR : o - 4 * NRL);
        }
        u32 m = n - nw;
        if (m < NOWN) return S_X + rank * 8 + 2 * NPL + m;
        m -= NOWN;
        if (!cross) return S_O + ((m & ~14u) | ((m & 6) << 1) | ((m & 8) >> 2));
        if (m < 3 * NRL) return S_RL + ((rank + 1 + m % 3) & 3) * 8 + m / 3;
        m -= 3 * NRL;
        const u32 b = (m % 12) >> 2;
        return S_O + (m / 12) * 16 + (b + (b >= pband)) * 4 + (m & 3);
    };
    auto run_id = [&](u32 sl) -> u32 {
        if (sl >= S_X && sl < S_RL) return 256 + sl - S_X;
        const u32 o = sl < S_X ? sl : (16 - NRL + (sl & 7)) * 4 + pband;
        return (o & 3) * 64 + blk(rank, o & 3) * 16 + (o >> 2);
    };
    auto run_src = [&](u32 sl) -> u32 { return sl < S_RL ? from : (rank & 4) | ((sl >> 3) & 3); };
    auto run_pos = [&](u32 sl) -> u32 { return sl < S_RL ? run_id(sl) : 320 + pband * 32 + sl - S_RL; };

    if (wave == 9) {
        const u32 q = (rank + 1 + bid) & 7, t = (bid >> 3) & 3, rg = (RPERM >> (4 * (bid >> 5))) & 7;
        if (q != rank && rg >= 2 * NPL) {
            asm volatile("s_barrier" ::: "memory");
            return;
        }
        const __amdgpu_buffer_rsrc_t rx = __builtin_amdgcn_make_buffer_rsrc((void*)a.x, 0, 4096 * KS, 0x00020000);
        u32x4 v[4][2];
#pragma unroll
        for (int rb = 0; rb < 4; ++rb)
#pragma unroll
            for (int p = 0; p < 2; ++p)
                v[rb][p] = __builtin_amdgcn_raw_buffer_load_b128(rx, (lane & 15) * KS + (lane >> 4) * 32 + p * 16, (q * MS + rg * 64 + rb * 16) * KS + t * 128, 0);
        u8* dst = a.ws[q];
        __builtin_amdgcn_sched_barrier(0);
        asm volatile("s_barrier" ::: "memory");
        __builtin_amdgcn_sched_barrier(0);
        GET_CALL;
        asm volatile("" :: "v"(v[0][0]), "v"(v[0][1]), "v"(v[1][0]), "v"(v[1][1]),
                          "v"(v[2][0]), "v"(v[2][1]), "v"(v[3][0]), "v"(v[3][1]));
        const u32 vo = OFF_XO + (par * 4 + (rg >> 1)) * PSTR + (rank * 4 + t) * 16384 + (rg & 1) * 8192 + lane * 16 + 4096;
        STAMP(4);
#define XW() WAITV(XWIN - 1)
        ST16(vo, v[0][0], dst, -4096); XW(); ST16(vo, v[0][1], dst, -3072); XW();
        ST16(vo, v[1][0], dst, -2048); XW(); ST16(vo, v[1][1], dst, -1024);
        XW();
        ST16(vo, v[2][0], dst, 0);     XW(); ST16(vo, v[2][1], dst, 1024);  XW();
        ST16(vo, v[3][0], dst, 2048);  XW(); ST16(vo, v[3][1], dst, 3072);
        WAITV(0);
        if (lane == 0) __hip_atomic_store((GAS u32*)(dst + OFF_XF + par * 1024 + rg * 128 + (rank * 4 + t) * 4), call, RLX_SYS);
        STAMP(1);
        return;
    }
    if (wave == 10) {
        lds_init(64 + lane);
        GET_CALL;
        if (!is_link) return;
        __builtin_amdgcn_s_setprio(2);
        bool part_ok = false;
        u64 f = 0;
        for (u32 k = 0;; ++k) {
            u32 n = li + 8 * k;
            if (k > 2) {
                while (LPOLL(C_START) < k) __builtin_amdgcn_s_sleep(2);
                u32 tk = 0;
                if (lane == 0) tk = __hip_atomic_fetch_add((GAS u32*)(myws + OFF_Q + (8 + (bid >> 3)) * 128), 1u, RLX_AG);
                n = 24 + uni(tk) - (nitem - 16) * (call - 1);
            }
            u32 lines = 0, sl = S_END;
            if (n < nitem) {
                sl = slot_of(n);
                while (sl >= S_O && !part_ok) {
                    const u32 pf = __hip_atomic_load((GAS u32*)(myws + OFF_PF + (64 + (lane & 15)) * 128), RLX_SYS);
                    part_ok = __all(pf >= call || (lane & 3) == blk(rank, (lane >> 2) & 3));
                    if (!part_ok && expired(call)) break;
                }
                const u32 psl = k == 0 && (lane == 1 || lane == 2) ? slot_of(li + 8) + (lane - 1) * S_EARLY : sl;
                for (;;) {
                    if (k == 1 && __builtin_amdgcn_readlane((u32)f, 1) >= call) {
                        lines = __builtin_amdgcn_readlane((u32)(f >> 32), 1) & 0xFF;
                        break;
                    }
                    f = __hip_atomic_load((GAS u64*)(myws + OFF_F + ((par * 8 + from) * 512 + psl) * 128), RLX_SYS);
                    lines = uni((u32)(f >> 32)) & 0xFF;
                    if (uni((u32)f) >= call && (sl < S_O || part_ok)) break;
                    if (k == 0 && lane == 2 && (u32)f >= call) __hip_atomic_store((LAS u32*)(L + C_E1), 1u, RLX_WG);
                    if (expired(call)) { give_up(call); sl = S_END; break; }
                    __builtin_amdgcn_s_sleep(2);
                }
            } else if (uni(__hip_atomic_load((GAS u32*)(myws + OFF_TMO), RLX_AG)) == call) {
                give_up(call);
            }
            while (LPOLL(C_DONE + (k & 3) * 4) < 8 * (k >> 2)) __builtin_amdgcn_s_sleep(1);
            CFENCE();
            lines = min(max(lines, 1u), (sl < S_O ? VARCAP : EXCCAP) / 128);
            const u32 src = sl >= S_O ? OFF_OP + par * OP_PAR + (rank * 128 + ((sl - S_O) & 127)) * OHALF : OFF_P + par * P_PAR + run_pos(sl) * RUN;
            if (lane == 0)
                __hip_atomic_store((LAS u64*)(L + C_RDY + (k & 7) * 8), ((u64)src << 32) | ((k + 1) << 16) | (sl << 6) | lines, RLX_WG);
            if (sl == S_END) return;
        }
    }
    if (wave == 8) {
        lds_init(lane);
        GET_CALL;
        if (!is_link) return;
        __builtin_amdgcn_s_setprio(3);
        const u8* base = a.ws[from];
        const u32 l16 = lane * 16, a_landed = lbase + C_LANDED;
        int issued = 0;
        auto publish = [&](int v) {
            asm volatile("s_mov_b64 exec, 1\n\tds_max_i32 %0, %1\n\ts_mov_b64 exec, -1" :: "v"(a_landed), "v"(v) : "memory");
        };
        auto pull = [&](u32 goff, u32 loff, u64 mask) {
            asm volatile("s_mov_b32 m0, %1\n\ts_mov_b64 exec, %3\n\ts_waitcnt vmcnt(%4)\n\t"
                         "global_load_lds_dwordx4 %0, %2 sc0 sc1\n\ts_mov_b64 exec, -1"
                         :: "v"(l16), "s"(lbase + loff), "s"(base + goff), "s"(mask), "n"(WINDOW - 1) : "memory");
            publish(issued - (WINDOW - 1));
            ++issued;
        };
        bool early = false;
        for (;;) {
            early = uni(__hip_atomic_load((GAS u32*)(myws + OFF_F + ((par * 8 + from) * 512 + slot_of(li) + S_EARLY) * 128), RLX_SYS)) >= call;
            if (early) break;
            if (expired(call)) { give_up(call); break; }
        }
        CFENCE();
        STAMP(2);
        u32 o0 = VARCAP, o1 = VARCAP;
        while (early && o0 < RUN && !(LPOLL(C_RDY) >> 16)) {
            pull(OFF_P + par * P_PAR + run_pos(slot_of(li)) * RUN + o0, L_BIG + o0, ~0ull);
            o0 += 1024;
        }
        if (o0 == RUN && LPOLL(C_E1)) {
            CFENCE();
            while (o1 < RUN && !(LPOLL(C_RDY) >> 16)) {
                pull(OFF_P + par * P_PAR + run_pos(slot_of(li + 8)) * RUN + o1, L_BIG + RUN + o1, ~0ull);
                o1 += 1024;
            }
        }
        if (lane == 0) __hip_atomic_store((LAS u32*)(L + C_VPOS), (u32)issued + 1, RLX_WG);
        for (u32 m = 0;; ++m) {
            u64 r64;
            auto dealt = [&]() { return uni((u32)(r64 = __hip_atomic_load((LAS u64*)(L + C_RDY + (m & 7) * 8), RLX_WG))) >> 16 == m + 1; };
            if (!dealt()) {
#define STEP(w) if (!dealt()) { WAITV(w); publish(issued - w); }
                STEP(5) STEP(4) STEP(3) STEP(2) STEP(1) STEP(0)
                while (!dealt()) __builtin_amdgcn_s_sleep(1);
            }
            const u32 rdy = uni((u32)r64), g0 = uni((u32)(r64 >> 32));
            CFENCE();
            const u32 sl = (rdy >> 6) & 511;
            if (sl == S_END) break;
            const u32 l0 = L_BIG + (m & 3) * RUN;
            for (u32 o = 0, ln = rdy & 63; ; o += 1024, ln -= 8) {
                pull(g0 + o, l0 + o, ln >= 8 ? ~0ull : (1ull << (8 * ln)) - 1);
                if (ln <= 8) break;
            }
            if (lane == 0) __hip_atomic_store((LAS u32*)(L + C_START), m + 1, RLX_WG);
            for (u32 o = m == 0 ? o0 : m == 1 ? o1 : sl >= S_O ? EXCCAP : VARCAP; o < RUN; o += 1024) pull(g0 + o, l0 + o, ~0ull);
        }
        WAITV(4); publish(issued - 4); WAITV(2); publish(issued - 2); WAITV(0); publish(issued);
        STAMP(21);
        return;
    }

    u32 phase = 0;
    auto wbar = [&]() {
        phase += 8;
        WAITL();
        if (lane == 0) __hip_atomic_fetch_add((LAS u32*)(L + C_WB), 1u, RLX_WG);
        while (LPOLL(C_WB) < phase) __builtin_amdgcn_s_sleep(1);
        CFENCE();
    };
    auto append = [&](auto G, u32 pos, const u32* e4, const u32x2* lut) {
        u32 carry = 0;
#pragma unroll
        for (int i = 0; i < G.value; ++i) {
            const u32 comp = perm(0u, e4[i], lut[i][0]), sh = pos << 3;
            __hip_atomic_fetch_or((LAS u32*)(L + (pos & ~3u)), (comp << (sh & 31)) | carry, RLX_WG);
            carry = __builtin_amdgcn_ubfe(comp, 0u - sh, sh);
            pos += lut[i][1];
        }
        __hip_atomic_fetch_or((LAS u32*)(L + (pos & ~3u)), carry, RLX_WG);
    };
    auto expand = [&](auto G, auto PER, u32 tab, u32 pos, const u32* q, u32* out) {
        u32 sel[G.value], at[G.value], lo[G.value], hi[G.value];
#pragma unroll
        for (int i = 0; i < G.value; ++i) {
            const u32 w = q[i / PER.value], sh = 8 * (i % PER.value);
            const LAS u32* p = (const LAS u32*)(L + (pos & ~3u));
            sel[i] = L32(L_LUTD + ((w >> sh) & 0xFF) * 4);
            lo[i] = p[0]; hi[i] = p[1]; at[i] = pos;
            pos += __builtin_popcount(escbits(w) & (0xFFu << sh));
        }
        __builtin_amdgcn_sched_barrier(0);
#pragma unroll
        for (int i = 0; i < G.value; ++i) out[i] = perm(tab, __builtin_amdgcn_alignbyte(hi[i], lo[i], at[i]), sel[i]);
    };
#define IC(n) std::integral_constant<int, n>{}
    auto panel_of = [&](u32 id) -> u32 { return id >= 256 ? (id - 256) >> 1 : 32 + ((id >> 1) & 7) * 4 + (id >> 6); };
    auto stage_off = [&](u32 id, u32 s, u32 par) -> u32 {
        return ((id - 256) >> 3 == rank ? OFF_XO + (par * 4 + ((id >> 1) & 3)) * PSTR : OFF_SP + panel_of(id) * PSTR) + (s * 4 + (wave >> 1)) * 16384 + (id & 1) * 8192 + (wave & 1) * 1024 + lane * 16 + 4096;
    };
    auto staged = [&](u32 what, u32 call) {
        u32 t;
        asm volatile("s_waitcnt vmcnt(0)\n\ts_mov_b64 exec, 1\n\tds_add_rtn_u32 %0, %1, %2\n\ts_mov_b64 exec, -1\n\ts_waitcnt lgkmcnt(0)"
                     : "=&v"(t) : "v"(lbase + C_ST + (what & 15) * 4), "v"(1u) : "memory");
        if ((uni(t) & 7) == 7 && lane == 0) asm volatile("global_store_dword %0, %1, %2 sc0 sc1" :: "v"(what >> 4), "v"(call), "s"(myws) : "memory");
    };
    const u32 v2 = L_SCR + wave * WSCR, v3 = v2 + V2S, ev = v3 + V3S;
    auto pack_levels = [&](auto NP, u32 N2, u32 e, auto cls2, auto cls3) -> u32 {
        u32 u[2 * NP.value], m = 0;
        u32x2 l2[2 * NP.value];
#pragma unroll
        for (int i = 0; i < NP.value; ++i) {
            const u32x2 v = L64(v2 + lane * (8 * NP.value) + i * 8);
            u[2 * i] = v[0]; u[2 * i + 1] = v[1];
        }
#pragma unroll
        for (int i = 0; i < NP.value; ++i) {
            u32 q8 = cls2(u[2 * i]) | (cls2(u[2 * i + 1]) << 8);
            const int valid = (int)N2 - (int)lane * (8 * NP.value) - i * 8;
            q8 |= (0x5555u * 8) << (2 * min(max(valid, 0), 8));
            if (valid > 0) L16(e + lane * (2 * NP.value) + i * 2) = (unsigned short)(q8 >> 3);
            l2[2 * i] = L64(L_LUTC + (q8 & 0x7F8));
            l2[2 * i + 1] = L64(L_LUTC + ((q8 >> 8) & 0x7F8));
        }
#pragma unroll
        for (int i = 0; i < 2 * NP.value; ++i) m += l2[i][1];
        const u32 in2 = scan64(m);
        const u32 N3 = last(in2);
        if (N3 > CAP3) return ~0u;
        if (m) append(std::integral_constant<int, 2 * NP.value>{}, v3 + in2 - m, u, l2);
        const u32 off3 = ((N2 + 7) >> 3) * 2, off4 = off3 + ((N3 + 7) >> 3) * 2;
        const u32x2 v = L64(v3 + lane * 8);
        const u32 w[2] = {v[0], v[1]};
        u32 q8 = cls3(w[0]) | (cls3(w[1]) << 8);
        const int valid = (int)N3 - (int)lane * 8;
        q8 |= (0x5555u * 8) << (2 * min(max(valid, 0), 8));
        if (valid > 0) L16(e + off3 + lane * 2) = (unsigned short)(q8 >> 3);
        const u32x2 l3[2] = {L64(L_LUTC + (q8 & 0x7F8)), L64(L_LUTC + ((q8 >> 8) & 0x7F8))};
        const u32 m3 = l3[0][1] + l3[1][1];
        const u32 in3 = scan64(m3);
        if (m3) append(std::integral_constant<int, 2>{}, e + off4 + in3 - m3, w, l3);
        return off4 + last(in3);
    };
    auto unpack_levels = [&](auto NP, u32 N2, u32 sec, u32 t2, u32 t3) {
        u32 q[NP.value], n = 0;
#pragma unroll
        for (int i = 0; i < NP.value; ++i) {
            q[i] = L16(sec + lane * (2 * NP.value) + i * 2) | 0x55550000u;
            if (lane * (8 * NP.value) + i * 8 >= N2) q[i] = 0x55555555u;
            n += __builtin_popcount(escbits(q[i]));
        }
        const u32 in2 = scan64(n), N3 = last(in2);
        const u32 off3 = sec + ((N2 + 7) >> 3) * 2, off4 = off3 + ((N3 + 7) >> 3) * 2;
        u32 q3 = L16(off3 + lane * 2) | 0x55550000u;
        if (lane * 8 >= N3) q3 = 0x55555555u;
        const u32 m3 = __builtin_popcount(escbits(q3));
        u32 z[2 * NP.value];
        expand(IC(2), IC(2), t3, off4 + scan64(m3) - m3, &q3, z);
        L64(v3 + lane * 8) = u32x2{z[0], z[1]};
        expand(IC(2 * NP.value), IC(2), t2, v3 + in2 - n, q, z);
#pragma unroll
        for (int i = 0; i < NP.value; ++i) L64(v2 + lane * (8 * NP.value) + i * 8) = u32x2{z[2 * i], z[2 * i + 1]};
    };
    u8* peer = a.ws[lane & 7];
    u32 npub = 0, owed = ~0u;

    for (u32 job = 0; job < (bid >= 192 ? 2u : 1u); ++job) {
        const u32 id = (bid >= 192 && job == (bid >= 224 ? 0u : 1u)) ? xrun : tile;
        const u32 t = wave >> 1, h = wave & 1;
        if (id >= 256 && ((id - 256) >> 3) == rank) {
            if (job == 0) BAR();
            continue;
        }
        if (job) {
            fetch_run(id);
            for (u32 o = lane * 16; o < WSCR; o += 1024) L128(v2 + o) = u32x4{0, 0, 0, 0};
        }
        if (id < 256) {
            u32 g[4][4];
#pragma unroll
            for (int i = 0; i < 4; ++i) {
                const u32 t0 = perm(f[i][1], f[i][0], 0x05010400u), t1 = perm(f[i][1], f[i][0], 0x07030602u);
                const u32 t2 = perm(f[i][3], f[i][2], 0x05010400u), t3 = perm(f[i][3], f[i][2], 0x07030602u);
                g[0][i] = perm(t2, t0, 0x05040100u); g[1][i] = perm(t2, t0, 0x07060302u);
                g[2][i] = perm(t3, t1, 0x05040100u); g[3][i] = perm(t3, t1, 0x07060302u);
            }
#pragma unroll
            for (int i = 0; i < 16; ++i) f[i >> 2][i & 3] = g[i >> 2][i & 3];
        }
        GET_CALL;
        asm volatile("" : "+v"(peer));
        const u32x4 f0 = {f[0][0], f[0][1], f[0][2], f[0][3]}, f1 = {f[1][0], f[1][1], f[1][2], f[1][3]};
        const u32x4 f2 = {f[2][0], f[2][1], f[2][2], f[2][3]}, f3 = {f[3][0], f[3][1], f[3][2], f[3][3]};
        auto keep = [&]() {
            if (id >= 256 && (((id >> 3) ^ rank ^ id) & 4)) return;
            const bool pan = id >= 256 || ((id >> 4) & 3) == blk(rank, id >> 6);
            const u32 vo = pan ? stage_off(id, rank, par) : OFF_LW + (id >> 1) * 65536 + t * 16384 + (id & 1) * 8192 + h * 1024 + lane * 16 + 4096;
            const u32 c = (OFF_PF + (pan ? panel_of(id) * 128 + rank * 8 : (128 + (id >> 1)) * 128) + (id & 1) * 4) * 16 + 8 + job;
            ST16(vo, f0, myws, -4096); ST16(vo, f1, myws, -2048); ST16(vo, f2, myws, 0); ST16(vo, f3, myws, 2048);
            if (is_link) owed = c; else staged(c, call);
        };
        u32 esum = 0;
#pragma unroll
        for (int i = 0; i < 4; ++i) esum = __builtin_amdgcn_udot4(f[i][i] & 0x7C7C7C7Cu, 0x01010101u, esum, false);
        const u32 e0 = ((last(scan64(esum)) + 3686u) >> 12) & 31;
        const u32 runo = OFF_P + par * P_PAR + id * RUN;
        u32x4 Q, Mq;
        u32 S[2];
        u32 e4[16];
        u32x2 lut[16];
        const u32 k1 = rep((33 - e0) & 31);
#pragma unroll
        for (int c = 0; c < 4; ++c) {
            u32 q = 0;
#pragma unroll
            for (int i = 0; i < 4; ++i) {
                e4[4 * c + i] = f[c][i] & 0x7C7C7C7Cu;
                const u32 x8 = classify8(e4[4 * c + i], k1, 0u, T13);
                q |= i ? x8 << (8 * i - 3) : x8 >> 3;
            }
            Q[c] = q;
        }
#pragma unroll
        for (int j = 0; j < 6; ++j) {
            const u32 x = j < 4 ? f[3][j] << 5 : bfi(0x40404040u, f[3][2 * j - 8] >> 1, f[3][2 * j - 7] >> 2);
            const u32 w = bfi(0x60606060u, x, bfi(0x1C1C1C1Cu, __builtin_rotateleft32(f[j >> 1][2 * (j & 1) + 1], 3), f[j >> 1][2 * (j & 1)]));
            if (j < 4) Mq[j] = w; else S[j - 4] = w;
        }
        {
            const u32 vo = runo + VARCAP + wave * HALFB + lane * 16;
            ST16(vo, Q, myws, 0);
            ST16(vo, Mq, myws, 1024);
            const u32x4 s4 = {S[0], S[1], (u32)__shfl_xor((int)S[0], 32), (u32)__shfl_xor((int)S[1], 32)};
            if (lane < 32) ST16(vo, s4, myws, 2048);
        }
        if (urgent && job == 0) BAR();
        u32 slot;
        bool to;
        if (id >= 256) {
            const u32 q = (id - 256) >> 3;
            slot = S_X + id - 256;
            to = lane < 8 && lane != rank && (lane == q ? (id & 7) >= 2 * NPL : !((q ^ lane ^ id) & 4));
        } else {
            slot = S_W + (id & 15) * 4 + (id >> 6);
            to = lane < 8 && lane != rank && ((id >> 4) & 3) == blk(lane, id >> 6) && (slot < WDIR || ((lane ^ rank) & 4));
        }
        if (urgent && job == 0) {
            WAITV(0);
            u32 arrived = 0;
            if (lane == 0) arrived = __hip_atomic_fetch_add((LAS u32*)(L + C_PUB1), 1u, RLX_WG);
            if (uni(arrived) == 7 && to)
                __hip_atomic_store((GAS u32*)(peer + OFF_F + ((par * 8 + rank) * 512 + (slot + S_EARLY)) * 128), call, RLX_SYS);
            STAMP(3);
        }
#pragma unroll
        for (int i = 0; i < 16; ++i) lut[i] = L64(L_LUTC + ((Q[i >> 2] >> (8 * (i & 3))) & 0xFF) * 8);
        u32 n = 0;
#pragma unroll
        for (int c = 0; c < 4; ++c) n += __builtin_popcount(escbits(Q[c]));
        const u32 inc = scan64(n);
        const u32 N2 = last(inc);
        u32 len = 0;
        bool raw = N2 > CAP2 || FORCE_RAW & 1;
        if (!raw) {
            append(std::integral_constant<int, 16>{}, v2 + inc - n, e4, lut);
            const u32 k2 = rep((35 - e0) & 31), k3 = rep((38 - e0) & 31);
            len = pack_levels(std::integral_constant<int, 3>{}, N2, ev, [&](u32 u) { return classify8(u, k2, T2H, T2L); },
                              [&](u32 u) { return classify8(u, k3, 0u, T13); });
            raw = len > SECCAP;
        }
        if (raw) {
            len = 0;
            const u32 vo = OFF_RAW + par * RAW_PAR + (id * 8 + wave) * 4096 + lane * 16;
            ST16(vo, f0, myws, 0); ST16(vo, f1, myws, 1024); ST16(vo, f2, myws, 2048); ST16(vo, f3, myws, 3072);
        }
        const u32 len16 = (len + 15) >> 4;
        if (lane == 0) L32(C_HDR + wave * 4) = len16 | (raw ? 0x80000000u : (N2 << 9) | (e0 << 20));
        wbar();
        const u32 hv = lane < 8 ? L32(C_HDR + lane * 4) : 0u;
        const u32 hin = scan64(hv & 0x1FF);
        const u32 hoff = 2 + hin - (hv & 0x1FF);
        if (lane < len16) {
            const u32x4 sec = L128(ev + lane * 16);
            ST16(runo + __builtin_amdgcn_readlane(hoff, wave) * 16 + lane * 16, sec, myws, 0);
        }
        if (wave == 0 && lane < 8) ST4(runo + lane * 4, (hv & ~0x1FFu) | hoff, myws);
        WAITV(0);
        u32 arrived = 0;
        npub += 8;
        if (lane == 0) arrived = __hip_atomic_fetch_add((LAS u32*)(L + C_PUB), 1u, RLX_WG);
        if (uni(arrived) == npub - 1) {
            const u64 fl = ((u64)((2 + last(hin) + 7) >> 3) << 32) | call;
            if (to) __hip_atomic_store((GAS u64*)(peer + OFF_F + ((par * 8 + rank) * 512 + slot) * 128), fl, RLX_SYS);
            STAMP(22 + job);
        }
        keep();
        if (bid >= 192 && job == 0) wbar();
    }

    GET_CALL;
    const float sc = a.sx[0] * a.sw[0];
    const u32 rg = wave >> 1, ch = wave & 1;
    auto out_store = [&](u32 e, u32 r, f32x4 v) {
        asm volatile("global_store_dwordx4 %0, %1, %2 sc1\n\ts_nop 1"
                     :: "v"((lane >> 4) * (16 * GN) + (lane & 15) * 16), "v"(v), "s"(a.out + e + r * GN) : "memory");
    };

    if (is_link) {
        auto landed = [&](u32 mark) {
            while (LPOLL(C_LANDED) < mark) __builtin_amdgcn_s_sleep(1);
            CFENCE();
        };
        auto done = [&](u32 m) {
            WAITL();
            asm volatile("s_mov_b64 exec, 1\n\tds_add_u32 %0, %1\n\ts_mov_b64 exec, -1" :: "v"(lbase + C_DONE + (m & 3) * 4), "v"(1u) : "memory");
        };
        u32 m = 0, sl, lines, pbase = 0;
        auto settle = [&]() {
            if (owed != ~0u) { staged(owed, call); owed = ~0u; }
        };
        auto next = [&]() {
            u32 rdy = LPOLL(C_RDY + (m & 7) * 8);
            if ((rdy >> 16) != m + 1) {
                settle();
                while (((rdy = LPOLL(C_RDY + (m & 7) * 8)) >> 16) != m + 1) __builtin_amdgcn_s_sleep(1);
            }
            sl = (rdy >> 6) & 511;
            lines = rdy & 63;
            if (sl == S_END) return;
            pbase += (lines + 7) >> 3;
            u32 ahead = 1;
            if (m < 2) {
                while (!(ahead = LPOLL(C_VPOS))) __builtin_amdgcn_s_sleep(1);
                if (m) ahead = max(ahead, 1 + 8 * HALFB / 1024) - 8 * HALFB / 1024;
            }
            if (LPOLL(C_LANDED) < pbase + ahead - 1) settle();
            landed(pbase + ahead - 1);
        };
        for (;; ++m) {
            next();
            if (sl >= S_O) break;
            const u32 id = run_id(sl), s = run_src(sl), vb = L_BIG + (m & 3) * RUN;
            const u8* base = a.ws[s];
            const bool relay = sl >= WDIR && sl < S_X && (sl & 3) != pband;
            const u32 rsl = (from & 3) * 8 + (sl >> 2) - (16 - NRL);
            auto pass_on = [&]() {
                const u32 ro = OFF_P + par * P_PAR + (320 + (sl & 3) * 32 + rsl) * RUN;
                const u32 o0 = wave * 640 + lane * 16, o1 = VARCAP + wave * HALFB + lane * 16;
                if (lane < 40 && o0 < lines * 128) { const u32x4 v = L128(vb + o0); ST16(ro + o0, v, myws, 0); }
                const u32x4 v0 = L128(vb + o1), v1 = L128(vb + o1 + 1024);
                ST16(ro + o1, v0, myws, 0); ST16(ro + o1, v1, myws, 1024);
                if (lane < 32) { const u32x4 v = L128(vb + o1 + 2048); ST16(ro + o1, v, myws, 2048); }
            };
            const u32 hdr = uni(L32(vb + wave * 4));
            u32x4 r[4];
            if (hdr >> 31) {
                const u32 vo = OFF_RAW + par * RAW_PAR + (id * 8 + wave) * 4096 + lane * 16;
                asm volatile("global_load_dwordx4 %0, %4, %5 sc0 sc1\n\tglobal_load_dwordx4 %1, %4, %5 offset:1024 sc0 sc1\n\t"
                             "global_load_dwordx4 %2, %4, %5 offset:2048 sc0 sc1\n\tglobal_load_dwordx4 %3, %4, %5 offset:3072 sc0 sc1\n\t"
                             "s_waitcnt vmcnt(0)"
                             : "=&v"(r[0]), "=&v"(r[1]), "=&v"(r[2]), "=&v"(r[3]) : "v"(vo), "s"(base) : "memory");
            } else {
                const u32 e0 = (hdr >> 20) & 31, N2 = (hdr >> 9) & 0x7FF, sec = vb + (hdr & 0x1FF) * 16;
                const u32 t1 = (((e0 - 1) & 31) << 10) | (e0 << 18) | (((e0 + 1) & 31) << 26);
                const u32 t2 = (((e0 - 2) & 31) << 10) | (((e0 + 2) & 31) << 18) | (((e0 - 3) & 31) << 26);
                const u32 t3 = (((e0 - 6) & 31) << 10) | (((e0 - 5) & 31) << 18) | (((e0 - 4) & 31) << 26);
                unpack_levels(std::integral_constant<int, 3>{}, N2, sec, t2, t3);
                settle();
                landed(pbase + (((wave + 1) * HALFB + 1023) >> 10));
                if (relay) pass_on();
                const u32 fx = vb + VARCAP + wave * HALFB;
                const u32x4 Q = L128(fx + lane * 16), Mq = L128(fx + 1024 + lane * 16);
                const u32x2 S = L64(fx + 2048 + (lane & 31) * 16 + (lane >> 5) * 8);
                const u32 q[4] = {Q[0], Q[1], Q[2], Q[3]};
                u32 n = 0, ex[16];
#pragma unroll
                for (int c = 0; c < 4; ++c) n += __builtin_popcount(escbits(q[c]));
                expand(IC(16), IC(4), t1, v2 + scan64(n) - n, q, ex);
#pragma unroll
                for (int c = 0; c < 4; ++c)
#pragma unroll
                    for (int i = 0; i < 4; ++i) {
                        const u32 w = c < 2 ? Mq[2 * c + (i >> 1)] : S[i >> 1];
                        const u32 sm = c < 3 ? (i & 1 ? __builtin_rotateright32(w, 3) : w) : bfi(0x80808080u, w << (1 + (i & 1)), Mq[i] >> 5);
                        r[c][i] = ex[4 * c + i] | (sm & 0x83838383u);
                    }
            }
            if (relay) {
                if (hdr >> 31) pass_on();
                WAITV(0);
                u32 arrived = 0;
                if (lane == 0) arrived = __hip_atomic_fetch_add((LAS u32*)(L + C_RL + (m & 3) * 4), 1u, RLX_WG);
                if ((uni(arrived) & 7) == 7 && lane == 0)
                    __hip_atomic_store((GAS u64*)(a.ws[partner(rank, sl & 3)] + OFF_F + ((par * 8 + rank) * 512 + (S_RL + rsl)) * 128), ((u64)lines << 32) | call, RLX_SYS);
            }
            done(m);
            settle();
            const u32 vo = stage_off(id, s, par);
            ST16(vo, r[0], myws, -4096); ST16(vo, r[1], myws, -2048); ST16(vo, r[2], myws, 0); ST16(vo, r[3], myws, 2048);
            owed = (OFF_PF + panel_of(id) * 128 + s * 8 + (id & 1) * 4) * 16 + (m & 7);
            pbase += 8 * HALFB / 1024;
        }
        settle();
        const u32 u = wave >> 1, rb = wave & 1;
        const u8* base = a.ws[from];
        while (sl != S_END) {
            const u32 id = sl - S_O, vb = L_BIG + (m & 3) * RUN, band = (id >> 2) & 3;
            const u32 e0 = ((cross ? 256 : 0) + (id & 3) * 64 + (u >> 1) * 32 + rb * 16) * GN
                           + (band * 32 + blk(from, band) * 8 + (id >> 4)) * 128 + (u & 1) * 64;
            f32x4 P[4];
#pragma unroll
            for (int k = 0; k < 4; ++k)
                asm volatile("global_load_dwordx4 %0, %1, %2 sc0 sc1"
                             : "=v"(P[k]) : "v"((lane >> 4) * (16 * GN) + (lane & 15) * 16), "s"(a.out + e0 + k * GN) : "memory");
            landed(pbase);
            const u32 hdr = uni(L32(vb + u * 4));
            const u32 n2 = hdr >> 31 ? 0 : (hdr >> 19) & 15;
            u32x2 hard = {(rb ^ 1) << 4, 0};
            u32 he = 0;
            float hp = 0.f;
            f32x4 r[4];
            if (hdr >> 31) {
                done(m);
                const u32 vo = OFF_OR + par * OR_PAR + ((rank * 128 + id) * 8 + wave) * 4096 + lane * 16;
                asm volatile("global_load_dwordx4 %0, %4, %5 sc0 sc1\n\tglobal_load_dwordx4 %1, %4, %5 offset:1024 sc0 sc1\n\t"
                             "global_load_dwordx4 %2, %4, %5 offset:2048 sc0 sc1\n\tglobal_load_dwordx4 %3, %4, %5 offset:3072 sc0 sc1\n\t"
                             "s_waitcnt vmcnt(0)"
                             : "=&v"(r[0]), "=&v"(r[1]), "=&v"(r[2]), "=&v"(r[3]) : "v"(vo), "s"(base) : "memory");
            } else {
                const u32 sec = vb + (hdr & 0xFF) * 16, N2 = (hdr >> 8) & 0x7FF;
                if (lane < n2) hard = L64(sec + lane * 8);
                if (n2) {
                    he = ((((hard[0] >> 9) & 3) * 4 + ((hard[0] >> 2) & 3)) * GN + ((hard[0] >> 5) & 15) * 4 + (hard[0] & 3)) * 4;
                    asm volatile("global_load_dword %0, %1, %2 sc0 sc1\n\ts_waitcnt vmcnt(0)" : "=&v"(hp) : "v"(he), "s"(a.out + e0) : "memory");
                }
                unpack_levels(std::integral_constant<int, 2>{}, N2, sec + n2 * 8, 0x06030100u, 0x08070500u);
                landed(pbase + (((u + 1) * HEADB + 1023) >> 10));
                const u32 hd = vb + EXCCAP + u * HEADB;
                const u32x2 Q = L64(hd + (lane & 31) * 16 + (lane >> 5) * 8), Nb = L64(hd + 512 + lane * 16 + rb * 8);
                const u32 c0 = __builtin_popcount(escbits(Q[0])), c1 = c0 + __builtin_popcount(escbits(Q[1]));
                const u32 q = rb ? Q[1] : Q[0];
                u32 sy4[4], ft[8];
                expand(IC(4), IC(4), 0x04020000u, v2 + scan64(c1) - c1 + (rb ? c0 : 0), &q, sy4);
#pragma unroll
                for (int g = 0; g < 4; ++g) {
                    const u32 sy = sy4[g], nb = Nb[g >> 1] >> (4 * (g & 1));
                    const u32 T = (((sy << 2) & 0x78787878u) | (nb & 0x07070707u)) ^ ones(nb & 0x08080808u);
                    const u32 F = ((sy << 7) & 0x80808080u) | ((sy >> 1) & 0x70707070u);
                    ft[2 * g] = perm(T, F, 0x05010400u); ft[2 * g + 1] = perm(T, F, 0x07030602u);
                }
                asm volatile("" : "+v"(ft[0]), "+v"(ft[1]), "+v"(ft[2]), "+v"(ft[3]), "+v"(ft[4]), "+v"(ft[5]), "+v"(ft[6]), "+v"(ft[7]));
                landed(pbase + 4 * HEADB / 1024 + (wave + 1) * 2);
                const u32x4 d0 = L128(vb + EXCCAP + 4 * HEADB + wave * 2048 + lane * 16), d1 = L128(vb + EXCCAP + 4 * HEADB + wave * 2048 + 1024 + lane * 16);
                done(m);
                const float dn = __uint_as_float(hdr & 0x7F800000u);
#pragma unroll
                for (int g = 0; g < 4; ++g) {
                    const u32 m0 = g < 2 ? d0[2 * g] : d1[2 * g - 4], m1 = g < 2 ? d0[2 * g + 1] : d1[2 * g - 3];
                    const int k0 = perm(m0, ft[2 * g], 0x01050400u), k1 = perm(m0, ft[2 * g], 0x03070602u);
                    const int k2 = perm(m1, ft[2 * g + 1], 0x01050400u), k3 = perm(m1, ft[2 * g + 1], 0x03070602u);
                    r[g] = f32x4{(float)k0, (float)k1, (float)k2, (float)k3} * dn;
                }
            }
            asm volatile("s_waitcnt vmcnt(0)" : "+v"(P[0]), "+v"(P[1]), "+v"(P[2]), "+v"(P[3]) :: "memory");
#pragma unroll
            for (int g = 0; g < 4; ++g)
                out_store(e0, g, f32x4{silu(r[g][0] + P[g][0], sc), silu(r[g][1] + P[g][1], sc), silu(r[g][2] + P[g][2], sc), silu(r[g][3] + P[g][3], sc)});
            if (n2) {
                WAITV(0);
                if (((hard[0] >> 4) & 1) == rb)
                    asm volatile("global_store_dword %0, %1, %2 sc1" :: "v"(he), "v"(silu(__uint_as_float(hard[1]) + hp, sc)), "s"(a.out + e0) : "memory");
            }
            ++m;
            pbase += (OHALF - EXCCAP) / 1024;
            u32 rdy;
            while (((rdy = LPOLL(C_RDY + (m & 7) * 8)) >> 16) != m + 1) __builtin_amdgcn_s_sleep(1);
            sl = (rdy >> 6) & 511;
            pbase += ((rdy & 63) + 7) >> 3;
        }
        if (LPOLL(C_DEAD)) {
            WAITV(0);
            for (u32 r = wave * 64 + lane; r < MS; r += 512)
                for (u32 c = 0; c < GN; c += 64) a.out[(size_t)r * GN + c] = __builtin_nanf("");
        }
    } else {
        const u32 npart = li < 2 ? NP0 : 64 - NP0 / 3, njob = npart + 48;
        for (u32 job = (bid >> 3) - 7, round = 0; job < njob + 128; round += job >= npart) {
            FRESH_LANE();
            const bool part = job < npart, own = part || job >= njob;
            const u32 n = part ? job + (li < 2 ? NP0 * li : 2 * NP0 + npart * (li - 2)) : own ? job - njob : (job - npart) % 24;
            const u32 band = part ? n / 96 : n & 3;
            u32 q = rank, layer = part ? n & 3 : (n >> 2) & 3, block = blk(rank, band), ti = part ? (n >> 2) & 7 : n >> 4;
            if (part) {
                block = (block + 1 + (n >> 5) % 3) & 3;
            } else if (!own) {
                const u32 bat = ((job < npart + 24 ? 0x76531420u : 0xFEDCBA98u) >> (4 * li)) & 15, j = (n >> 2) % 3;
                q = bat & 1 ? (~rank & 4) | ((partner(rank, band) + 1 + j) & 3) : (rank & 4) | ((rank + 1 + j) & 3);
                layer = (bat & 1) * 2 + (n >= 12);
                ti = bat >> 1;
            }
            const u32 ct = ti * 4 + band, gct = band * 32 + block * 8 + ti;
            u32 nk = part ? 4 : own ? 32 : 28;
            asm volatile("" : "+s"(nk));
            const u32 ls = (lane >> 2) & 7;
            const u32 fa_o = own && (layer < NPL || ls == rank) ? OFF_XF + par * 1024 + layer * 256 + lane * 4
                                                                 : OFF_PF + (q * 4 + layer) * 128 + ls * 8 + (lane >> 5) * 4;
            const u32 fb_o = OFF_PF + (part ? (128 + gct) * 128 : (32 + ct) * 128 + ls * 8) + (lane >> 5) * 4;
            u32 todo = part ? 1u << rank : own ? 0xFF : 0xFF & ~(1u << q);
            u32 ord = 0, nord = 0;
            auto extend = [&](u32 fa, u32 fb) {
                u64 b = __ballot(fa >= call && fb >= call);
                b &= b >> 32; b &= b >> 2; b &= b >> 1;
#pragma unroll
                for (u32 s = 0; s < 8; ++s)
                    if ((todo >> s) & (u32)(b >> (4 * s)) & 1) { ord |= s << (4 * nord); ++nord; todo &= ~(1u << s); }
            };
            auto await = [&](u32 n) {
                while (nord < n) {
                    const u32 fa = __hip_atomic_load((GAS u32*)(myws + fa_o), RLX_SYS), fb = __hip_atomic_load((GAS u32*)(myws + fb_o), RLX_SYS);
                    extend(fa, fb);
                    if (nord >= n) break;
                    if (LPOLL(C_DEAD) || expired(call)) { give_up(call); extend(call, call); }
                    __builtin_amdgcn_s_sleep(GSLEEP);
                }
                if (lane == 0) L64(C_ORD) = u32x2{ord, todo};
            };
            if (wave == 0) await(part || (!own && ((PROG >> (li + (job < npart + 24 ? 0 : 8))) & 1)) ? 1 : nk >> 2);
            BAR();
            bool dead = LPOLL(C_DEAD);
            ord = LPOLL(C_ORD); todo = LPOLL(C_ORD + 4);
            if (wave == 0) STAMP(44 + min(round, 3u) * 3);
            f32x4 acc[2][4];
#pragma unroll
            for (int i = 0; i < 8; ++i) acc[i >> 2][i & 3] = f32x4{0.f, 0.f, 0.f, 0.f};
            for (u32 o = lane * 16; o < WSCR; o += 1024) L128(v2 + o) = u32x4{0, 0, 0, 0};
            if (!dead) {
                const u32 ga = (own ? OFF_XO + (par * 4 + layer) * PSTR : OFF_SP + (q * 4 + layer) * PSTR) + wave * 2048 + lane * 16;
                const u32 gb = (part ? OFF_LW + gct * 65536 - rank * 65536 : OFF_SP + (32 + ct) * PSTR) + wave * 2048 + lane * 16;
                auto dma = [&](u32 k, int j) {
                    const u32 l = lbase + L_BIG + (k & 3) * 32768 + wave * 2048 + (j >> 1) * 16384 + (j & 1) * 1024;
                    DMA16((j >> 1 ? gb : ga) + (((ord >> (k & ~3u)) & 7) * 4 + (k & 3)) * 16384 + (j & 1) * 1024, l, myws);
                };
                auto stage = [&](u32 k) { dma(k, 0); dma(k, 1); dma(k, 2); dma(k, 3); };
                u32x4 fr[2][12];
                auto step = [&](auto B, auto MM, auto RD, auto ST, u32 k) {
                    const u32 sa = L_BIG + (k & 3) * 32768 + rg * 4096 + lane * 16, sb = L_BIG + (k & 3) * 32768 + 16384 + ch * 8192 + lane * 16;
#pragma unroll
                    for (int i = 0; i < 8; ++i) {
                        if (MM.value) {
                            const u32x4 *pa = &fr[B.value ^ 1][(i & 1) * 2], *pb = &fr[B.value ^ 1][4 + (i >> 1) * 2];
                            const i32x8 af = {(int)pa[0][0], (int)pa[0][1], (int)pa[0][2], (int)pa[0][3], (int)pa[1][0], (int)pa[1][1], (int)pa[1][2], (int)pa[1][3]};
                            const i32x8 bf = {(int)pb[0][0], (int)pb[0][1], (int)pb[0][2], (int)pb[0][3], (int)pb[1][0], (int)pb[1][1], (int)pb[1][2], (int)pb[1][3]};
                            acc[i & 1][i >> 1] = __builtin_amdgcn_mfma_scale_f32_16x16x128_f8f6f4(af, bf, acc[i & 1][i >> 1], 1, 1, 0, 0x7F7F7F7F, 0, 0x7F7F7F7F);
                        }
                        __builtin_amdgcn_sched_barrier(0);
                        if (RD.value && i < 6) {
                            const u32 o = i < 2 ? sa + i * 2048 : sb + (i - 2) * 2048;
                            fr[B.value][2 * i] = L128(o);
                            fr[B.value][2 * i + 1] = L128(o + 1024);
                        }
                        if (ST.value && (i & 1)) dma(k + 3, i >> 1);
                        __builtin_amdgcn_sched_barrier(0);
                    }
                };
                constexpr std::integral_constant<int, 0> N{};
                constexpr std::integral_constant<int, 1> Y{};
                stage(0); stage(1); stage(2);
                WAITV(8); BAR();
                step(N, N, Y, Y, 0);
                for (u32 k = 1; k < nk - 3; k += 2) {
                    const bool ask = todo && (k & 3) == 1;
                    if (ask && wave == 0) await((k + 7) >> 2);
                    WAITV(8); BAR();
                    if (ask) { ord = LPOLL(C_ORD); todo = LPOLL(C_ORD + 4); }
                    step(Y, Y, Y, Y, k);
                    WAITV(8); BAR();
                    step(N, Y, Y, Y, k + 1);
                }
                WAITV(8); BAR();
                step(Y, Y, Y, N, 1);
                WAITV(4); BAR();
                step(N, Y, Y, N, 2);
                WAITV(0); BAR();
                step(Y, Y, Y, N, 3);
                step(N, Y, N, N, 0);
                dead = LPOLL(C_DEAD);
            }
            FRESH_LANE();
            u32 ticket = 0;
            asm volatile("s_cmp_eq_u32 %4, 0\n\ts_cselect_b64 exec, 1, 0\n\tglobal_atomic_add %0, %1, %2, %3 sc0\n\ts_mov_b64 exec, -1"
                         : "+v"(ticket) : "v"(OFF_Q + (job < njob ? li : 15) * 128), "v"(1u), "s"(myws), "s"(wave) : "memory", "scc");
            if (wave == 0) STAMP(45 + min(round, 3u) * 3);
            const u32 eo = (layer * 128 + rg * 32) * GN + gct * 128 + ch * 64;
#define ROW(g) (f32x4{acc[g >> 2][0][g & 3], acc[g >> 2][1][g & 3], acc[g >> 2][2][g & 3], acc[g >> 2][3][g & 3]})
            if (part) {
                if (!dead) {
#pragma unroll
                    for (int g = 0; g < 8; ++g) out_store(eo, (g >> 2) * 16 + (g & 3), ROW(g));
                    WAITV(0);
                    BAR();
                    if (threadIdx.x == 64 && __hip_atomic_fetch_add((GAS u32*)(myws + OFF_CT + (64 + (gct >> 3)) * 128), 1u, RLX_AG) == 32 * call - 1)
                        __hip_atomic_store((GAS u32*)(myws + OFF_PF + (64 + (gct >> 3)) * 128), call, RLX_AG);
                }
            } else if (own) {
                const bool fail = dead || uni(__hip_atomic_load((GAS u32*)(myws + OFF_TMO), RLX_AG)) == call;
                const float s2 = fail ? __builtin_nanf("") : sc;
#pragma unroll
                for (int g = 0; g < 8; ++g)
                    out_store(eo, (g >> 2) * 16 + (g & 3), f32x4{silu(acc[g >> 2][0][g & 3], s2), silu(acc[g >> 2][1][g & 3], s2),
                                          silu(acc[g >> 2][2][g & 3], s2), silu(acc[g >> 2][3][g & 3], s2)});
                WAITV(0);
            } else if (!dead) {
                const u32 id = ct * 4 + (layer & 1) * 2 + (wave >> 2), to = OFF_OP + par * OP_PAR + (q * 128 + id) * OHALF;
                u32 esum = 0;
#pragma unroll
                for (int i = 0; i < 8; ++i) esum += (__float_as_uint(acc[i >> 2][i & 3][(i + (i >> 2)) & 3]) >> 19) & 0xFFF;
                const u32 etop = min(max((last(scan64(esum)) + 36536u) >> 13, 40u), 158u);
                const float up = __uint_as_float((285 - etop) << 23), dn = __uint_as_float((etop - 31) << 23);
                u32 d[16], sy[8], Nb[4] = {0, 0, 0, 0}, Q[2] = {0, 0}, hardm = 0, off = 0;
                u32x2 lut[8];
                const u32 vm = to + EXCCAP + 4 * HEADB + (wave & 3) * 4096 + lane * 16;
#pragma unroll
                for (int g = 0; g < 8; ++g) {
                    u32 k[4];
#pragma unroll
                    for (int cc = 0; cc < 4; ++cc) {
                        const float v = acc[g >> 2][cc][g & 3] * up;
                        k[cc] = (u32)cvt_i32(v);
                        off |= __float_as_uint((float)(int)k[cc]) ^ __float_as_uint(v);
                    }
                    d[2 * g] = perm(k[1], k[0], 0x06050201u); d[2 * g + 1] = perm(k[3], k[2], 0x06050201u);
                    const u32 a01 = perm(k[1], k[0], 0x07030400u), a23 = perm(k[3], k[2], 0x07030400u);
                    const u32 F = perm(a23, a01, 0x05040100u), T = perm(a23, a01, 0x07060302u);
                    off |= F & 0x0F0F0F0Fu;
                    const u32 sg = (T >> 4) & 0x08080808u, Tm = T ^ ones(sg);
                    sy[g] = ((Tm >> 2) & 0x1E1E1E1Eu) | ((F >> 7) & 0x01010101u) | ((F << 1) & 0xE0E0E0E0u);
                    Nb[g >> 1] |= ((Tm & 0x07070707u) | sg) << (4 * (g & 1));
                    const u32 x8 = lookup8(sy[g], 0x00000018u, 0x00100008u);
                    lut[g] = L64(L_LUTC + x8);
                    Q[g >> 2] |= g & 3 ? x8 << (8 * (g & 3) - 3) : x8 >> 3;
                    if (g & 1) {
                        ST16(vm + (g >> 1) * 1024, (u32x4{d[2 * g - 2], d[2 * g - 1], d[2 * g], d[2 * g + 1]}), myws, 0);
                        asm volatile("" : "+v"(off), "+v"(sy[g - 1]), "+v"(sy[g]), "+v"(Nb[g >> 1]), "+v"(Q[g >> 2]));
                        __builtin_amdgcn_sched_barrier(0);
                    }
                }
                if (__any(off != 0)) {
                    float up2 = up;
                    asm volatile("" : "+v"(up2));
#pragma unroll
                    for (int i = 0; i < 32; ++i) {
                        const float v = acc[i >> 4][i & 3][(i >> 2) & 3];
                        hardm |= (u32)(__float_as_uint((float)(cvt_i32(v * up2) & ~15) * dn) != __float_as_uint(v)) << i;
                    }
                }
                {
                    const u32 vh = to + EXCCAP + (wave & 3) * HEADB + lane * 16;
                    ST16(vh, (u32x4{Nb[0], Nb[1], Nb[2], Nb[3]}), myws, 512);
                    const u32x4 q4 = {Q[0], Q[1], (u32)__shfl_xor((int)Q[0], 32), (u32)__shfl_xor((int)Q[1], 32)};
                    if (lane < 32) ST16(vh, q4, myws, 0);
                }
                const u32 cnt = (__builtin_popcount(escbits(Q[0])) + __builtin_popcount(escbits(Q[1]))) | (__builtin_popcount(hardm) << 16);
                const u32 inc = scan64(cnt), tot = last(inc);
                const u32 N2 = tot & 0xFFFF, n2 = tot >> 16;
                u32 len = ~0u;
                if (N2 <= OCAP2 && n2 <= HARDCAP) {
                    append(std::integral_constant<int, 8>{}, v2 + ((inc - cnt) & 0xFFFF), sy, lut);
                    len = pack_levels(std::integral_constant<int, 2>{}, N2, ev + n2 * 8, [&](u32 u) { return lookup8(u, 0x00180000u, 0x10000800u); },
                                      [&](u32 u) { return lookup8(u - 0x01010101u, 0x18100008u, 0u); });
                    if (len != ~0u) len += n2 * 8;
                    if (n2) {
                        const u32 dump = L_BIG + wave * 8192;
#pragma unroll
                        for (int g = 0; g < 8; ++g) *(LAS f32x4*)(L + dump + g * 1024 + lane * 16) = ROW(g);
                        u32 o = ev + ((inc - cnt) >> 16) * 8;
                        while (hardm) {
                            const u32 b = __builtin_ctz(hardm);
                            hardm &= hardm - 1;
                            L64(o) = u32x2{lane * 32 + b, L32(dump + (b >> 2) * 1024 + lane * 16 + (b & 3) * 4)};
                            o += 8;
                        }
                    }
                }
                const bool raw = len > OSECCAP || FORCE_RAW & 2;
                if (raw) {
                    len = 0;
                    const u32 vo = OFF_OR + par * OR_PAR + ((q * 128 + id) * 4 + (wave & 3)) * 8192 + lane * 16 + 4096;
                    ST16(vo, ROW(0), myws, -4096); ST16(vo, ROW(1), myws, -3072); ST16(vo, ROW(2), myws, -2048); ST16(vo, ROW(3), myws, -1024);
                    ST16(vo, ROW(4), myws, 0);     ST16(vo, ROW(5), myws, 1024);  ST16(vo, ROW(6), myws, 2048);  ST16(vo, ROW(7), myws, 3072);
                }
                const u32 len16 = (len + 15) >> 4;
                if (lane == 0) L32(C_HDR + wave * 4) = len16 | (raw ? 0x80000000u : (N2 << 8) | (n2 << 19) | ((etop - 31) << 23));
                BAR();
                const u32 hv = lane < 4 ? L32(C_HDR + (wave & 4) * 4 + lane * 4) : 0u;
                const u32 hin = scan64(hv & 0xFF);
                const u32 hoff = 1 + hin - (hv & 0xFF);
                if (lane < len16) {
                    const u32x4 sec = L128(ev + lane * 16);
                    ST16(to + __builtin_amdgcn_readlane(hoff, wave & 3) * 16 + lane * 16, sec, myws, 0);
                }
                if ((wave & 3) == 0 && lane < 4) ST4(to + lane * 4, (hv & ~0xFFu) | hoff, myws);
                WAITV(0);
                BAR();
                if ((wave & 3) == 1 && lane == 0) {
                    const u64 fl = ((u64)((1 + last(hin) + 7) >> 3) << 32) | call;
                    __hip_atomic_store((GAS u64*)(a.ws[q] + OFF_F + ((par * 8 + rank) * 512 + (S_O + id)) * 128), fl, RLX_SYS);
                }
            }
            if (dead) WAITV(0);
            asm volatile("" : "+v"(ticket) :: "memory");
            if (threadIdx.x == 0) {
                u32 nx = njob + ticket - 328 * (call - 1);
                if (job < njob) {
                    nx = 25 + ticket - njob * (call - 1);
                    if (nx >= njob) nx = njob + __hip_atomic_fetch_add((GAS u32*)(myws + OFF_Q + 15 * 128), 1u, RLX_AG) - 328 * (call - 1);
                }
                L32(C_NEXT) = nx;
            }
            BAR();
            job = LPOLL(C_NEXT);
            if (wave == 0) STAMP(46 + min(round, 3u) * 3);
        }
    }
    if (threadIdx.x == 0) __hip_atomic_store((GAS u32*)(myws + OFF_CNT + bid * 128), call, RLX_AG);
}

extern "C" void kernel_launch(int rank, int nranks, ncclComm_t comm,
                              hipStream_t stream,
                              void* const* d_in, const int* in_sizes, int n_in,
                              void* d_out, int out_size,
                              const int* in_dtypes, int out_dtype,
                              void* const* d_out_all,
                              void* const* d_ws_all, size_t ws_bytes) {
    if (nranks != NR || (unsigned)rank >= (unsigned)NR || ws_bytes < WS_NEED || n_in != 4
        || in_sizes[0] != 4096 * KS || in_sizes[1] != KS * GN || out_size != MS * GN) {
        fprintf(stderr, "unexpected problem shape\n");
        return;
    }
    Args a;
    a.x = (const u8*)d_in[0];
    a.w = (const u8*)d_in[1];
    a.sx = (const float*)d_in[2];
    a.sw = (const float*)d_in[3];
    a.out = (float*)d_out;
    for (int p = 0; p < NR; ++p) a.ws[p] = (u8*)d_ws_all[p];
    a.self = a.ws[rank];
    a.rank = rank;
    a.pad = 0;
    fused_gemm_rs<<<NWG, NTHR, 0, stream>>>(a);
    HIP_CHECK(hipGetLastError());
}
```
